```python
import numpy as np

kernel_name = "gpu_gemm_ar_m4096_k4096_n16384_bf16_relu"


ABSMAX_RTOL = 0.008

COMPUTE_DTYPE = "bfloat16"
EPILOGUE = "relu"
IN_QUANT = None
OUT_QUANT = None
IN_DTYPES = {"x": "bfloat16", "w_mat": "bfloat16"}
WORLD_SIZE = 8

def _to_bf16(x):
    u = np.ascontiguousarray(x, dtype=np.float32).view(np.uint32)
    return ((u + 0x7FFF + ((u >> 16) & 1)) & 0xFFFF0000).view(np.float32)

def setup_inputs():

    x = _to_bf16(np.random.randn(4096, 4096).astype(np.float32))
    w_mat = _to_bf16((np.random.randn(4096, 16384) / np.sqrt(4096)).astype(np.float32))
    return {"x": x, "w_mat": w_mat}

def _epilogue(y):
    return np.maximum(y, 0.0).astype(np.float32)

def reference(x, w_mat):
    y = (x @ w_mat).astype(np.float32)
    return _epilogue(y)
```

```cpp
#include <hip/hip_runtime.h>
#include <rccl/rccl.h>
#include <cstdio>

namespace {

typedef unsigned u32;
typedef unsigned long long u64;
typedef u32 u32x4 __attribute__((ext_vector_type(4)));
typedef u32 u32x2 __attribute__((ext_vector_type(2)));
typedef short bf16x8 __attribute__((ext_vector_type(8)));
typedef float f32x4 __attribute__((ext_vector_type(4)));
#define LAS __attribute__((address_space(3)))
#define GAS __attribute__((address_space(1)))

constexpr int NR = 8;
constexpr int NFULL = 16384;
constexpr int NWG = 256;
constexpr int DPP_ = 16;
constexpr int NIN = 192, NOUT = 512;
constexpr int NSEQ = (NIN + NOUT) / DPP_;
constexpr int NTASK = 640;
constexpr u64 DEADLINE = 50000000ull;

constexpr size_t OFF_CTR = 0;
constexpr size_t OFF_TMO = 16384;
constexpr size_t OFF_RDY = 16384 + 256;
constexpr size_t OFF_FL = 32768;
constexpr size_t OFF_DBG = 65536;
constexpr size_t OFF_XA = 1u << 20;
constexpr size_t OFF_WB = OFF_XA + (32u << 20);
constexpr size_t SLOT = 43008;
constexpr size_t OFF_CX = OFF_WB + (16u << 20);
constexpr size_t OFF_CW = OFF_CX + 128 * SLOT;
constexpr size_t OFF_CO = OFF_CW + 512 * SLOT;
constexpr size_t WS_NEED = OFF_CO + 512 * SLOT;

constexpr int L_LUTE = 0;
constexpr int L_LUTC = 2048;
constexpr int L_CTL = 4096;
constexpr int L_RB = 4608;
constexpr int STAGE = 49152;
constexpr int LDS_BYTES = L_RB + 3 * STAGE;
constexpr int TB = 2112;
constexpr int ENC_W = 5248 + 4 * TB;
constexpr int LSLOT = 41984;
constexpr int L_DT = L_RB + 2 * LSLOT;
static_assert(L_DT + 6 * 3 * TB <= LDS_BYTES, "lds");
static_assert(8 * ENC_W <= 3 * STAGE, "lds");
constexpr int C_HDR = 0, C_SIZE = 8, C_CALL = 16, C_DEAD = 17, C_LANDED = 18, C_CLAIM = 19, C_DONE = 20  ,
              C_STORED = 24  , C_FLAG = 64  ;

struct Params {
    const char* x; const char* w; char* out;
    char* ws[NR];
    int rank; int pad;
};

__shared__ __attribute__((aligned(16))) unsigned char smem[LDS_BYTES];

#define CBAR() asm volatile("" ::: "memory")
__device__ __forceinline__ LAS unsigned char* lp(u32 o) { return (LAS unsigned char*)smem + o; }
__device__ __forceinline__ u32 ld8(u32 o) { return *lp(o); }
__device__ __forceinline__ u32 ld32(u32 o) { return *(LAS u32*)lp(o); }
__device__ __forceinline__ u32x2 ld64(u32 o) { return *(LAS u32x2*)lp(o); }
__device__ __forceinline__ u32x4 ld128(u32 o) { return *(LAS u32x4*)lp(o); }
__device__ __forceinline__ void st8(u32 o, u32 v) { *lp(o) = (unsigned char)v; }
__device__ __forceinline__ void st32(u32 o, u32 v) { *(LAS u32*)lp(o) = v; }
__device__ __forceinline__ void st64(u32 o, u32 a, u32 b) { *(LAS u32x2*)lp(o) = (u32x2){a, b}; }
__device__ __forceinline__ void st128(u32 o, u32x4 v) { *(LAS u32x4*)lp(o) = v; }
__device__ __forceinline__ void or32(u32 o, u32 v) { __hip_atomic_fetch_or((LAS u32*)lp(o), v, __ATOMIC_RELAXED, __HIP_MEMORY_SCOPE_WORKGROUP); }
__device__ __forceinline__ volatile LAS u32* ctl(int i) { return (volatile LAS u32*)lp(L_CTL + 4 * i); }
__device__ __forceinline__ u32 uni(u32 v) { return __builtin_amdgcn_readfirstlane(v); }
__device__ __forceinline__ u32 ctl_ld(int i) { return uni(*ctl(i)); }
__device__ __forceinline__ u32 ctl_inc(int i) {
    u32 r; u64 sv;
    asm volatile("s_mov_b64 %1, exec\n\ts_mov_b64 exec, 1\n\tds_add_rtn_u32 %0, %2, %3\n\ts_waitcnt lgkmcnt(0)\n\ts_mov_b64 exec, %1"
                 : "=&v"(r), "=&s"(sv) : "v"((u32)(size_t)lp(L_CTL + 4 * i)), "v"(1u) : "memory");
    return uni(r);
}
#define JOIN() __builtin_amdgcn_wave_barrier()
__device__ __forceinline__ u32 perm(u32 hi, u32 lo, u32 sel) { return __builtin_amdgcn_perm(hi, lo, sel); }
__device__ __forceinline__ u32 bfi(u32 m, u32 a, u32 b) { return (m & a) | (~m & b); }
__device__ __forceinline__ u64 now() { return __builtin_amdgcn_s_memrealtime(); }
__device__ __forceinline__ void wg_barrier() { asm volatile("s_waitcnt lgkmcnt(0)\n\ts_barrier" ::: "memory"); }
__device__ __forceinline__ void vm_drain() { asm volatile("s_waitcnt vmcnt(0)" ::: "memory"); }

__device__ __forceinline__ void gst_agent(void* p, u32x4 v) { asm volatile("global_store_dwordx4 %0, %1, off sc1\n\ts_nop 1" ::"v"(p), "v"(v) : "memory"); }
__device__ __forceinline__ void gst_sys(void* p, u32x4 v) { asm volatile("global_store_dwordx4 %0, %1, off sc0 sc1\n\ts_nop 1" ::"v"(p), "v"(v) : "memory"); }

__device__ __forceinline__ u32 scan(u32 x) {
    x += __builtin_amdgcn_update_dpp(0u, x, 0x111, 0xf, 0xf, false);
    x += __builtin_amdgcn_update_dpp(0u, x, 0x112, 0xf, 0xf, false);
    x += __builtin_amdgcn_update_dpp(0u, x, 0x114, 0xf, 0xf, false);
    x += __builtin_amdgcn_update_dpp(0u, x, 0x118, 0xf, 0xf, false);
    x += __builtin_amdgcn_update_dpp(0u, x, 0x142, 0xa, 0xf, false);
    x += __builtin_amdgcn_update_dpp(0u, x, 0x143, 0xc, 0xf, false);
    return x;
}
__device__ __forceinline__ u32 last(u32 x) { return __builtin_amdgcn_readlane(x, 63); }
__device__ __forceinline__ u32 offsets4(u32 m, u32 (&o)[4]) {
    const u32 c01 = __popc(m & 0xffu) | (__popc(m & 0xff00u) << 16), c23 = __popc(m & 0xff0000u) | (__popc(m >> 24) << 16);
    const u32 s01 = scan(c01), s23 = scan(c23);
    const u32 t01 = last(s01), t23 = last(s23);
    const u32 e01 = s01 - c01, e23 = s23 - c23;
    const u32 b1 = t01 & 0xffffu, b2 = b1 + (t01 >> 16), b3 = b2 + (t23 & 0xffffu);
    o[0] = e01 & 0xffffu; o[1] = (e01 >> 16) + b1; o[2] = (e23 & 0xffffu) + b2; o[3] = (e23 >> 16) + b3;
    return b3 + (t23 >> 16);
}
__device__ __forceinline__ u32 gather8(u32 y0, u32 y1) {
    return __builtin_amdgcn_udot4(y1, 0x80402010u, __builtin_amdgcn_udot4(y0, 0x08040201u, 0u, false), false);
}
__device__ __forceinline__ u32 spread1(u32 n4) { return __umul24(n4, 0x204081u) & 0x01010101u; }
__device__ __forceinline__ u32 spread2(u32 n4) { return __umul24(n4, 0x408102u) & 0x02020202u; }
__device__ __forceinline__ u32 subb(u32 a, u32 brep) {
    return ((a | 0x80808080u) - (brep & 0x7f7f7f7fu)) ^ ((a ^ ~brep) & 0x80808080u);
}
template <int K> __device__ __forceinline__ u32 geb(u32 d) {
    return ((((d & 0x7f7f7f7fu) + (u32)(0x80 - K) * 0x01010101u) | d) >> 7) & 0x01010101u;
}
__device__ __forceinline__ void fetch8(u32 a, u32& s0, u32& s1) {
    const u32 q = a & ~3u;
    const u32 d0 = ld32(q), d1 = ld32(q + 4), d2 = ld32(q + 8);
    s0 = __builtin_amdgcn_alignbyte(d1, d0, a); s1 = __builtin_amdgcn_alignbyte(d2, d1, a);
}
__device__ __forceinline__ void scatter8(u32 a, u32 c0, u32 c1) {
    const u32 q = a & ~3u, s = (a & 3u) * 8u;
    const u64 lo = (((u64)c1 << 32) | c0) << s;
    or32(q, (u32)lo); or32(q + 4, (u32)(lo >> 32)); or32(q + 8, (c1 >> 1) >> (31u - s));
}

__device__ __forceinline__ void dec_l2(u32 l2a, u32 n2, u32 base, bool wide, u32 t2, int lane) {
    const u32 pb = (n2 + 7) >> 3, l2b = l2a + pb, l3 = l2b + pb;
    const u32 tab = (base - 2) | ((base - 1) << 8) | ((base + 3) << 16);
    const u32 top = (base + 5) * 0x01010101u;
    u32 run = 0;
    for (u32 p = 0; p < pb; p += 64) {
        const u32 idx = p + lane;
        const u32 a8 = ld8(l2a + idx), b8 = ld8(l2b + idx), more = a8 & b8;
        const u32 cnt = __popc(more), inc = scan(cnt), o3 = run + inc - cnt;
        run += last(inc);
        const u32 c0 = spread1(a8 & 15u) | spread2(b8 & 15u), c1 = spread1(a8 >> 4) | spread2(b8 >> 4);
        u32 s0, s1;
        if (wide) fetch8(l3 + o3, s0, s1);
        else {
            const u32 bit = l3 * 8u + o3 * 4u, q = (bit >> 5) << 2;
            const u32 xx = __builtin_amdgcn_alignbit(ld32(q + 4), ld32(q), bit);
            const u32 lo = xx & 0x0f0f0f0fu, hi = (xx >> 4) & 0x0f0f0f0fu;
            s0 = perm(hi, lo, 0x05010400u); s1 = perm(hi, lo, 0x07030602u);
            const u32 g0 = ((s0 + 0x0e0e0e0eu) >> 4) & 0x01010101u, g1 = ((s1 + 0x0e0e0e0eu) >> 4) & 0x01010101u;
            s0 = top - s0 - g0 * 6u; s1 = top - s1 - g1 * 6u;
        }
        const u32x2 sel = ld64(L_LUTE + more * 8u);
        st64(t2 + idx * 8u, perm(0u, tab, c0) | perm(s1, s0, sel.x), perm(0u, tab, c1) | perm(s1, s0, sel.y));
    }
    CBAR();
}

__device__ __forceinline__ void dec_input(u32 rec, u32 hdr, bool wide, u32 tmp, char* dst, int lane) {
    const u32 n2 = min((hdr >> 12) & 0xfffu, 2048u), base = hdr >> 24, t2 = tmp + 2 * TB;
    dec_l2(rec + 2560, n2, base, wide, t2, lane);
    const u32 pa = ld32(rec + 2048 + lane * 4), pb = ld32(rec + 2304 + lane * 4), more = pa & pb;
    const u32 tab = base | ((base + 1) << 8) | ((base + 2) << 16);
    u32 o[4]; offsets4(more, o);
#pragma unroll
    for (int j = 0; j < 4; ++j) {
        const u32x2 mb = ld64(rec + j * 512 + lane * 8);
        const u32 c0 = spread1((pa >> (8 * j)) & 15u) | spread2((pb >> (8 * j)) & 15u);
        const u32 c1 = spread1((pa >> (8 * j + 4)) & 15u) | spread2((pb >> (8 * j + 4)) & 15u);
        u32 s0, s1; fetch8(t2 + o[j], s0, s1);
        const u32x2 sel = ld64(L_LUTE + ((more >> (8 * j)) & 255u) * 8u);
        const u32 e0 = perm(0u, tab, c0) | perm(s1, s0, sel.x), e1 = perm(0u, tab, c1) | perm(s1, s0, sel.y);
        const u32 h0 = bfi(0x80808080u, mb.x, e0 >> 1), h1 = bfi(0x80808080u, mb.y, e1 >> 1);
        const u32 l0 = bfi(0x80808080u, e0 << 7, mb.x), l1 = bfi(0x80808080u, e1 << 7, mb.y);
        gst_agent(dst + j * 1024, (u32x4){perm(h0, l0, 0x05010400u), perm(h0, l0, 0x07030602u), perm(h1, l1, 0x05010400u), perm(h1, l1, 0x07030602u)});
    }
}

__device__ __forceinline__ void dec_output(u32 rec, u32 hdr, bool wide, u32 tmp, char* dst, int lane) {
    const u32 n1 = min(hdr & 0xfffu, 2048u), n2 = min((hdr >> 12) & 0xfffu, 2048u), base = hdr >> 24;
    const u32 t1l = tmp, t1h = tmp + TB, t2 = tmp + 2 * TB;
    const u32 pbo = rec + 256 + n1, nb = (n1 + 7) >> 3;
    dec_l2(pbo + nb, n2, base, wide, t2, lane);
    const u32 tab = base | ((base + 1) << 8) | ((base + 2) << 16);
    u32 run = 0;
    for (u32 p = 0; p < nb; p += 64) {
        const u32 idx = p + lane;
        const u32x2 bb = ld64(rec + 256 + idx * 8u);
        const u32 b8 = ld8(pbo + idx);
        const u32 a0 = (bb.x >> 7) & 0x01010101u, a1 = (bb.y >> 7) & 0x01010101u;
        const u32 more = gather8(a0, a1) & b8;
        const u32 cnt = __popc(more), inc = scan(cnt), o2 = run + inc - cnt;
        run += last(inc);
        const u32 c0 = a0 | spread2(b8 & 15u), c1 = a1 | spread2(b8 >> 4);
        u32 s0, s1; fetch8(t2 + o2, s0, s1);
        const u32x2 sel = ld64(L_LUTE + more * 8u);
        const u32 e0 = perm(0u, tab, c0) | perm(s1, s0, sel.x), e1 = perm(0u, tab, c1) | perm(s1, s0, sel.y);
        st64(t1l + idx * 8u, bfi(0x80808080u, e0 << 7, bb.x), bfi(0x80808080u, e1 << 7, bb.y));
        st64(t1h + idx * 8u, (e0 >> 1) & 0x7f7f7f7fu, (e1 >> 1) & 0x7f7f7f7fu);
    }
    CBAR();
    const u32 m = ld32(rec + lane * 4);
    u32 o[4]; offsets4(m, o);
#pragma unroll
    for (int j = 0; j < 4; ++j) {
        u32 l0, l1, h0, h1;
        fetch8(t1l + o[j], l0, l1); fetch8(t1h + o[j], h0, h1);
        const u32x2 sel = ld64(L_LUTE + ((m >> (8 * j)) & 255u) * 8u);
        const u32 L0 = perm(l1, l0, sel.x), L1 = perm(l1, l0, sel.y), H0 = perm(h1, h0, sel.x), H1 = perm(h1, h0, sel.y);
        gst_agent(dst + (size_t)(j >> 1) * (16 * NFULL * 2) + (j & 1) * 64,
                  (u32x4){perm(H0, L0, 0x05010400u), perm(H0, L0, 0x07030602u), perm(H1, L1, 0x05010400u), perm(H1, L1, 0x07030602u)});
    }
}

__device__ __forceinline__ u32 enc_l2(u32 l2a, u32 n2, u32 base, u32 t2, u32 t3, bool& wide, int lane) {
    const u32 pb = (n2 + 7) >> 3, l2b = l2a + pb, l3 = l2b + pb;
    const u32 brep = (base - 2) * 0x01010101u;
    u32 run = 0;
    for (u32 p = 0; p < pb; p += 64) {
        const u32 idx = p + lane;
        const u32x2 e = ld64(t2 + idx * 8u);
        const int left = (int)n2 - (int)(idx * 8u);
        const u32 valid = left >= 8 ? 255u : (left > 0 ? (1u << left) - 1u : 0u);
        const u32 g0 = subb(e.x, brep), g1 = subb(e.y, brep);
        const u32 more = gather8(geb<6>(g0), geb<6>(g1)) & valid;
        const u32 a8 = gather8((g0 ^ (g0 >> 2)) & 0x01010101u, (g1 ^ (g1 >> 2)) & 0x01010101u) | more;
        const u32 b8 = gather8((g0 >> 2) & 0x01010101u, (g1 >> 2) & 0x01010101u) | more;
        if (left > 0) { st8(l2a + idx, a8 & valid); st8(l2b + idx, b8 & valid); }
        const u32 cnt = __popc(more), inc = scan(cnt), o3 = run + inc - cnt;
        run += last(inc);
        const u32x2 sel = ld64(L_LUTC + more * 8u);
        scatter8(t3 + o3, perm(e.y, e.x, sel.x), perm(e.y, e.x, sel.y));
    }
    CBAR();
    const u32 n3 = run;
    bool out = false;
    for (u32 i = lane; i < n3; i += 64) out |= ((base + 5u - ld8(t3 + i)) & 255u) >= 22u;
    wide = __any(out);
    if (wide) {
        for (u32 i = lane; i < n3; i += 64) st8(l3 + i, ld8(t3 + i));
        return l3 + n3;
    }
    const u32 nb3 = (n3 + 1) >> 1;
    for (u32 i = lane; i < nb3; i += 64) {
        const u32 h0 = (base + 5u - ld8(t3 + 2 * i)) & 255u, h1 = (base + 5u - ld8(t3 + 2 * i + 1)) & 255u;
        const u32 q0 = h0 < 2u ? h0 : h0 - 6u, q1 = h1 < 2u ? h1 : h1 - 6u;
        st8(l3 + i, (q0 & 15u) | (q1 << 4));
    }
    return l3 + nb3;
}

__device__ __forceinline__ void zero_lds(u32 a, int bytes, int lane) {
    for (int i = lane * 16; i < bytes; i += 1024) st128(a + i, (u32x4){0u, 0u, 0u, 0u});
}

__device__ __forceinline__ void enc_input(const u32x4 (&v)[4], int wid, int lane) {
    const u32 rec = L_RB + wid * ENC_W, t2 = rec + 5248 + 2 * TB, t3 = t2 + TB;
    zero_lds(t2, 2 * TB, lane);
    u32 e[4][2], sum = 0;
#pragma unroll
    for (int j = 0; j < 4; ++j) {
        const u32 l0 = perm(v[j].y, v[j].x, 0x06040200u), h0 = perm(v[j].y, v[j].x, 0x07050301u);
        const u32 l1 = perm(v[j].w, v[j].z, 0x06040200u), h1 = perm(v[j].w, v[j].z, 0x07050301u);
        e[j][0] = bfi(0x01010101u, l0 >> 7, h0 << 1); e[j][1] = bfi(0x01010101u, l1 >> 7, h1 << 1);
        st64(rec + j * 512 + lane * 8, bfi(0x80808080u, h0, l0), bfi(0x80808080u, h1, l1));
        sum = __builtin_amdgcn_udot4(e[j][0], 0x01010101u, __builtin_amdgcn_udot4(e[j][1], 0x01010101u, sum, false), false);
    }
    const int tot = (int)last(scan(sum));
    const u32 base = (u32)min(max((tot - 369) >> 11, 16), 250);
    const u32 brep = base * 0x01010101u;
    u32 pa = 0, pb = 0, more = 0;
#pragma unroll
    for (int j = 0; j < 4; ++j) {
        const u32 d0 = subb(e[j][0], brep), d1 = subb(e[j][1], brep);
        more |= gather8(geb<3>(d0), geb<3>(d1)) << (8 * j);
        pa |= gather8(d0 & 0x01010101u, d1 & 0x01010101u) << (8 * j);
        pb |= gather8((d0 >> 1) & 0x01010101u, (d1 >> 1) & 0x01010101u) << (8 * j);
    }
    st32(rec + 2048 + lane * 4, pa | more); st32(rec + 2304 + lane * 4, pb | more);
    u32 o[4]; const u32 n2 = offsets4(more, o);
    CBAR();
#pragma unroll
    for (int j = 0; j < 4; ++j) {
        const u32x2 sel = ld64(L_LUTC + ((more >> (8 * j)) & 255u) * 8u);
        scatter8(t2 + o[j], perm(e[j][1], e[j][0], sel.x), perm(e[j][1], e[j][0], sel.y));
    }
    CBAR();
    bool wide; const u32 end = enc_l2(rec + 2560, n2, base, t2, t3, wide, lane);
    if (lane == 0) { *ctl(C_HDR + wid) = (n2 << 12) | (base << 24); *ctl(C_SIZE + wid) = ((end - rec + 15) >> 4) | (wide ? 0x8000u : 0u); }
}

__device__ __forceinline__ void enc_output(const u32x4 (&v)[4], int wid, int lane) {
    const u32 rec = L_RB + wid * ENC_W, t1l = rec + 5248, t1h = t1l + TB, t2 = t1h + TB, t3 = t2 + TB;
    zero_lds(t1l, 4 * TB, lane);
    u32 l[4][2], h[4][2], m = 0, sum = 0;
#pragma unroll
    for (int j = 0; j < 4; ++j) {
        l[j][0] = perm(v[j].y, v[j].x, 0x06040200u); h[j][0] = perm(v[j].y, v[j].x, 0x07050301u);
        l[j][1] = perm(v[j].w, v[j].z, 0x06040200u); h[j][1] = perm(v[j].w, v[j].z, 0x07050301u);
        m |= gather8(geb<1>(l[j][0] | h[j][0]), geb<1>(l[j][1] | h[j][1])) << (8 * j);
        sum = __builtin_amdgcn_udot4(bfi(0x01010101u, l[j][0] >> 7, h[j][0] << 1), 0x01010101u, sum, false);
        sum = __builtin_amdgcn_udot4(bfi(0x01010101u, l[j][1] >> 7, h[j][1] << 1), 0x01010101u, sum, false);
    }
    st32(rec + lane * 4, m);
    u32 o[4]; const u32 n1 = offsets4(m, o);
    const u32 tot = last(scan(sum));
    const u32 base = uni((u32)min(max((int)((float)tot / (float)max(n1, 1u) - 0.18f), 16), 250));
    const u32 brep = base * 0x01010101u;
    CBAR();
#pragma unroll
    for (int j = 0; j < 4; ++j) {
        const u32x2 sel = ld64(L_LUTC + ((m >> (8 * j)) & 255u) * 8u);
        scatter8(t1l + o[j], perm(l[j][1], l[j][0], sel.x), perm(l[j][1], l[j][0], sel.y));
        scatter8(t1h + o[j], perm(h[j][1], h[j][0], sel.x), perm(h[j][1], h[j][0], sel.y));
    }
    CBAR();
    const u32 nb = (n1 + 7) >> 3, pbo = rec + 256 + n1;
    u32 run = 0, pbs = 0;
#pragma unroll
    for (int p = 0; p < 4; ++p) {
        if ((u32)p * 64u < nb) {
            const u32 idx = p * 64 + lane;
            const u32x2 ll = ld64(t1l + idx * 8u), hh = ld64(t1h + idx * 8u);
            const int left = (int)n1 - (int)(idx * 8u);
            const u32 valid = left >= 8 ? 255u : (left > 0 ? (1u << left) - 1u : 0u);
            const u32 e0 = bfi(0x01010101u, ll.x >> 7, hh.x << 1), e1 = bfi(0x01010101u, ll.y >> 7, hh.y << 1);
            const u32 d0 = subb(e0, brep), d1 = subb(e1, brep);
            const u32 g0 = geb<3>(d0), g1 = geb<3>(d1);
            const u32 more = gather8(g0, g1) & valid;
            if (left > 0) st64(rec + 256 + idx * 8u, bfi(0x80808080u, (d0 | g0) << 7, ll.x), bfi(0x80808080u, (d1 | g1) << 7, ll.y));
            pbs |= ((gather8((d0 >> 1) & 0x01010101u, (d1 >> 1) & 0x01010101u) | more) & valid) << (8 * p);
            const u32 cnt = __popc(more), inc = scan(cnt), o2 = run + inc - cnt;
            run += last(inc);
            const u32x2 sel = ld64(L_LUTC + more * 8u);
            scatter8(t2 + o2, perm(e1, e0, sel.x), perm(e1, e0, sel.y));
        }
    }
    CBAR();
#pragma unroll
    for (int p = 0; p < 4; ++p)
        if ((u32)(p * 64 + lane) < nb) st8(pbo + p * 64 + lane, pbs >> (8 * p));
    CBAR();
    bool wide; const u32 end = enc_l2(pbo + nb, run, base, t2, t3, wide, lane);
    if (lane == 0) { *ctl(C_HDR + wid) = n1 | (run << 12) | (base << 24); *ctl(C_SIZE + wid) = ((end - rec + 15) >> 4) | (wide ? 0x8000u : 0u); }
}

__device__ __forceinline__ u32 emit_block(char* slot, int wid, int lane) {
    wg_barrier();
    const u32 sz = lane < 8 ? *ctl(C_SIZE + lane) : 0u;
    const u32 inc = scan(sz & 0x7fffu) + 3u;
    const u32 mysz = __builtin_amdgcn_readlane(sz, wid) & 0x7fffu, myend = __builtin_amdgcn_readlane(inc, wid);
    const u32 total = last(inc);
    if (wid == 0) {
        const u32 en = inc | (sz & 0x8000u);
        u32x4 hv;
        hv.x = __builtin_amdgcn_readlane(en, 0) | (__builtin_amdgcn_readlane(en, 1) << 16);
        hv.y = __builtin_amdgcn_readlane(en, 2) | (__builtin_amdgcn_readlane(en, 3) << 16);
        hv.z = __builtin_amdgcn_readlane(en, 4) | (__builtin_amdgcn_readlane(en, 5) << 16);
        hv.w = __builtin_amdgcn_readlane(en, 6) | (__builtin_amdgcn_readlane(en, 7) << 16);
        if (lane < 2) hv = ld128(L_CTL + 4 * C_HDR + lane * 16);
        if (lane < 3) gst_sys(slot + lane * 16, hv);
    }
    const u32 rec = L_RB + wid * ENC_W;
    char* d = slot + (size_t)(myend - mysz) * 16;
    for (u32 i = lane; i < mysz; i += 64) gst_sys(d + (size_t)i * 16, ld128(rec + i * 16));
    vm_drain();
    wg_barrier();
    return total;
}

__device__ __forceinline__ void flag_store(const Params& P, int dest, int pos, u32 val) {
    u32* f = (u32*)(P.ws[dest] + OFF_FL) + ((P.rank * DPP_ + (pos & 15)) * 64 + (pos >> 4));
    __hip_atomic_store((GAS u32*)f, val, __ATOMIC_RELAXED, __HIP_MEMORY_SCOPE_SYSTEM);
}
__device__ __forceinline__ void flag_all(const Params& P, int pos, u32 val) {
#pragma unroll
    for (int p = 0; p < NR; ++p) if (p != P.rank) flag_store(P, p, pos, val);
}
__device__ __forceinline__ void ready_add(char* ws, int idx) {
    __hip_atomic_fetch_add((GAS u32*)(ws + OFF_RDY + idx * 128), 1u, __ATOMIC_RELAXED, __HIP_MEMORY_SCOPE_AGENT);
}
__device__ __forceinline__ char* ws_of(const Params& P, int r) {
    char* p = P.ws[0];
#pragma unroll
    for (int i = 1; i < NR; ++i) p = r == i ? P.ws[i] : p;
    return p;
}

#define STAMP(i) do {} while (0)
#define STAMPW(i) do {} while (0)

__device__ __forceinline__ void pack_task(const Params& P, char* ws, int task, int wid, int lane) {
    int pos, dest = -1;
    if (task < 16) pos = task;
    else if (task < 272) { pos = 16 + ((task - 16) >> 3); dest = (P.rank + 1 + (task & 7)) & 7; }
    else if (task < 288) pos = 48 + task - 272;
    else if (task < 544) { pos = 64 + ((task - 288) >> 3); dest = (P.rank + 1 + (task & 7)) & 7; }
    else pos = 96 + task - 544;
    u32x4 v[4];
    if (dest < 0) {
        const int xb = pos < 16 ? pos : (pos < 64 ? pos - 32 : pos - 64), pm = xb >> 3, k8 = xb & 7;
        const char* s = P.x + (size_t)(pm * 256 + wid * 32 + (lane & 15)) * 1024 + k8 * 128 + (lane >> 4) * 16;
#pragma unroll
        for (int j = 0; j < 4; ++j) v[j] = *(const GAS u32x4*)(s + (j >> 1) * 16384 + (j & 1) * 64);
        char* d = ws + OFF_XA + (size_t)(pm * 64 + P.rank * 8 + k8) * 32768 + wid * 4096 + lane * 16;
#pragma unroll
        for (int j = 0; j < 4; ++j) gst_agent(d + j * 1024, v[j]);
        enc_input(v, wid, lane);
        const u32 len = emit_block(ws + OFF_CX + (size_t)xb * SLOT, wid, lane);
        if (threadIdx.x == 0) { flag_all(P, pos, (*ctl(C_CALL) << 12) | len); ready_add(ws, pm); }
    } else {
        const int wb = pos < 48 ? pos - 16 : pos - 32, pn = wb >> 2, qq = wb & 3;
        const char* s = P.w + (size_t)(qq * 128 + (wid >> 2) * 64 + (lane >> 2)) * (NFULL * 2) + (dest * 2048 + pn * 128 + (wid & 3) * 32) * 2 + (lane & 3) * 16;
        const u32 tr = L_RB + wid * ENC_W + 5248;
        u32x4 t[4];
#pragma unroll
        for (int i = 0; i < 4; ++i) t[i] = *(const GAS u32x4*)(s + (size_t)i * 16 * NFULL * 2);
#pragma unroll
        for (int i = 0; i < 4; ++i) st128(tr + i * 1024 + lane * 16, t[i]);
        CBAR();
        const u32 ra = tr + (lane >> 4) * 512 + (((lane >> 2) & 3) * 8 + (lane & 3)) * 2;
#pragma unroll
        for (int j = 0; j < 4; ++j) {
            u32 q[8];
#pragma unroll
            for (int e = 0; e < 8; ++e) q[e] = *(LAS unsigned short*)lp(ra + (j & 1) * 2048 + e * 64 + (j >> 1) * 8);
            v[j] = (u32x4){q[0] | (q[1] << 16), q[2] | (q[3] << 16), q[4] | (q[5] << 16), q[6] | (q[7] << 16)};
        }
        CBAR();
        if (dest == P.rank) {
            char* d = ws + OFF_WB + (size_t)(pn * 64 + P.rank * 8 + qq * 2) * 16384 + wid * 4096 + lane * 16;
#pragma unroll
            for (int j = 0; j < 4; ++j) gst_agent(d + j * 1024, v[j]);
            vm_drain();
            wg_barrier();
            if (threadIdx.x == 0) ready_add(ws, 16 + pn);
        } else {
            enc_input(v, wid, lane);
            const u32 len = emit_block(ws + OFF_CW + (size_t)(dest * 64 + wb) * SLOT, wid, lane);
            if (threadIdx.x == 0) {
                u32* f = (u32*)(ws_of(P, dest) + OFF_FL) + ((P.rank * DPP_ + (pos & 15)) * 64 + (pos >> 4));
                __hip_atomic_store((GAS u32*)f, (*ctl(C_CALL) << 12) | len, __ATOMIC_RELAXED, __HIP_MEMORY_SCOPE_SYSTEM);
            }
        }
    }
}

__device__ __forceinline__ void tile_of(int set, int t16, int& pm, int& pn) {
    const int pair = set < 4 ? (set & 1) : (set >> 1), half = set < 4 ? (set >> 1) : (set & 1);
    pm = 2 * pair + (t16 >> 3); pn = 8 * half + (t16 & 7);
}

__device__ __forceinline__ void decoder(const Params& P, char* ws, int src, int d, u32 call, int wid, int lane) {
    const u32 seq = call & 0xfffffu;
    const u64 tstart = now();
    GAS u32* tmo = (GAS u32*)(ws + OFF_TMO);
    if (wid == 1) {
        const GAS u32* fl = (const GAS u32*)(ws + OFF_FL) + (src * DPP_ + d) * 64 + lane;
        const u64 t0 = now();
        for (;;) {
            const u32 f = __hip_atomic_load(fl, __ATOMIC_RELAXED, __HIP_MEMORY_SCOPE_SYSTEM);
            const bool ok = (f >> 12) == seq;
            if (ok) *ctl(C_FLAG + lane) = f | 0x80000000u;
            if (__all(ok || lane >= NSEQ)) break;
            if (uni(__hip_atomic_load(tmo, __ATOMIC_RELAXED, __HIP_MEMORY_SCOPE_AGENT)) == call || now() - t0 > DEADLINE) {
                if (lane == 0) { __hip_atomic_store(tmo, call, __ATOMIC_RELAXED, __HIP_MEMORY_SCOPE_AGENT); *ctl(C_DEAD) = 1; }
                CBAR();
                if (!ok) *ctl(C_FLAG + lane) = 0x80000000u;
                break;
            }
            __builtin_amdgcn_s_sleep(8);
        }
    } else if (wid == 0) {
        const char* sws = ws_of(P, src);
        u32 t0 = 0, t1 = 0, t2 = 0, cur = 0;
        for (int i = 0; i < NSEQ; ++i) {
            const int pos = i * DPP_ + d;
            const u32 need = 8u * (i >> 1);
            u32 f = ctl_ld(C_FLAG + i);
            if (!(f >> 31) || ctl_ld(C_DONE + (i & 1)) < need) {
                vm_drain();
                cur = (u32)i << 6; *ctl(C_LANDED) = cur;
                while (!((f = ctl_ld(C_FLAG + i)) >> 31) || ctl_ld(C_DONE + (i & 1)) < need) {
                    if (now() - tstart > 2 * DEADLINE) break;
                    __builtin_amdgcn_s_sleep(1);
                }
            }
            CBAR();
            STAMPW(2 + i);
            size_t so;
            if (pos < NIN) {
                const bool isx = pos < 16 || (pos >= 48 && pos < 64) || pos >= 96;
                so = isx ? OFF_CX + (size_t)(pos < 16 ? pos : (pos < 64 ? pos - 32 : pos - 64)) * SLOT
                         : OFF_CW + (size_t)(P.rank * 64 + (pos < 48 ? pos - 16 : pos - 32)) * SLOT;
            } else so = OFF_CO + (size_t)(pos - NIN) * SLOT;
            const u32 len = min(f & 0xfffu, (u32)(LSLOT / 16));
            const char* s = sws + so + lane * 16;
            const u32 lb = (u32)(size_t)lp(L_RB + (i & 1) * LSLOT);
            for (u32 k = 0; k * 64u < len; ++k) {
                if (k * 64u + lane < len) {
                    u32 keep;
                    asm volatile("s_mov_b32 %0, m0\n\ts_mov_b32 m0, %2\n\ts_nop 0\n\tglobal_load_lds_dwordx4 %1, off sc0 sc1\n\ts_mov_b32 m0, %0"
                                 : "=&s"(keep) : "v"(s + (size_t)k * 1024), "s"(lb + k * 1024u) : "memory");
                }
                asm volatile("s_waitcnt vmcnt(3)" ::: "memory");
                if (t0 > cur) { cur = t0; *ctl(C_LANDED) = cur; }
                t0 = t1; t1 = t2; t2 = ((u32)i << 6) + k + 1;
            }
        }
        asm volatile("s_waitcnt vmcnt(2)" ::: "memory"); if (t0 > cur) *ctl(C_LANDED) = t0;
        asm volatile("s_waitcnt vmcnt(1)" ::: "memory"); if (t1 > cur) *ctl(C_LANDED) = t1;
        vm_drain();
        *ctl(C_LANDED) = (u32)NSEQ << 6;
        STAMPW(50);
    } else {
        const u32 tmp = L_DT + (wid - 2) * 3 * TB;
        for (;;) {
            const u32 r = ctl_inc(C_CLAIM);
            const u32 i = r >> 3, sub = r & 7u;
            if (i >= (u32)NSEQ) break;
            const u32 slot = L_RB + (i & 1u) * LSLOT;
            while (ctl_ld(C_LANDED) < (i << 6) + 1u) { if (now() - tstart > 2 * DEADLINE) break; __builtin_amdgcn_s_sleep(1); }
            CBAR();
            const u32 hdr = uni(ld32(slot + sub * 4));
            const u32 en = uni(*(LAS unsigned short*)lp(slot + 32 + sub * 2));
            const u32 st = sub ? (uni(*(LAS unsigned short*)lp(slot + 30 + sub * 2)) & 0x7fffu) : 3u;
            const u32 endp = min(((en & 0x7fffu) + 63u) >> 6, 63u);
            while (ctl_ld(C_LANDED) < (i << 6) + endp) { if (now() - tstart > 2 * DEADLINE) break; __builtin_amdgcn_s_sleep(1); }
            CBAR();
            const bool dead = ctl_ld(C_DEAD) != 0;
            const u32 rec = slot + min(st, (u32)(LSLOT / 16)) * 16u;
            const int pos = i * DPP_ + d;
            if (pos < NIN) {
                const bool isx = pos < 16 || (pos >= 48 && pos < 64) || pos >= 96;
                size_t o; int rdy;
                if (isx) { const int xb = pos < 16 ? pos : (pos < 64 ? pos - 32 : pos - 64); rdy = xb >> 3; o = OFF_XA + (size_t)(rdy * 64 + src * 8 + (xb & 7)) * 32768; }
                else { const int wb = pos < 48 ? pos - 16 : pos - 32; rdy = 16 + (wb >> 2); o = OFF_WB + (size_t)((wb >> 2) * 64 + src * 8 + (wb & 3) * 2) * 16384; }
                if (!dead) dec_input(rec, hdr, (en >> 15) != 0, tmp, ws + o + sub * 4096 + lane * 16, lane);
                asm volatile("s_waitcnt lgkmcnt(0)" ::: "memory");
                ctl_inc(C_DONE + (i & 1u));
                vm_drain();
                if (ctl_inc(C_STORED + i) == 7u) { if (lane == 0) ready_add(ws, rdy); JOIN(); }
            } else {
                const int ob = pos - NIN;
                int pm, pn; tile_of(ob >> 5, (ob & 31) >> 1, pm, pn);
                char* dst = P.out + ((size_t)(pm * 256 + (sub >> 1) * 64 + (ob & 1) * 32 + (lane & 15)) * NFULL + src * 2048 + pn * 128 + (sub & 1) * 64 + (lane >> 4) * 8) * 2;
                if (!dead) dec_output(rec, hdr, (en >> 15) != 0, tmp, dst, lane);
                else gst_agent(dst, (u32x4){0x7fc07fc0u, 0x7fc07fc0u, 0x7fc07fc0u, 0x7fc07fc0u});
                asm volatile("s_waitcnt lgkmcnt(0)" ::: "memory");
                ctl_inc(C_DONE + (i & 1u));
            }
        }
    }
}

__device__ __forceinline__ void gemm_tile(const Params& P, char* ws, int set, int t16, u32 call, int wid, int lane) {
    int pm, pn; tile_of(set, t16, pm, pn);
    GAS u32* tmo = (GAS u32*)(ws + OFF_TMO);
    if (threadIdx.x == 0) {
        const GAS u32* rx = (const GAS u32*)(ws + OFF_RDY + pm * 128);
        const GAS u32* rw = (const GAS u32*)(ws + OFF_RDY + (16 + pn) * 128);
        const u64 t0 = now();
        while ((int)(__hip_atomic_load(rx, __ATOMIC_RELAXED, __HIP_MEMORY_SCOPE_AGENT) - call * 64u) < 0 ||
               (int)(__hip_atomic_load(rw, __ATOMIC_RELAXED, __HIP_MEMORY_SCOPE_AGENT) - call * 32u) < 0) {
            if (__hip_atomic_load(tmo, __ATOMIC_RELAXED, __HIP_MEMORY_SCOPE_AGENT) == call || now() - t0 > DEADLINE) {
                __hip_atomic_store(tmo, call, __ATOMIC_RELAXED, __HIP_MEMORY_SCOPE_AGENT);
                break;
            }
            __builtin_amdgcn_s_sleep(16);
        }
        *ctl(C_DEAD) = __hip_atomic_load(tmo, __ATOMIC_RELAXED, __HIP_MEMORY_SCOPE_AGENT) == call;
        __builtin_amdgcn_fence(__ATOMIC_ACQUIRE, "agent");
        vm_drain();
    }
    wg_barrier();
    STAMP(8 + (set >> 3) * 4);
    const char* ga = ws + OFF_XA + (size_t)pm * (2u << 20);
    const char* gb = ws + OFF_WB + (size_t)pn * (1u << 20);
    const u32 voa = wid * 4096 + lane * 16, vob = wid * 2048 + lane * 16;
    const u32 lbase = (u32)(size_t)lp(L_RB);
    const u32 lda = lbase + wid * 4096, ldb = lbase + 32768 + wid * 2048;
#define STAGE_LD(so, kt) do { u32 keep_; asm volatile( \
        "s_mov_b32 %0, m0\n\ts_mov_b32 m0, %5\n\ts_nop 3\n\t" \
        "global_load_lds_dwordx4 %1, %3\n\tglobal_load_lds_dwordx4 %1, %3 offset:1024\n\t" \
        "global_load_lds_dwordx4 %1, %3 offset:2048\n\tglobal_load_lds_dwordx4 %1, %3 offset:3072\n\t" \
        "s_mov_b32 m0, %6\n\ts_nop 0\n\t" \
        "global_load_lds_dwordx4 %2, %4\n\tglobal_load_lds_dwordx4 %2, %4 offset:1024\n\t" \
        "s_mov_b32 m0, %0" : "=&s"(keep_) \
        : "v"(voa), "v"(vob), "s"(ga + (size_t)(kt) * 32768), "s"(gb + (size_t)(kt) * 16384), "s"(lda + (so)), "s"(ldb + (so)) : "memory"); } while (0)
    const u32 ra = L_RB + (wid >> 1) * 8192 + lane * 16, rb = L_RB + 32768 + (wid & 1) * 8192 + lane * 16;
#define FRAGS(A, B, so, kb) do { _Pragma("unroll") for (int q_ = 0; q_ < 4; ++q_) { \
        B[q_] = *(const LAS bf16x8*)lp(rb + (so) + q_ * 2048 + (kb) * 1024); A[q_] = *(const LAS bf16x8*)lp(ra + (so) + q_ * 2048 + (kb) * 1024); } } while (0)
#define MMA(A, B) do { __builtin_amdgcn_s_setprio(1); _Pragma("unroll") for (int m_ = 0; m_ < 4; ++m_) _Pragma("unroll") for (int n_ = 0; n_ < 4; ++n_) \
        acc[m_][n_] = __builtin_amdgcn_mfma_f32_16x16x32_bf16(B[n_], A[m_], acc[m_][n_], 0, 0, 0); __builtin_amdgcn_s_setprio(0); } while (0)
    f32x4 acc[4][4];
#pragma unroll
    for (int m = 0; m < 4; ++m)
#pragma unroll
        for (int n = 0; n < 4; ++n) acc[m][n] = (f32x4){0.f, 0.f, 0.f, 0.f};
    bf16x8 a0[4], b0[4], a1[4], b1[4];
    STAGE_LD(0, 0); STAGE_LD(STAGE, 1); STAGE_LD(2 * STAGE, 2);
    asm volatile("s_waitcnt vmcnt(12)\n\ts_barrier" ::: "memory");
    FRAGS(a0, b0, 0u, 0);
    u32 so = 0;
    for (int kt = 0; kt < 63; ++kt) {
        FRAGS(a1, b1, so, 1);
        MMA(a0, b0);
        if (kt < 62) asm volatile("s_waitcnt vmcnt(6) lgkmcnt(0)\n\ts_barrier" ::: "memory");
        else asm volatile("s_waitcnt vmcnt(0) lgkmcnt(0)\n\ts_barrier" ::: "memory");
        if (kt < 61) STAGE_LD(so, kt + 3);
        so = so == 2 * STAGE ? 0 : so + STAGE;
        FRAGS(a0, b0, so, 0);
        MMA(a1, b1);
    }
    FRAGS(a1, b1, so, 1);
    MMA(a0, b0);
    MMA(a1, b1);
#undef STAGE_LD
#undef FRAGS
#undef MMA
    wg_barrier();
    STAMP(9 + (set >> 3) * 4);
    const bool dead = ctl_ld(C_DEAD) != 0;
    char* dst = P.out + ((size_t)(pm * 256 + (wid >> 1) * 64 + (lane & 15)) * NFULL + P.rank * 2048 + pn * 128 + (wid & 1) * 64 + (lane >> 4) * 8) * 2;
#pragma unroll
    for (int b = 0; b < 2; ++b) {
        u32x4 v[4];
#pragma unroll
        for (int j = 0; j < 4; ++j) {
            const f32x4 p = acc[2 * b + (j >> 1)][2 * (j & 1)], q = acc[2 * b + (j >> 1)][2 * (j & 1) + 1];
            float f[8] = {p[0], p[1], p[2], p[3], q[0], q[1], q[2], q[3]};
            u32 r[4];
#pragma unroll
            for (int e = 0; e < 4; ++e) {
                const float lo = f[2 * e] < 0.f ? 0.f : f[2 * e], hi = f[2 * e + 1] < 0.f ? 0.f : f[2 * e + 1];
                asm("v_cvt_pk_bf16_f32 %0, %1, %2" : "=v"(r[e]) : "v"(lo), "v"(hi));
                r[e] = dead ? 0x7fc07fc0u : (r[e] & 0x7fff7fffu);
            }
            v[j] = (u32x4){r[0], r[1], r[2], r[3]};
            gst_agent(dst + (size_t)(2 * b + (j >> 1)) * (16 * NFULL * 2) + (j & 1) * 64, v[j]);
        }
        enc_output(v, wid, lane);
        const int ob = set * 32 + t16 * 2 + b;
        const u32 len = emit_block(ws + OFF_CO + (size_t)ob * SLOT, wid, lane);
        if (threadIdx.x == 0) flag_all(P, NIN + ob, (call << 12) | len);
        JOIN();
    }
    STAMP(10 + (set >> 3) * 4);
}

__global__ void __launch_bounds__(512, 2) fused_gemm_ar(const Params P) {
    const int tid = threadIdx.x, lane = tid & 63, wid = uni(tid >> 6), blk = blockIdx.x;
    char* ws = ws_of(P, P.rank);
    STAMP(0);
    if (tid < 256) {
        u32 e[2] = {0x0c0c0c0cu, 0x0c0c0c0cu}, c[2] = {0x0c0c0c0cu, 0x0c0c0c0cu};
        u64 ee = 0x0c0c0c0c0c0c0c0cull, cc = ee; int k = 0;
#pragma unroll
        for (int i = 0; i < 8; ++i)
            if ((tid >> i) & 1) { ee ^= (u64)(0x0c ^ k) << (8 * i); cc ^= (u64)(0x0c ^ i) << (8 * k); ++k; }
        e[0] = (u32)ee; e[1] = (u32)(ee >> 32); c[0] = (u32)cc; c[1] = (u32)(cc >> 32);
        st64(L_LUTE + tid * 8, e[0], e[1]); st64(L_LUTC + tid * 8, c[0], c[1]);
    } else if (tid < 256 + 128) {
        u32 v = 0;
        if (tid == 256 + C_CALL) v = __hip_atomic_fetch_add((GAS u32*)(ws + OFF_CTR + blk * 64), 1u, __ATOMIC_RELAXED, __HIP_MEMORY_SCOPE_AGENT) + 1u;
        *ctl(tid - 256) = v;
    }
    wg_barrier();
    const u32 call = ctl_ld(C_CALL);
    const int xcd = blk & 7, idx = blk >> 3;
    const bool is_dec = idx < 14;
    for (int task = blk; task < NTASK; task = task < NWG ? NWG + (blk - 112) : task + (NWG - 112)) {
        pack_task(P, ws, task, wid, lane);
        JOIN();
        if (is_dec) break;
    }
    STAMP(1);
    if (is_dec) {
        const int j = idx >= 7 ? idx - 7 : idx;
        decoder(P, ws, (P.rank + 1 + j) & 7, (idx >= 7 ? 8 : 0) + xcd, call, wid, lane);
    } else if (idx < 30) {
        for (int rr = 0; rr < 2; ++rr) gemm_tile(P, ws, xcd + 8 * rr, idx - 14, call, wid, lane);
    }
}

}

extern "C" void kernel_launch(int rank, int nranks, ncclComm_t comm, hipStream_t stream,
                              void* const* d_in, const int* in_sizes, int n_in,
                              void* d_out, int out_size, const int* in_dtypes, int out_dtype,
                              void* const* d_out_all, void* const* d_ws_all, size_t ws_bytes) {
    if (nranks != NR || (unsigned)rank >= (unsigned)NR || n_in != 2 || ws_bytes < WS_NEED ||
        in_sizes[0] != 4096 * 512 || in_sizes[1] != 512 * NFULL || out_size != 4096 * NFULL) {
        fprintf(stderr, "kernel_launch: unexpected problem shape\n");
        return;
    }
    Params P;
    P.x = (const char*)d_in[0]; P.w = (const char*)d_in[1]; P.out = (char*)d_out;
    for (int p = 0; p < NR; ++p) P.ws[p] = (char*)d_ws_all[p];
    P.rank = rank; P.pad = 0;
    hipLaunchKernelGGL(fused_gemm_ar, dim3(NWG), dim3(512), 0, stream, P);
    hipError_t e = hipGetLastError();
    if (e != hipSuccess) fprintf(stderr, "launch failed: %s\n", hipGetErrorString(e));
}
```

```cpp
#include <hip/hip_runtime.h>
#include <rccl/rccl.h>
#include <cstdio>

namespace {

typedef unsigned u32;
typedef unsigned long long u64;
typedef u32 u32x4 __attribute__((ext_vector_type(4)));
typedef u32 u32x2 __attribute__((ext_vector_type(2)));
typedef short bf16x8 __attribute__((ext_vector_type(8)));
typedef float f32x4 __attribute__((ext_vector_type(4)));
#define LAS __attribute__((address_space(3)))
#define GAS __attribute__((address_space(1)))

constexpr int NR = 8;
constexpr int NFULL = 16384;
constexpr int NWG = 256;
constexpr int DPP_ = 16;
constexpr int NIN = 192, NOUT = 512;
constexpr int NPOS = NIN + NOUT;
constexpr int NDEC = 7 * DPP_;
constexpr int NTASK = 640;
constexpr u64 DEADLINE = 50000000ull;

constexpr size_t OFF_CTR = 0;
constexpr size_t OFF_TMO = 16384;
constexpr size_t OFF_RDY = 16384 + 256;
constexpr size_t OFF_CLM = 24576;
constexpr size_t OFF_FL = 32768;
constexpr size_t OFF_DBG = 65536;
constexpr size_t OFF_XA = 1u << 20;
constexpr size_t OFF_WB = OFF_XA + (32u << 20);
constexpr size_t SLOT = 43008;
constexpr size_t OFF_CX = OFF_WB + (16u << 20);
constexpr size_t OFF_CW = OFF_CX + 128 * SLOT;
constexpr size_t OFF_CO = OFF_CW + 512 * SLOT;
constexpr size_t WS_NEED = OFF_CO + 512 * SLOT;

constexpr int L_LUTE = 0;
constexpr int L_LUTC = 2048;
constexpr int L_CTL = 4096;
constexpr int L_RB = 4608;
constexpr int STAGE = 49152;
constexpr int LDS_BYTES = L_RB + 3 * STAGE;
constexpr int TB = 2112;
constexpr int ENC_W = 5248 + 3 * TB;
constexpr int LSLOT = 41984;
constexpr int L_DT = L_RB + 2 * LSLOT;
static_assert(L_DT + 6 * 3 * TB <= LDS_BYTES, "lds");
static_assert(8 * ENC_W <= 3 * STAGE, "lds");
constexpr int C_HDR = 0, C_SIZE = 8, C_CALL = 16, C_DEAD = 17, C_LANDED = 18, C_CLAIM = 19, C_DONE = 20  ,
              C_NBLK = 22, C_PROG = 23, C_STORED = 24  , C_QUEUE = 40  ;

struct Params {
    const char* x; const char* w; char* out;
    char* ws[NR];
    int rank; int pad;
};

__shared__ __attribute__((aligned(16))) unsigned char smem[LDS_BYTES];

#define CBAR() asm volatile("" ::: "memory")
__device__ __forceinline__ LAS unsigned char* lp(u32 o) { return (LAS unsigned char*)smem + o; }
__device__ __forceinline__ u32 ld8(u32 o) { return *lp(o); }
__device__ __forceinline__ u32 ld32(u32 o) { return *(LAS u32*)lp(o); }
__device__ __forceinline__ u32x2 ld64(u32 o) { return *(LAS u32x2*)lp(o); }
__device__ __forceinline__ u32x4 ld128(u32 o) { return *(LAS u32x4*)lp(o); }
__device__ __forceinline__ void st8(u32 o, u32 v) { *lp(o) = (unsigned char)v; }
__device__ __forceinline__ void st32(u32 o, u32 v) { *(LAS u32*)lp(o) = v; }
__device__ __forceinline__ void st64(u32 o, u32 a, u32 b) { *(LAS u32x2*)lp(o) = (u32x2){a, b}; }
__device__ __forceinline__ void st128(u32 o, u32x4 v) { *(LAS u32x4*)lp(o) = v; }
__device__ __forceinline__ void or32(u32 o, u32 v) { __hip_atomic_fetch_or((LAS u32*)lp(o), v, __ATOMIC_RELAXED, __HIP_MEMORY_SCOPE_WORKGROUP); }
__device__ __forceinline__ volatile LAS u32* ctl(int i) { return (volatile LAS u32*)lp(L_CTL + 4 * i); }
__device__ __forceinline__ u32 uni(u32 v) { return __builtin_amdgcn_readfirstlane(v); }
__device__ __forceinline__ u32 ctl_ld(int i) { return uni(*ctl(i)); }
__device__ __forceinline__ u32 ctl_inc(int i) {
    u32 r; u64 sv;
    asm volatile("s_mov_b64 %1, exec\n\ts_mov_b64 exec, 1\n\tds_add_rtn_u32 %0, %2, %3\n\ts_waitcnt lgkmcnt(0)\n\ts_mov_b64 exec, %1"
                 : "=&v"(r), "=&s"(sv) : "v"((u32)(size_t)lp(L_CTL + 4 * i)), "v"(1u) : "memory");
    return uni(r);
}
#define JOIN() __builtin_amdgcn_wave_barrier()
__device__ __forceinline__ u32 perm(u32 hi, u32 lo, u32 sel) { return __builtin_amdgcn_perm(hi, lo, sel); }
__device__ __forceinline__ u32 bfi(u32 m, u32 a, u32 b) { return (m & a) | (~m & b); }
__device__ __forceinline__ u64 now() { return __builtin_amdgcn_s_memrealtime(); }
__device__ __forceinline__ void wg_barrier() { asm volatile("s_waitcnt lgkmcnt(0)\n\ts_barrier" ::: "memory"); }
__device__ __forceinline__ void vm_drain() { asm volatile("s_waitcnt vmcnt(0)" ::: "memory"); }

__device__ __forceinline__ void gst_agent(void* p, u32x4 v) { asm volatile("global_store_dwordx4 %0, %1, off sc1\n\ts_nop 1" ::"v"(p), "v"(v) : "memory"); }
__device__ __forceinline__ void gst_sys(void* p, u32x4 v) { asm volatile("global_store_dwordx4 %0, %1, off sc0 sc1\n\ts_nop 1" ::"v"(p), "v"(v) : "memory"); }

__device__ __forceinline__ u32 scan(u32 x) {
    x += __builtin_amdgcn_update_dpp(0u, x, 0x111, 0xf, 0xf, false);
    x += __builtin_amdgcn_update_dpp(0u, x, 0x112, 0xf, 0xf, false);
    x += __builtin_amdgcn_update_dpp(0u, x, 0x114, 0xf, 0xf, false);
    x += __builtin_amdgcn_update_dpp(0u, x, 0x118, 0xf, 0xf, false);
    x += __builtin_amdgcn_update_dpp(0u, x, 0x142, 0xa, 0xf, false);
    x += __builtin_amdgcn_update_dpp(0u, x, 0x143, 0xc, 0xf, false);
    return x;
}
__device__ __forceinline__ u32 last(u32 x) { return __builtin_amdgcn_readlane(x, 63); }
__device__ __forceinline__ u32 offsets4(u32 m, u32 (&o)[4]) {
    const u32 c01 = __popc(m & 0xffu) | (__popc(m & 0xff00u) << 16), c23 = __popc(m & 0xff0000u) | (__popc(m >> 24) << 16);
    const u32 s01 = scan(c01), s23 = scan(c23);
    const u32 t01 = last(s01), t23 = last(s23);
    const u32 e01 = s01 - c01, e23 = s23 - c23;
    const u32 b1 = t01 & 0xffffu, b2 = b1 + (t01 >> 16), b3 = b2 + (t23 & 0xffffu);
    o[0] = e01 & 0xffffu; o[1] = (e01 >> 16) + b1; o[2] = (e23 & 0xffffu) + b2; o[3] = (e23 >> 16) + b3;
    return b3 + (t23 >> 16);
}
__device__ __forceinline__ u32 gather8(u32 y0, u32 y1) {
    return __builtin_amdgcn_udot4(y1, 0x80402010u, __builtin_amdgcn_udot4(y0, 0x08040201u, 0u, false), false);
}
__device__ __forceinline__ u32 spread1(u32 n4) { return __umul24(n4, 0x204081u) & 0x01010101u; }
__device__ __forceinline__ u32 spread2(u32 n4) { return __umul24(n4, 0x408102u) & 0x02020202u; }
__device__ __forceinline__ u32 subb(u32 a, u32 brep) {
    return ((a | 0x80808080u) - (brep & 0x7f7f7f7fu)) ^ ((a ^ ~brep) & 0x80808080u);
}
template <int K> __device__ __forceinline__ u32 geb(u32 d) {
    return ((((d & 0x7f7f7f7fu) + (u32)(0x80 - K) * 0x01010101u) | d) >> 7) & 0x01010101u;
}
__device__ __forceinline__ void fetch8(u32 a, u32& s0, u32& s1) {
    const u32 q = a & ~3u;
    const u32 d0 = ld32(q), d1 = ld32(q + 4), d2 = ld32(q + 8);
    s0 = __builtin_amdgcn_alignbyte(d1, d0, a); s1 = __builtin_amdgcn_alignbyte(d2, d1, a);
}
__device__ __forceinline__ void scatter8(u32 a, u32 c0, u32 c1) {
    const u32 q = a & ~3u, s = (a & 3u) * 8u;
    const u64 lo = (((u64)c1 << 32) | c0) << s;
    const u32 w0 = (u32)lo, w1 = (u32)(lo >> 32), w2 = (c1 >> 1) >> (31u - s);
    if (w0) or32(q, w0);
    if (w1) or32(q + 4, w1);
    if (w2) or32(q + 8, w2);
}
__device__ __forceinline__ void zero_lds(u32 a, u32 bytes, int lane) {
    for (u32 i = lane * 16; i < bytes; i += 1024) st128(a + i, (u32x4){0u, 0u, 0u, 0u});
}

__device__ __forceinline__ void dec_l2(u32 l2a, u32 n2, u32 base, bool wide, u32 t2, int lane) {
    const u32 pb = (n2 + 7) >> 3, l2b = l2a + pb, l3 = l2b + pb;
    const u32 tab = (base - 2) | ((base - 1) << 8) | ((base + 3) << 16);
    const u32 top = (base + 5) * 0x01010101u;
    u32 run = 0;
    for (u32 p = 0; p < pb; p += 64) {
        const u32 idx = p + lane;
        const u32 a8 = ld8(l2a + idx), b8 = ld8(l2b + idx), more = a8 & b8;
        const u32 cnt = __popc(more), inc = scan(cnt), o3 = run + inc - cnt;
        run += last(inc);
        const u32 c0 = spread1(a8 & 15u) | spread2(b8 & 15u), c1 = spread1(a8 >> 4) | spread2(b8 >> 4);
        u32 s0, s1;
        if (wide) fetch8(l3 + o3, s0, s1);
        else {
            const u32 bit = l3 * 8u + o3 * 4u, q = (bit >> 5) << 2;
            const u32 xx = __builtin_amdgcn_alignbit(ld32(q + 4), ld32(q), bit);
            const u32 lo = xx & 0x0f0f0f0fu, hi = (xx >> 4) & 0x0f0f0f0fu;
            s0 = perm(hi, lo, 0x05010400u); s1 = perm(hi, lo, 0x07030602u);
            const u32 g0 = ((s0 + 0x0e0e0e0eu) >> 4) & 0x01010101u, g1 = ((s1 + 0x0e0e0e0eu) >> 4) & 0x01010101u;
            s0 = top - s0 - g0 * 6u; s1 = top - s1 - g1 * 6u;
        }
        const u32x2 sel = ld64(L_LUTE + more * 8u);
        st64(t2 + idx * 8u, perm(0u, tab, c0) | perm(s1, s0, sel.x), perm(0u, tab, c1) | perm(s1, s0, sel.y));
    }
    CBAR();
}

__device__ __forceinline__ void dec_input(u32 rec, u32 hdr, bool wide, u32 tmp, char* dst, int lane) {
    const u32 n2 = min((hdr >> 12) & 0xfffu, 2048u), base = hdr >> 24, t2 = tmp + 2 * TB;
    dec_l2(rec + 2560, n2, base, wide, t2, lane);
    const u32 pa = ld32(rec + 2048 + lane * 4), pb = ld32(rec + 2304 + lane * 4), more = pa & pb;
    const u32 tab = base | ((base + 1) << 8) | ((base + 2) << 16);
    u32 o[4]; offsets4(more, o);
#pragma unroll
    for (int j = 0; j < 4; ++j) {
        const u32x2 mb = ld64(rec + j * 512 + lane * 8);
        const u32 c0 = spread1((pa >> (8 * j)) & 15u) | spread2((pb >> (8 * j)) & 15u);
        const u32 c1 = spread1((pa >> (8 * j + 4)) & 15u) | spread2((pb >> (8 * j + 4)) & 15u);
        u32 s0, s1; fetch8(t2 + o[j], s0, s1);
        const u32x2 sel = ld64(L_LUTE + ((more >> (8 * j)) & 255u) * 8u);
        const u32 e0 = perm(0u, tab, c0) | perm(s1, s0, sel.x), e1 = perm(0u, tab, c1) | perm(s1, s0, sel.y);
        const u32 h0 = bfi(0x80808080u, mb.x, e0 >> 1), h1 = bfi(0x80808080u, mb.y, e1 >> 1);
        const u32 l0 = bfi(0x80808080u, e0 << 7, mb.x), l1 = bfi(0x80808080u, e1 << 7, mb.y);
        gst_agent(dst + j * 1024, (u32x4){perm(h0, l0, 0x05010400u), perm(h0, l0, 0x07030602u), perm(h1, l1, 0x05010400u), perm(h1, l1, 0x07030602u)});
    }
}

__device__ __forceinline__ void dec_output(u32 rec, u32 hdr, bool wide, u32 tmp, char* dst, int lane) {
    const u32 n1 = min(hdr & 0xfffu, 2048u), n2 = min((hdr >> 12) & 0xfffu, 2048u), base = hdr >> 24;
    const u32 t1l = tmp, t1h = tmp + TB, t2 = tmp + 2 * TB;
    const u32 pbo = rec + 256 + n1, nb = (n1 + 7) >> 3;
    dec_l2(pbo + nb, n2, base, wide, t2, lane);
    const u32 tab = base | ((base + 1) << 8) | ((base + 2) << 16);
    u32 run = 0;
    for (u32 p = 0; p < nb; p += 64) {
        const u32 idx = p + lane;
        const u32x2 bb = ld64(rec + 256 + idx * 8u);
        const u32 b8 = ld8(pbo + idx);
        const u32 a0 = (bb.x >> 7) & 0x01010101u, a1 = (bb.y >> 7) & 0x01010101u;
        const u32 more = gather8(a0, a1) & b8;
        const u32 cnt = __popc(more), inc = scan(cnt), o2 = run + inc - cnt;
        run += last(inc);
        const u32 c0 = a0 | spread2(b8 & 15u), c1 = a1 | spread2(b8 >> 4);
        u32 s0, s1; fetch8(t2 + o2, s0, s1);
        const u32x2 sel = ld64(L_LUTE + more * 8u);
        const u32 e0 = perm(0u, tab, c0) | perm(s1, s0, sel.x), e1 = perm(0u, tab, c1) | perm(s1, s0, sel.y);
        st64(t1l + idx * 8u, bfi(0x80808080u, e0 << 7, bb.x), bfi(0x80808080u, e1 << 7, bb.y));
        st64(t1h + idx * 8u, (e0 >> 1) & 0x7f7f7f7fu, (e1 >> 1) & 0x7f7f7f7fu);
    }
    CBAR();
    const u32 m = ld32(rec + lane * 4);
    u32 o[4]; offsets4(m, o);
#pragma unroll
    for (int j = 0; j < 4; ++j) {
        u32 l0, l1, h0, h1;
        fetch8(t1l + o[j], l0, l1); fetch8(t1h + o[j], h0, h1);
        const u32x2 sel = ld64(L_LUTE + ((m >> (8 * j)) & 255u) * 8u);
        const u32 L0 = perm(l1, l0, sel.x), L1 = perm(l1, l0, sel.y), H0 = perm(h1, h0, sel.x), H1 = perm(h1, h0, sel.y);
        gst_agent(dst + (size_t)(j >> 1) * (16 * NFULL * 2) + (j & 1) * 64,
                  (u32x4){perm(H0, L0, 0x05010400u), perm(H0, L0, 0x07030602u), perm(H1, L1, 0x05010400u), perm(H1, L1, 0x07030602u)});
    }
}

__device__ __forceinline__ u32 enc_l2(u32 l2a, u32 n2, u32 base, u32 t2, bool& wide, int lane) {
    const u32 pb = (n2 + 7) >> 3, l2b = l2a + pb, l3 = l2b + pb;
    const u32 top = (base + 5) * 0x01010101u;
    u32 run;
    wide = false;
    for (;;) {
        if (lane < 16) st8(l3 + lane, 0u);
        zero_lds((l3 + 15u) & ~15u, (wide ? n2 : n2 >> 1) + 8u, lane);
        CBAR();
        u32 out = 0;
        run = 0;
        for (u32 p = 0; p < pb; p += 64) {
            const u32 idx = p + lane;
            const u32x2 e = ld64(t2 + idx * 8u);
            const int left = (int)n2 - (int)(idx * 8u);
            const u32 valid = left >= 8 ? 255u : (left > 0 ? (1u << left) - 1u : 0u);
            const u32 h0 = subb(top, e.x), h1 = subb(top, e.y);
            const u32 n0 = ~geb<8>(h0) & (h0 >> 1), n1 = ~geb<8>(h1) & (h1 >> 1);
            const u32 more = ~gather8(n0 & 0x01010101u, n1 & 0x01010101u) & valid;
            const u32 a8 = gather8((h0 >> 2) & ~h0 & 0x01010101u, (h1 >> 2) & ~h1 & 0x01010101u) | more;
            const u32 b8 = ~gather8((h0 >> 2) & 0x01010101u, (h1 >> 2) & 0x01010101u) | more;
            if (left > 0) { st8(l2a + idx, a8 & valid); st8(l2b + idx, b8 & valid); }
            const u32 cnt = __popc(more), inc = scan(cnt), o3 = run + inc - cnt;
            run += last(inc);
            const u32x2 sel = ld64(L_LUTC + more * 8u);
            if (wide) scatter8(l3 + o3, perm(e.y, e.x, sel.x), perm(e.y, e.x, sel.y));
            else {
                u32 c0 = perm(h1, h0, sel.x), c1 = perm(h1, h0, sel.y);
                out |= geb<22>(c0) | geb<22>(c1);
                c0 -= (((c0 + 0x08080808u) >> 4) & 0x01010101u) * 6u; c1 -= (((c1 + 0x08080808u) >> 4) & 0x01010101u) * 6u;
                const u32 xx = perm(c1 | (c1 >> 4), c0 | (c0 >> 4), 0x06040200u);
                const u32 bit = l3 * 8u + o3 * 4u, q = (bit >> 5) << 2;
                const u64 w = (u64)xx << (bit & 31u);
                if ((u32)w) or32(q, (u32)w);
                if ((u32)(w >> 32)) or32(q + 4, (u32)(w >> 32));
            }
        }
        CBAR();
        if (wide || !__any(out != 0)) break;
        wide = true;
    }
    return l3 + (wide ? run : (run + 1) >> 1);
}

__device__ __forceinline__ void enc_input(const u32x4 (&v)[4], int wid, int lane) {
    const u32 rec = L_RB + wid * ENC_W, t2 = rec + 5248 + 2 * TB;
    u32 e[4][2], sum = 0;
#pragma unroll
    for (int j = 0; j < 4; ++j) {
        const u32 l0 = perm(v[j].y, v[j].x, 0x06040200u), h0 = perm(v[j].y, v[j].x, 0x07050301u);
        const u32 l1 = perm(v[j].w, v[j].z, 0x06040200u), h1 = perm(v[j].w, v[j].z, 0x07050301u);
        e[j][0] = bfi(0x01010101u, l0 >> 7, h0 << 1); e[j][1] = bfi(0x01010101u, l1 >> 7, h1 << 1);
        st64(rec + j * 512 + lane * 8, bfi(0x80808080u, h0, l0), bfi(0x80808080u, h1, l1));
        sum = __builtin_amdgcn_udot4(e[j][0], 0x01010101u, __builtin_amdgcn_udot4(e[j][1], 0x01010101u, sum, false), false);
    }
    const int tot = (int)last(scan(sum));
    const u32 base = (u32)min(max((tot - 369) >> 11, 16), 250);
    const u32 brep = base * 0x01010101u;
    u32 pa = 0, pb = 0, more = 0;
#pragma unroll
    for (int j = 0; j < 4; ++j) {
        const u32 d0 = subb(e[j][0], brep), d1 = subb(e[j][1], brep);
        more |= gather8(geb<3>(d0), geb<3>(d1)) << (8 * j);
        pa |= gather8(d0 & 0x01010101u, d1 & 0x01010101u) << (8 * j);
        pb |= gather8((d0 >> 1) & 0x01010101u, (d1 >> 1) & 0x01010101u) << (8 * j);
    }
    st32(rec + 2048 + lane * 4, pa | more); st32(rec + 2304 + lane * 4, pb | more);
    u32 o[4]; const u32 n2 = offsets4(more, o);
    zero_lds(t2, n2, lane);
    CBAR();
#pragma unroll
    for (int j = 0; j < 4; ++j) {
        const u32x2 sel = ld64(L_LUTC + ((more >> (8 * j)) & 255u) * 8u);
        scatter8(t2 + o[j], perm(e[j][1], e[j][0], sel.x), perm(e[j][1], e[j][0], sel.y));
    }
    CBAR();
    bool wide; const u32 end = enc_l2(rec + 2560, n2, base, t2, wide, lane);
    if (lane == 0) { *ctl(C_HDR + wid) = (n2 << 12) | (base << 24); *ctl(C_SIZE + wid) = ((end - rec + 15) >> 4) | (wide ? 0x8000u : 0u); }
}

__device__ __forceinline__ void enc_output(const u32x4 (&v)[4], int wid, int lane) {
    const u32 rec = L_RB + wid * ENC_W, t1l = rec + 5248, t1h = t1l + TB, t2 = t1h + TB;
    u32 l[4][2], h[4][2], m = 0, sum = 0;
#pragma unroll
    for (int j = 0; j < 4; ++j) {
        l[j][0] = perm(v[j].y, v[j].x, 0x06040200u); h[j][0] = perm(v[j].y, v[j].x, 0x07050301u);
        l[j][1] = perm(v[j].w, v[j].z, 0x06040200u); h[j][1] = perm(v[j].w, v[j].z, 0x07050301u);
        m |= gather8(geb<1>(l[j][0] | h[j][0]), geb<1>(l[j][1] | h[j][1])) << (8 * j);
        sum = __builtin_amdgcn_udot4(bfi(0x01010101u, l[j][0] >> 7, h[j][0] << 1), 0x01010101u, sum, false);
        sum = __builtin_amdgcn_udot4(bfi(0x01010101u, l[j][1] >> 7, h[j][1] << 1), 0x01010101u, sum, false);
    }
    st32(rec + lane * 4, m);
    u32 o[4]; const u32 n1 = offsets4(m, o);
    for (u32 i = lane * 16; i < n1; i += 1024) { const u32x4 z = {0u, 0u, 0u, 0u}; st128(t1l + i, z); st128(t1h + i, z); st128(t2 + i, z); }
    const u32 tot = last(scan(sum));
    const u32 base = uni((u32)min(max((int)((float)tot / (float)max(n1, 1u) - 0.18f), 16), 250));
    const u32 brep = base * 0x01010101u;
    CBAR();
#pragma unroll
    for (int j = 0; j < 4; ++j) {
        const u32x2 sel = ld64(L_LUTC + ((m >> (8 * j)) & 255u) * 8u);
        scatter8(t1l + o[j], perm(l[j][1], l[j][0], sel.x), perm(l[j][1], l[j][0], sel.y));
        scatter8(t1h + o[j], perm(h[j][1], h[j][0], sel.x), perm(h[j][1], h[j][0], sel.y));
    }
    CBAR();
    const u32 nb = (n1 + 7) >> 3, pbo = rec + 256 + n1;
    u32 run = 0, pbs = 0;
#pragma unroll
    for (int p = 0; p < 4; ++p) {
        if ((u32)p * 64u < nb) {
            const u32 idx = p * 64 + lane;
            const u32x2 ll = ld64(t1l + idx * 8u), hh = ld64(t1h + idx * 8u);
            const int left = (int)n1 - (int)(idx * 8u);
            const u32 valid = left >= 8 ? 255u : (left > 0 ? (1u << left) - 1u : 0u);
            const u32 e0 = bfi(0x01010101u, ll.x >> 7, hh.x << 1), e1 = bfi(0x01010101u, ll.y >> 7, hh.y << 1);
            const u32 d0 = subb(e0, brep), d1 = subb(e1, brep);
            const u32 g0 = geb<3>(d0), g1 = geb<3>(d1);
            const u32 more = gather8(g0, g1) & valid;
            if (left > 0) st64(rec + 256 + idx * 8u, bfi(0x80808080u, (d0 | g0) << 7, ll.x), bfi(0x80808080u, (d1 | g1) << 7, ll.y));
            pbs |= ((gather8((d0 >> 1) & 0x01010101u, (d1 >> 1) & 0x01010101u) | more) & valid) << (8 * p);
            const u32 cnt = __popc(more), inc = scan(cnt), o2 = run + inc - cnt;
            run += last(inc);
            const u32x2 sel = ld64(L_LUTC + more * 8u);
            scatter8(t2 + o2, perm(e1, e0, sel.x), perm(e1, e0, sel.y));
        }
    }
    CBAR();
#pragma unroll
    for (int p = 0; p < 4; ++p)
        if ((u32)(p * 64 + lane) < nb) st8(pbo + p * 64 + lane, pbs >> (8 * p));
    CBAR();
    bool wide; const u32 end = enc_l2(pbo + nb, run, base, t2, wide, lane);
    if (lane == 0) { *ctl(C_HDR + wid) = n1 | (run << 12) | (base << 24); *ctl(C_SIZE + wid) = ((end - rec + 15) >> 4) | (wide ? 0x8000u : 0u); }
}

__device__ __forceinline__ u32 emit_block(char* slot, int wid, int lane) {
    const u32 rec = L_RB + wid * ENC_W + lane * 16;
    CBAR();
    const u32x4 r0 = ld128(rec), r1 = ld128(rec + 1024), r2 = ld128(rec + 2048);
    wg_barrier();
    const u32 sz = *ctl(C_SIZE + (lane & 7));
    u32 inc = sz & 0x7fffu;
    inc += __builtin_amdgcn_update_dpp(0u, inc, 0x111, 0xf, 0xf, false);
    inc += __builtin_amdgcn_update_dpp(0u, inc, 0x112, 0xf, 0xf, false);
    inc += __builtin_amdgcn_update_dpp(0u, inc, 0x114, 0xf, 0xf, false);
    inc += 3u;
    const u32 mysz = __builtin_amdgcn_readlane(sz, wid) & 0x7fffu, myend = __builtin_amdgcn_readlane(inc, wid);
    const u32 total = __builtin_amdgcn_readlane(inc, 7);
    char* d = slot + (size_t)(myend - mysz + lane) * 16;
    if ((u32)lane < mysz) gst_sys(d, r0);
    if ((u32)lane + 64u < mysz) gst_sys(d + 1024, r1);
    if ((u32)lane + 128u < mysz) gst_sys(d + 2048, r2);
    for (u32 i = 192; i + lane < mysz; i += 64) gst_sys(d + (size_t)i * 16, ld128(rec + i * 16));
    if (wid == 0) {
        const u32 en = inc | (sz & 0x8000u);
        u32x4 hv;
        hv.x = __builtin_amdgcn_readlane(en, 0) | (__builtin_amdgcn_readlane(en, 1) << 16);
        hv.y = __builtin_amdgcn_readlane(en, 2) | (__builtin_amdgcn_readlane(en, 3) << 16);
        hv.z = __builtin_amdgcn_readlane(en, 4) | (__builtin_amdgcn_readlane(en, 5) << 16);
        hv.w = __builtin_amdgcn_readlane(en, 6) | (__builtin_amdgcn_readlane(en, 7) << 16);
        if (lane < 2) hv = ld128(L_CTL + 4 * C_HDR + lane * 16);
        if (lane < 3) gst_sys(slot + lane * 16, hv);
    }
    vm_drain();
    wg_barrier();
    return total;
}

__device__ __forceinline__ void flag_store(const Params& P, int dest, int pos, u32 val) {
    u32* f = (u32*)(P.ws[dest] + OFF_FL) + (P.rank * 1024 + pos);
    __hip_atomic_store((GAS u32*)f, val, __ATOMIC_RELAXED, __HIP_MEMORY_SCOPE_SYSTEM);
}
__device__ __forceinline__ void flag_all(const Params& P, int pos, u32 val) {
#pragma unroll
    for (int p = 0; p < NR; ++p) if (p != P.rank) flag_store(P, p, pos, val);
}
__device__ __forceinline__ void ready_add(char* ws, int idx) {
    __hip_atomic_fetch_add((GAS u32*)(ws + OFF_RDY + idx * 128), 1u, __ATOMIC_RELAXED, __HIP_MEMORY_SCOPE_AGENT);
}
__device__ __forceinline__ char* ws_of(const Params& P, int r) {
    char* p = P.ws[0];
#pragma unroll
    for (int i = 1; i < NR; ++i) p = r == i ? P.ws[i] : p;
    return p;
}

#define STAMP(i) do {} while (0)
#define STAMPW(i) do {} while (0)

__device__ __forceinline__ bool pos_is_x(int pos) { return pos < 16 || (pos >= 48 && pos < 160); }
__device__ __forceinline__ int pos_xb(int pos) { return pos < 16 ? pos : pos - 32; }
__device__ __forceinline__ int pos_wb(int pos) { return pos < 48 ? pos - 16 : pos - 128; }
__device__ __forceinline__ void task_of(int rank, int task, int& pos, int& dest) {
    dest = -1;
    if (task < 16) pos = task;
    else if (task < 272) { pos = 16 + ((task - 16) >> 3); dest = (rank + 1 + (task & 7)) & 7; }
    else if (task < 384) pos = 48 + task - 272;
    else { pos = 160 + ((task - 384) >> 3); dest = (rank + 1 + (task & 7)) & 7; }
}
__device__ __forceinline__ void pack_load(const Params& P, int task, int wid, int lane, u32x4 (&t)[4]) {
    int pos, dest; task_of(P.rank, task, pos, dest);
    const char* s; size_t s1, s2;
    if (dest < 0) {
        const int xb = pos_xb(pos);
        s = P.x + (size_t)((xb >> 3) * 256 + wid * 32 + (lane & 15)) * 1024 + (xb & 7) * 128 + (lane >> 4) * 16;
        s1 = 64; s2 = 16384;
    } else {
        const int wb = pos_wb(pos);
        s = P.w + (size_t)((wb & 3) * 128 + (wid >> 2) * 64 + (lane >> 2)) * (NFULL * 2) + (dest * 2048 + (wb >> 2) * 128 + (wid & 3) * 32) * 2 + (lane & 3) * 16;
        s1 = 16 * NFULL * 2; s2 = 2 * s1;
    }
    t[0] = *(const GAS u32x4*)s; t[1] = *(const GAS u32x4*)(s + s1); t[2] = *(const GAS u32x4*)(s + s2); t[3] = *(const GAS u32x4*)(s + s1 + s2);
}
__device__ __forceinline__ void pack_task(const Params& P, char* ws, int task, u32 call, int wid, int lane, u32x4 (&v)[4]) {
    int pos, dest; task_of(P.rank, task, pos, dest);
    size_t lo, so; int rdy;
    if (dest < 0) {
        const int xb = pos_xb(pos);
        rdy = xb >> 3; lo = OFF_XA + (size_t)(rdy * 64 + P.rank * 8 + (xb & 7)) * 32768; so = OFF_CX + (size_t)xb * SLOT;
    } else {
        const int wb = pos_wb(pos);
        rdy = 16 + (wb >> 2); lo = OFF_WB + (size_t)((wb >> 2) * 64 + P.rank * 8 + (wb & 3) * 2) * 16384; so = OFF_CW + (size_t)(dest * 64 + wb) * SLOT;
        const u32 tr = L_RB + wid * ENC_W + 5248;
#pragma unroll
        for (int i = 0; i < 4; ++i) st128(tr + i * 1024 + lane * 16, v[i]);
        CBAR();
        const u32 ra = tr + (lane >> 4) * 512 + (((lane >> 2) & 3) * 8 + (lane & 3)) * 2;
#pragma unroll
        for (int j = 0; j < 4; ++j) {
            u32 q[8];
#pragma unroll
            for (int e = 0; e < 8; ++e) q[e] = *(LAS unsigned short*)lp(ra + (j & 1) * 2048 + e * 64 + (j >> 1) * 8);
            v[j] = (u32x4){q[0] | (q[1] << 16), q[2] | (q[3] << 16), q[4] | (q[5] << 16), q[6] | (q[7] << 16)};
        }
        CBAR();
    }
    const bool local = dest < 0 || dest == P.rank;
    if (local) {
        char* d = ws + lo + wid * 4096 + lane * 16;
#pragma unroll
        for (int j = 0; j < 4; ++j) gst_agent(d + j * 1024, v[j]);
    }
    u32 len = 0;
    if (dest != P.rank) { enc_input(v, wid, lane); len = emit_block(ws + so, wid, lane); }
    else { vm_drain(); wg_barrier(); }
    if (threadIdx.x == 0) {
        if (dest < 0) flag_all(P, pos, (call << 12) | len);
        else if (!local) __hip_atomic_store((GAS u32*)(ws_of(P, dest) + OFF_FL) + (P.rank * 1024 + pos), (call << 12) | len, __ATOMIC_RELAXED, __HIP_MEMORY_SCOPE_SYSTEM);
        if (local) ready_add(ws, rdy);
    }
    JOIN();
}

__device__ __forceinline__ void tile_of(int set, int t16, int& pm, int& pn) {
    pm = 2 * (set & 7) + (t16 >> 3); pn = (set & 8) + (t16 & 7);
}

__device__ __forceinline__ void decoder(const Params& P, char* ws, int src, int d, u32 call, int wid, int lane) {
    const u32 seq = call & 0xfffffu;
    const u64 tstart = now();
    GAS u32* tmo = (GAS u32*)(ws + OFF_TMO);
    if (wid == 1) {
        const GAS u32* fl = (const GAS u32*)(ws + OFF_FL) + src * 1024;
        bool dead = false;
        for (u32 i = 0;; ++i) {
            u32 pos = d;
            if (i) {
                while (ctl_ld(C_PROG) < i) { if (now() - tstart > 2 * DEADLINE) break; __builtin_amdgcn_s_sleep(2); }
                u32 t = 0;
                if (lane == 0) t = __hip_atomic_fetch_add((GAS u32*)(ws + OFF_CLM + src * 128), 1u, __ATOMIC_RELAXED, __HIP_MEMORY_SCOPE_AGENT);
                pos = uni(t) - (call - 1u) * (u32)NPOS + (u32)DPP_;
            }
            if (pos >= (u32)NPOS) { *ctl(C_QUEUE + (i & 7u)) = ((i + 1u) << 22) | (1023u << 12); break; }
            u32 f = 0;
            while (!dead) {
                f = uni(__hip_atomic_load(fl + pos, __ATOMIC_RELAXED, __HIP_MEMORY_SCOPE_SYSTEM));
                if ((f >> 12) == seq) break;
                if (uni(__hip_atomic_load(tmo, __ATOMIC_RELAXED, __HIP_MEMORY_SCOPE_AGENT)) == call || now() - tstart > DEADLINE) {
                    __hip_atomic_store(tmo, call, __ATOMIC_RELAXED, __HIP_MEMORY_SCOPE_AGENT);
                    *ctl(C_DEAD) = 1; dead = true; f = 0;
                }
                __builtin_amdgcn_s_sleep(1);
            }
            CBAR();
            *ctl(C_QUEUE + (i & 7u)) = ((i + 1u) << 22) | (pos << 12) | (f & 0xfffu);
        }
    } else if (wid == 0) {
        const char* sws = ws_of(P, src);
        u32 t0 = 0, t1 = 0, t2 = 0, cur = 0;
        u32 i = 0;
        for (;; ++i) {
            const u32 need = 8u * (i >> 1);
            u32 f = ctl_ld(C_QUEUE + (i & 7u));
            if ((f >> 22) != i + 1u || ctl_ld(C_DONE + (i & 1u)) < need) {
                asm volatile("s_waitcnt vmcnt(2)" ::: "memory"); if (t0 > cur) *ctl(C_LANDED) = t0;
                asm volatile("s_waitcnt vmcnt(1)" ::: "memory"); if (t1 > cur) *ctl(C_LANDED) = t1;
                vm_drain();
                cur = i << 6; *ctl(C_LANDED) = cur;
                while (((f = ctl_ld(C_QUEUE + (i & 7u))) >> 22) != i + 1u || ctl_ld(C_DONE + (i & 1u)) < need) {
                    if (now() - tstart > 2 * DEADLINE) { f = 1023u << 12; break; }
                    __builtin_amdgcn_s_sleep(1);
                }
            }
            CBAR();
            const int pos = (f >> 12) & 1023u;
            if (pos >= NPOS) break;
            *ctl(C_PROG) = i + 1u;
            size_t so;
            if (pos < NIN) so = pos_is_x(pos) ? OFF_CX + (size_t)pos_xb(pos) * SLOT : OFF_CW + (size_t)(P.rank * 64 + pos_wb(pos)) * SLOT;
            else so = OFF_CO + (size_t)(pos - NIN) * SLOT;
            const u32 len = min(f & 0xfffu, (u32)(LSLOT / 16));
            const char* s = sws + so + lane * 16;
            const u32 lb = (u32)(size_t)lp(L_RB + (i & 1u) * LSLOT);
            for (u32 k = 0; k * 64u < len; ++k) {
                if (k * 64u + lane < len) {
                    u32 keep;
                    asm volatile("s_mov_b32 %0, m0\n\ts_mov_b32 m0, %2\n\ts_nop 0\n\tglobal_load_lds_dwordx4 %1, off sc0 sc1\n\ts_mov_b32 m0, %0"
                                 : "=&s"(keep) : "v"(s + (size_t)k * 1024), "s"(lb + k * 1024u) : "memory");
                }
                asm volatile("s_waitcnt vmcnt(3)" ::: "memory");
                if (t0 > cur) { cur = t0; *ctl(C_LANDED) = cur; }
                t0 = t1; t1 = t2; t2 = (i << 6) + k + 1;
            }
        }
        asm volatile("s_waitcnt vmcnt(2)" ::: "memory"); if (t0 > cur) *ctl(C_LANDED) = t0;
        asm volatile("s_waitcnt vmcnt(1)" ::: "memory"); if (t1 > cur) *ctl(C_LANDED) = t1;
        vm_drain();
        *ctl(C_NBLK) = i + 1u;
        *ctl(C_LANDED) = 0xffffffffu;
        STAMPW(50);
    } else {
        const u32 tmp = L_DT + (wid - 2) * 3 * TB;
        for (;;) {
            const u32 r = ctl_inc(C_CLAIM);
            const u32 i = r >> 3, sub = r & 7u;
            const u32 slot = L_RB + (i & 1u) * LSLOT;
            bool late = false;
            while (ctl_ld(C_LANDED) < (i << 6) + 1u) { if (now() - tstart > 2 * DEADLINE) { late = true; break; } __builtin_amdgcn_s_sleep(1); }
            CBAR();
            if (late || ctl_ld(C_NBLK) - 1u <= i) break;
            const int pos = min((ctl_ld(C_QUEUE + (i & 7u)) >> 12) & 1023u, (u32)(NPOS - 1));
            const u32 hdr = uni(ld32(slot + sub * 4));
            const u32 en = uni(*(LAS unsigned short*)lp(slot + 32 + sub * 2));
            const u32 st = sub ? (uni(*(LAS unsigned short*)lp(slot + 30 + sub * 2)) & 0x7fffu) : 3u;
            const u32 endp = min(((en & 0x7fffu) + 63u) >> 6, 63u);
            while (ctl_ld(C_LANDED) < (i << 6) + endp) { if (now() - tstart > 2 * DEADLINE) break; __builtin_amdgcn_s_sleep(1); }
            CBAR();
            const bool dead = ctl_ld(C_DEAD) != 0;
            const u32 rec = slot + min(st, (u32)(LSLOT / 16)) * 16u;
            if (pos < NIN) {
                size_t o; int rdy;
                if (pos_is_x(pos)) { const int xb = pos_xb(pos); rdy = xb >> 3; o = OFF_XA + (size_t)(rdy * 64 + src * 8 + (xb & 7)) * 32768; }
                else { const int wb = pos_wb(pos); rdy = 16 + (wb >> 2); o = OFF_WB + (size_t)((wb >> 2) * 64 + src * 8 + (wb & 3) * 2) * 16384; }
                if (!dead) dec_input(rec, hdr, (en >> 15) != 0, tmp, ws + o + sub * 4096 + lane * 16, lane);
                asm volatile("s_waitcnt lgkmcnt(0)" ::: "memory");
                ctl_inc(C_DONE + (i & 1u));
                vm_drain();
                if ((ctl_inc(C_STORED + (i & 15u)) & 7u) == 7u) { if (lane == 0) ready_add(ws, rdy); JOIN(); }
            } else {
                const int ob = pos - NIN;
                int pm, pn; tile_of(ob >> 5, (ob & 31) >> 1, pm, pn);
                char* dst = P.out + ((size_t)(pm * 256 + (sub >> 1) * 64 + (ob & 1) * 32 + (lane & 15)) * NFULL + src * 2048 + pn * 128 + (sub & 1) * 64 + (lane >> 4) * 8) * 2;
                if (!dead) dec_output(rec, hdr, (en >> 15) != 0, tmp, dst, lane);
                else gst_agent(dst, (u32x4){0x7fc07fc0u, 0x7fc07fc0u, 0x7fc07fc0u, 0x7fc07fc0u});
                asm volatile("s_waitcnt lgkmcnt(0)" ::: "memory");
                ctl_inc(C_DONE + (i & 1u));
            }
        }
    }
}

__device__ __forceinline__ void gemm_tile(const Params& P, char* ws, int set, int t16, u32 call, int wid, int lane) {
    int pm, pn; tile_of(set, t16, pm, pn);
    GAS u32* tmo = (GAS u32*)(ws + OFF_TMO);
    if (threadIdx.x == 0) {
        const GAS u32* rx = (const GAS u32*)(ws + OFF_RDY + pm * 128);
        const GAS u32* rw = (const GAS u32*)(ws + OFF_RDY + (16 + pn) * 128);
        const u64 t0 = now();
        while ((int)(__hip_atomic_load(rx, __ATOMIC_RELAXED, __HIP_MEMORY_SCOPE_AGENT) - call * 64u) < 0 ||
               (int)(__hip_atomic_load(rw, __ATOMIC_RELAXED, __HIP_MEMORY_SCOPE_AGENT) - call * 32u) < 0) {
            if (__hip_atomic_load(tmo, __ATOMIC_RELAXED, __HIP_MEMORY_SCOPE_AGENT) == call || now() - t0 > DEADLINE) {
                __hip_atomic_store(tmo, call, __ATOMIC_RELAXED, __HIP_MEMORY_SCOPE_AGENT);
                break;
            }
            __builtin_amdgcn_s_sleep(16);
        }
        *ctl(C_DEAD) = __hip_atomic_load(tmo, __ATOMIC_RELAXED, __HIP_MEMORY_SCOPE_AGENT) == call;
        __builtin_amdgcn_fence(__ATOMIC_ACQUIRE, "agent");
        vm_drain();
    }
    wg_barrier();
    STAMP(8 + (set >> 3) * 4);
    const char* ga = ws + OFF_XA + (size_t)pm * (2u << 20);
    const char* gb = ws + OFF_WB + (size_t)pn * (1u << 20);
    const u32 voa = wid * 4096 + lane * 16, vob = wid * 2048 + lane * 16;
    const u32 lbase = (u32)(size_t)lp(L_RB);
    const u32 lda = lbase + wid * 4096, ldb = lbase + 32768 + wid * 2048;
#define GLDS(vo, gp, lo, off) do { u32 keep_; asm volatile( \
        "s_mov_b32 %0, m0\n\ts_mov_b32 m0, %3\n\ts_nop 3\n\tglobal_load_lds_dwordx4 %1, %2 offset:" #off "\n\ts_mov_b32 m0, %0" \
        : "=&s"(keep_) : "v"(vo), "s"(gp), "s"(lo) : "memory"); } while (0)
#define STAGE_LD(so, kt) do { const char* a_ = ga + (size_t)(kt) * 32768; const char* b_ = gb + (size_t)(kt) * 16384; \
        GLDS(voa, a_, lda + (so), 0); GLDS(voa, a_, lda + (so), 1024); GLDS(voa, a_, lda + (so), 2048); GLDS(voa, a_, lda + (so), 3072); \
        GLDS(vob, b_, ldb + (so), 0); GLDS(vob, b_, ldb + (so), 1024); } while (0)
    const u32 ra = L_RB + (wid >> 1) * 8192 + lane * 16, rb = L_RB + 32768 + (wid & 1) * 8192 + lane * 16;
#define FRAGS(A, B, so, kb) do { _Pragma("unroll") for (int q_ = 0; q_ < 4; ++q_) { \
        B[q_] = *(const LAS bf16x8*)lp(rb + (so) + q_ * 2048 + (kb) * 1024); A[q_] = *(const LAS bf16x8*)lp(ra + (so) + q_ * 2048 + (kb) * 1024); } } while (0)
#define MROW(A, B, m_) do { _Pragma("unroll") for (int n_ = 0; n_ < 4; ++n_) \
        acc[m_][n_] = __builtin_amdgcn_mfma_f32_16x16x32_bf16(B[n_], A[m_], acc[m_][n_], 0, 0, 0); } while (0)
#define SB() __builtin_amdgcn_sched_barrier(0)
#define MM2(A, B, m_, n_) do { acc[m_][n_] = __builtin_amdgcn_mfma_f32_16x16x32_bf16(B[n_], A[m_], acc[m_][n_], 0, 0, 0); \
        acc[m_][n_ + 1] = __builtin_amdgcn_mfma_f32_16x16x32_bf16(B[n_ + 1], A[m_], acc[m_][n_ + 1], 0, 0, 0); } while (0)
#define RDB(B, q_, kb) B[q_] = *(const LAS bf16x8*)lp(rb + so + (q_) * 2048 + (kb) * 1024)
#define RDA(A, q_, kb) A[q_] = *(const LAS bf16x8*)lp(ra + so + (q_) * 2048 + (kb) * 1024)
#define HALF(A, B, NA, NB, kb, G0, G1, G2) do { SB(); \
        MM2(A, B, 0, 0); SB(); RDA(NA, 0, kb); RDB(NB, 0, kb); SB(); MM2(A, B, 0, 2); SB(); RDB(NB, 1, kb); RDB(NB, 2, kb); SB(); \
        MM2(A, B, 1, 0); SB(); RDB(NB, 3, kb); RDA(NA, 1, kb); SB(); MM2(A, B, 1, 2); SB(); RDA(NA, 2, kb); RDA(NA, 3, kb); SB(); \
        MM2(A, B, 2, 0); SB(); G0; SB(); MM2(A, B, 2, 2); SB(); G1; SB(); \
        MM2(A, B, 3, 0); SB(); G2; SB(); MM2(A, B, 3, 2); SB(); } while (0)
#define MMA(A, B) do { MROW(A, B, 0); MROW(A, B, 1); MROW(A, B, 2); MROW(A, B, 3); } while (0)
    f32x4 acc[4][4];
#pragma unroll
    for (int m = 0; m < 4; ++m)
#pragma unroll
        for (int n = 0; n < 4; ++n) acc[m][n] = (f32x4){0.f, 0.f, 0.f, 0.f};
    bf16x8 a0[4], b0[4], a1[4], b1[4];
    STAGE_LD(0, 0); STAGE_LD(STAGE, 1);
    { const char* a_ = ga + 2 * 32768; GLDS(voa, a_, lda + 2 * STAGE, 0); GLDS(voa, a_, lda + 2 * STAGE, 1024); GLDS(voa, a_, lda + 2 * STAGE, 2048); }
    asm volatile("s_waitcnt vmcnt(9)\n\ts_barrier" ::: "memory");
    FRAGS(a0, b0, 0u, 0);
    u32 so = 0, sp = 2 * STAGE;
    for (int kt = 0; kt < 61; ++kt) {
        const char* a2 = ga + (size_t)(kt + 2) * 32768; const char* b2 = gb + (size_t)(kt + 2) * 16384; const char* a3 = a2 + 32768;
        HALF(a0, b0, a1, b1, 1, GLDS(voa, a2, lda + sp, 3072), GLDS(vob, b2, ldb + sp, 0), GLDS(vob, b2, ldb + sp, 1024));
        asm volatile("s_waitcnt vmcnt(6) lgkmcnt(0)\n\ts_barrier" ::: "memory");
        sp = so; so = so == 2 * STAGE ? 0 : so + STAGE;
        HALF(a1, b1, a0, b0, 0, GLDS(voa, a3, lda + sp, 0), GLDS(voa, a3, lda + sp, 1024), GLDS(voa, a3, lda + sp, 2048));
    }
    {
        const char* a2 = ga + (size_t)63 * 32768; const char* b2 = gb + (size_t)63 * 16384;
        GLDS(voa, a2, lda + sp, 3072); GLDS(vob, b2, ldb + sp, 0); GLDS(vob, b2, ldb + sp, 1024);
        FRAGS(a1, b1, so, 1); MMA(a0, b0);
        asm volatile("s_waitcnt vmcnt(6) lgkmcnt(0)\n\ts_barrier" ::: "memory");
        so = so == 2 * STAGE ? 0 : so + STAGE;
        FRAGS(a0, b0, so, 0); MMA(a1, b1);
        FRAGS(a1, b1, so, 1); MMA(a0, b0);
        asm volatile("s_waitcnt vmcnt(0) lgkmcnt(0)\n\ts_barrier" ::: "memory");
        so = so == 2 * STAGE ? 0 : so + STAGE;
        FRAGS(a0, b0, so, 0); MMA(a1, b1);
        FRAGS(a1, b1, so, 1); MMA(a0, b0); MMA(a1, b1);
    }
#undef GLDS
#undef STAGE_LD
#undef FRAGS
#undef MROW
#undef MMA
#undef MM2
#undef RDA
#undef RDB
#undef HALF
#undef SB
    wg_barrier();
    STAMP(9 + (set >> 3) * 4);
    const bool dead = ctl_ld(C_DEAD) != 0;
    char* dst = P.out + ((size_t)(pm * 256 + (wid >> 1) * 64 + (lane & 15)) * NFULL + P.rank * 2048 + pn * 128 + (wid & 1) * 64 + (lane >> 4) * 8) * 2;
#pragma unroll
    for (int b = 0; b < 2; ++b) {
        u32x4 v[4];
#pragma unroll
        for (int j = 0; j < 4; ++j) {
            const f32x4 p = acc[2 * b + (j >> 1)][2 * (j & 1)], q = acc[2 * b + (j >> 1)][2 * (j & 1) + 1];
            float f[8] = {p[0], p[1], p[2], p[3], q[0], q[1], q[2], q[3]};
            u32 r[4];
#pragma unroll
            for (int e = 0; e < 4; ++e) {
                const float lo = f[2 * e] < 0.f ? 0.f : f[2 * e], hi = f[2 * e + 1] < 0.f ? 0.f : f[2 * e + 1];
                asm("v_cvt_pk_bf16_f32 %0, %1, %2" : "=v"(r[e]) : "v"(lo), "v"(hi));
                r[e] = dead ? 0x7fc07fc0u : (r[e] & 0x7fff7fffu);
            }
            v[j] = (u32x4){r[0], r[1], r[2], r[3]};
            gst_agent(dst + (size_t)(2 * b + (j >> 1)) * (16 * NFULL * 2) + (j & 1) * 64, v[j]);
        }
        enc_output(v, wid, lane);
        const int ob = set * 32 + t16 * 2 + b;
        const u32 len = emit_block(ws + OFF_CO + (size_t)ob * SLOT, wid, lane);
        if (threadIdx.x == 0) flag_all(P, NIN + ob, (call << 12) | len);
        JOIN();
    }
    STAMP(10 + (set >> 3) * 4);
}

__global__ void __launch_bounds__(512, 2) fused_gemm_ar(const Params P) {
    const int tid = threadIdx.x, lane = tid & 63, wid = uni(tid >> 6), blk = blockIdx.x;
    char* ws = ws_of(P, P.rank);
    STAMP(0);
    const int xcd = blk & 7, idx = blk >> 3;
    const bool is_dec = blk < NDEC;
    u32x4 nx[4];
    if (!is_dec) pack_load(P, blk - NDEC, wid, lane, nx);
    if (tid < 256) {
        u32 e[2] = {0x0c0c0c0cu, 0x0c0c0c0cu}, c[2] = {0x0c0c0c0cu, 0x0c0c0c0cu};
        u64 ee = 0x0c0c0c0c0c0c0c0cull, cc = ee; int k = 0;
#pragma unroll
        for (int i = 0; i < 8; ++i)
            if ((tid >> i) & 1) { ee ^= (u64)(0x0c ^ k) << (8 * i); cc ^= (u64)(0x0c ^ i) << (8 * k); ++k; }
        e[0] = (u32)ee; e[1] = (u32)(ee >> 32); c[0] = (u32)cc; c[1] = (u32)(cc >> 32);
        st64(L_LUTE + tid * 8, e[0], e[1]); st64(L_LUTC + tid * 8, c[0], c[1]);
    } else if (tid < 256 + 128) {
        u32 v = 0;
        if (tid == 256 + C_CALL) v = __hip_atomic_fetch_add((GAS u32*)(ws + OFF_CTR + blk * 64), 1u, __ATOMIC_RELAXED, __HIP_MEMORY_SCOPE_AGENT) + 1u;
        *ctl(tid - 256) = v;
    }
    wg_barrier();
    const u32 call = ctl_ld(C_CALL);
    if (!is_dec) {
        for (int task = blk - NDEC; task < NTASK; task += NWG - NDEC) {
            u32x4 v[4] = {nx[0], nx[1], nx[2], nx[3]};
            if (task + NWG - NDEC < NTASK) pack_load(P, task + NWG - NDEC, wid, lane, nx);
            pack_task(P, ws, task, call, wid, lane, v);
        }
    }
    STAMP(1);
    if (is_dec) {
        const int j = idx >= 7 ? idx - 7 : idx;
        decoder(P, ws, (P.rank + 1 + j) & 7, (idx >= 7 ? 8 : 0) + xcd, call, wid, lane);
    } else if (idx < 30) {
        for (int rr = 0; rr < 2; ++rr) gemm_tile(P, ws, xcd + 8 * rr, idx - 14, call, wid, lane);
    }
}

}

extern "C" void kernel_launch(int rank, int nranks, ncclComm_t comm, hipStream_t stream,
                              void* const* d_in, const int* in_sizes, int n_in,
                              void* d_out, int out_size, const int* in_dtypes, int out_dtype,
                              void* const* d_out_all, void* const* d_ws_all, size_t ws_bytes) {
    if (nranks != NR || (unsigned)rank >= (unsigned)NR || n_in != 2 || ws_bytes < WS_NEED ||
        in_sizes[0] != 4096 * 512 || in_sizes[1] != 512 * NFULL || out_size != 4096 * NFULL) {
        fprintf(stderr, "kernel_launch: unexpected problem shape\n");
        return;
    }
    Params P;
    P.x = (const char*)d_in[0]; P.w = (const char*)d_in[1]; P.out = (char*)d_out;
    for (int p = 0; p < NR; ++p) P.ws[p] = (char*)d_ws_all[p];
    P.rank = rank; P.pad = 0;
    hipLaunchKernelGGL(fused_gemm_ar, dim3(NWG), dim3(512), 0, stream, P);
    hipError_t e = hipGetLastError();
    if (e != hipSuccess) fprintf(stderr, "launch failed: %s\n", hipGetErrorString(e));
}
```

```cpp
#include <hip/hip_runtime.h>
#include <rccl/rccl.h>
#include <cstdio>

namespace {

typedef unsigned u32;
typedef unsigned long long u64;
typedef u32 u32x4 __attribute__((ext_vector_type(4)));
typedef u32 u32x2 __attribute__((ext_vector_type(2)));
typedef short bf16x8 __attribute__((ext_vector_type(8)));
typedef float f32x4 __attribute__((ext_vector_type(4)));
#define LAS __attribute__((address_space(3)))
#define GAS __attribute__((address_space(1)))

constexpr int NR = 8;
constexpr int NFULL = 16384;
constexpr int NWG = 256;
constexpr int DPP_ = 16;
constexpr int NIN = 192, NOUT = 512;
constexpr int NPOS = NIN + NOUT;
constexpr int NDEC = 7 * DPP_;
constexpr int NTASK = 640;
constexpr u64 DEADLINE = 50000000ull;

constexpr size_t OFF_CTR = 0;
constexpr size_t OFF_TMO = 16384;
constexpr size_t OFF_RDY = 16384 + 256;
constexpr size_t OFF_CLM = 24576;
constexpr size_t OFF_FL = 32768;
constexpr size_t OFF_DBG = 65536;
constexpr size_t OFF_XA = 1u << 20;
constexpr size_t OFF_WB = OFF_XA + (32u << 20);
constexpr size_t SLOT = 43008;
constexpr size_t OFF_CX = OFF_WB + (16u << 20);
constexpr size_t OFF_CW = OFF_CX + 128 * SLOT;
constexpr size_t OFF_CO = OFF_CW + 512 * SLOT;
constexpr size_t WS_NEED = OFF_CO + 512 * SLOT;

constexpr int L_LUTE = 0;
constexpr int L_LUTC = 2048;
constexpr int L_CTL = 4096;
constexpr int L_RB = 4608;
constexpr int STAGE = 49152;
constexpr int LDS_BYTES = L_RB + 3 * STAGE;
constexpr int TB = 2112;
constexpr int ENC_W = 5248 + 3 * TB;
constexpr int LSLOT = 41984;
constexpr int L_DT = L_RB + 2 * LSLOT;
static_assert(L_DT + 6 * 3 * TB <= LDS_BYTES, "lds");
static_assert(8 * ENC_W <= 3 * STAGE, "lds");
constexpr int C_HDR = 0, C_SIZE = 8, C_CALL = 16, C_DEAD = 17, C_LANDED = 18, C_CLAIM = 19, C_DONE = 20  ,
              C_NBLK = 22, C_PROG = 23, C_STORED = 24  , C_QUEUE = 40  ;

struct Params {
    const char* x; const char* w; char* out;
    char* ws[NR];
    int rank; int pad;
};

__shared__ __attribute__((aligned(16))) unsigned char smem[LDS_BYTES];

#define CBAR() asm volatile("" ::: "memory")
__device__ __forceinline__ LAS unsigned char* lp(u32 o) { return (LAS unsigned char*)smem + o; }
__device__ __forceinline__ u32 ld8(u32 o) { return *lp(o); }
__device__ __forceinline__ u32 ld32(u32 o) { return *(LAS u32*)lp(o); }
__device__ __forceinline__ u32x2 ld64(u32 o) { return *(LAS u32x2*)lp(o); }
__device__ __forceinline__ u32x4 ld128(u32 o) { return *(LAS u32x4*)lp(o); }
__device__ __forceinline__ void st8(u32 o, u32 v) { *lp(o) = (unsigned char)v; }
__device__ __forceinline__ void st32(u32 o, u32 v) { *(LAS u32*)lp(o) = v; }
__device__ __forceinline__ void st64(u32 o, u32 a, u32 b) { *(LAS u32x2*)lp(o) = (u32x2){a, b}; }
__device__ __forceinline__ void st128(u32 o, u32x4 v) { *(LAS u32x4*)lp(o) = v; }
__device__ __forceinline__ void or32(u32 o, u32 v) { __hip_atomic_fetch_or((LAS u32*)lp(o), v, __ATOMIC_RELAXED, __HIP_MEMORY_SCOPE_WORKGROUP); }
__device__ __forceinline__ volatile LAS u32* ctl(int i) { return (volatile LAS u32*)lp(L_CTL + 4 * i); }
__device__ __forceinline__ u32 uni(u32 v) { return __builtin_amdgcn_readfirstlane(v); }
__device__ __forceinline__ u32 ctl_ld(int i) { return uni(*ctl(i)); }
__device__ __forceinline__ u32 ctl_inc(int i) {
    u32 r; u64 sv;
    asm volatile("s_mov_b64 %1, exec\n\ts_mov_b64 exec, 1\n\tds_add_rtn_u32 %0, %2, %3\n\ts_waitcnt lgkmcnt(0)\n\ts_mov_b64 exec, %1"
                 : "=&v"(r), "=&s"(sv) : "v"((u32)(size_t)lp(L_CTL + 4 * i)), "v"(1u) : "memory");
    return uni(r);
}
#define JOIN() __builtin_amdgcn_wave_barrier()
__device__ __forceinline__ u32 perm(u32 hi, u32 lo, u32 sel) { return __builtin_amdgcn_perm(hi, lo, sel); }
__device__ __forceinline__ u32 bfi(u32 m, u32 a, u32 b) { return (m & a) | (~m & b); }
__device__ __forceinline__ u64 now() { return __builtin_amdgcn_s_memrealtime(); }
__device__ __forceinline__ void wg_barrier() { asm volatile("s_waitcnt lgkmcnt(0)\n\ts_barrier" ::: "memory"); }
__device__ __forceinline__ void vm_drain() { asm volatile("s_waitcnt vmcnt(0)" ::: "memory"); }

__device__ __forceinline__ void gst_agent(void* p, u32x4 v) { asm volatile("global_store_dwordx4 %0, %1, off sc1\n\ts_nop 1" ::"v"(p), "v"(v) : "memory"); }
__device__ __forceinline__ void gst_sys(void* p, u32x4 v) { asm volatile("global_store_dwordx4 %0, %1, off sc0 sc1\n\ts_nop 1" ::"v"(p), "v"(v) : "memory"); }

__device__ __forceinline__ u32 scan(u32 x) {
    x += __builtin_amdgcn_update_dpp(0u, x, 0x111, 0xf, 0xf, false);
    x += __builtin_amdgcn_update_dpp(0u, x, 0x112, 0xf, 0xf, false);
    x += __builtin_amdgcn_update_dpp(0u, x, 0x114, 0xf, 0xf, false);
    x += __builtin_amdgcn_update_dpp(0u, x, 0x118, 0xf, 0xf, false);
    x += __builtin_amdgcn_update_dpp(0u, x, 0x142, 0xa, 0xf, false);
    x += __builtin_amdgcn_update_dpp(0u, x, 0x143, 0xc, 0xf, false);
    return x;
}
__device__ __forceinline__ u32 last(u32 x) { return __builtin_amdgcn_readlane(x, 63); }
__device__ __forceinline__ u32 offsets4(u32 m, u32 (&o)[4]) {
    const u32 c01 = __popc(m & 0xffu) | (__popc(m & 0xff00u) << 16), c23 = __popc(m & 0xff0000u) | (__popc(m >> 24) << 16);
    const u32 s01 = scan(c01), s23 = scan(c23);
    const u32 t01 = last(s01), t23 = last(s23);
    const u32 e01 = s01 - c01, e23 = s23 - c23;
    const u32 b1 = t01 & 0xffffu, b2 = b1 + (t01 >> 16), b3 = b2 + (t23 & 0xffffu);
    o[0] = e01 & 0xffffu; o[1] = (e01 >> 16) + b1; o[2] = (e23 & 0xffffu) + b2; o[3] = (e23 >> 16) + b3;
    return b3 + (t23 >> 16);
}
__device__ __forceinline__ u32 gather8(u32 y0, u32 y1) {
    return __builtin_amdgcn_udot4(y1, 0x80402010u, __builtin_amdgcn_udot4(y0, 0x08040201u, 0u, false), false);
}
__device__ __forceinline__ u32 spread1(u32 n4) { return __umul24(n4, 0x204081u) & 0x01010101u; }
__device__ __forceinline__ u32 spread2(u32 n4) { return __umul24(n4, 0x408102u) & 0x02020202u; }
__device__ __forceinline__ u32 subb(u32 a, u32 brep) {
    return ((a | 0x80808080u) - (brep & 0x7f7f7f7fu)) ^ ((a ^ ~brep) & 0x80808080u);
}
template <int K> __device__ __forceinline__ u32 geb(u32 d) {
    return ((((d & 0x7f7f7f7fu) + (u32)(0x80 - K) * 0x01010101u) | d) >> 7) & 0x01010101u;
}
__device__ __forceinline__ void fetch8(u32 a, u32& s0, u32& s1) {
    const u32 q = a & ~3u;
    const u32 d0 = ld32(q), d1 = ld32(q + 4), d2 = ld32(q + 8);
    s0 = __builtin_amdgcn_alignbyte(d1, d0, a); s1 = __builtin_amdgcn_alignbyte(d2, d1, a);
}
__device__ __forceinline__ void scatter8(u32 a, u32 c0, u32 c1) {
    const u32 q = a & ~3u, s = (a & 3u) * 8u;
    const u64 lo = (((u64)c1 << 32) | c0) << s;
    const u32 w0 = (u32)lo, w1 = (u32)(lo >> 32), w2 = (c1 >> 1) >> (31u - s);
    if (w0) or32(q, w0);
    if (w1) or32(q + 4, w1);
    if (w2) or32(q + 8, w2);
}
__device__ __forceinline__ void zero_lds(u32 a, u32 bytes, int lane) {
    for (u32 i = lane * 16; i < bytes; i += 1024) st128(a + i, (u32x4){0u, 0u, 0u, 0u});
}

__device__ __forceinline__ void dec_l2(u32 l2a, u32 n2, u32 base, bool wide, u32 t2, int lane) {
    const u32 pb = (n2 + 7) >> 3, l2b = l2a + pb, l3 = l2b + pb;
    const u32 tab = (base - 2) | ((base - 1) << 8) | ((base + 3) << 16);
    const u32 top = (base + 5) * 0x01010101u;
    u32 run = 0;
    for (u32 p = 0; p < pb; p += 64) {
        const u32 idx = p + lane;
        const u32 a8 = ld8(l2a + idx), b8 = ld8(l2b + idx), more = a8 & b8;
        const u32 cnt = __popc(more), inc = scan(cnt), o3 = run + inc - cnt;
        run += last(inc);
        const u32 c0 = spread1(a8 & 15u) | spread2(b8 & 15u), c1 = spread1(a8 >> 4) | spread2(b8 >> 4);
        u32 s0, s1;
        if (wide) fetch8(l3 + o3, s0, s1);
        else {
            const u32 bit = l3 * 8u + o3 * 4u, q = (bit >> 5) << 2;
            const u32 xx = __builtin_amdgcn_alignbit(ld32(q + 4), ld32(q), bit);
            const u32 lo = xx & 0x0f0f0f0fu, hi = (xx >> 4) & 0x0f0f0f0fu;
            s0 = perm(hi, lo, 0x05010400u); s1 = perm(hi, lo, 0x07030602u);
            const u32 g0 = ((s0 + 0x0e0e0e0eu) >> 4) & 0x01010101u, g1 = ((s1 + 0x0e0e0e0eu) >> 4) & 0x01010101u;
            s0 = top - s0 - g0 * 6u; s1 = top - s1 - g1 * 6u;
        }
        const u32x2 sel = ld64(L_LUTE + more * 8u);
        st64(t2 + idx * 8u, perm(0u, tab, c0) | perm(s1, s0, sel.x), perm(0u, tab, c1) | perm(s1, s0, sel.y));
    }
    CBAR();
}

__device__ __forceinline__ void dec_input(u32 rec, u32 hdr, bool wide, u32 tmp, char* dst, int lane) {
    const u32 n2 = min((hdr >> 12) & 0xfffu, 2048u), base = hdr >> 24, t2 = tmp + 2 * TB;
    dec_l2(rec + 2560, n2, base, wide, t2, lane);
    const u32 pa = ld32(rec + 2048 + lane * 4), pb = ld32(rec + 2304 + lane * 4), more = pa & pb;
    const u32 tab = base | ((base + 1) << 8) | ((base + 2) << 16);
    u32 o[4]; offsets4(more, o);
#pragma unroll
    for (int j = 0; j < 4; ++j) {
        const u32x2 mb = ld64(rec + j * 512 + lane * 8);
        const u32 c0 = spread1((pa >> (8 * j)) & 15u) | spread2((pb >> (8 * j)) & 15u);
        const u32 c1 = spread1((pa >> (8 * j + 4)) & 15u) | spread2((pb >> (8 * j + 4)) & 15u);
        u32 s0, s1; fetch8(t2 + o[j], s0, s1);
        const u32x2 sel = ld64(L_LUTE + ((more >> (8 * j)) & 255u) * 8u);
        const u32 e0 = perm(0u, tab, c0) | perm(s1, s0, sel.x), e1 = perm(0u, tab, c1) | perm(s1, s0, sel.y);
        const u32 h0 = bfi(0x80808080u, mb.x, e0 >> 1), h1 = bfi(0x80808080u, mb.y, e1 >> 1);
        const u32 l0 = bfi(0x80808080u, e0 << 7, mb.x), l1 = bfi(0x80808080u, e1 << 7, mb.y);
        gst_agent(dst + j * 1024, (u32x4){perm(h0, l0, 0x05010400u), perm(h0, l0, 0x07030602u), perm(h1, l1, 0x05010400u), perm(h1, l1, 0x07030602u)});
    }
}

__device__ __forceinline__ void dec_output(u32 rec, u32 hdr, bool wide, u32 tmp, char* dst, int lane) {
    const u32 n1 = min(hdr & 0xfffu, 2048u), n2 = min((hdr >> 12) & 0xfffu, 2048u), base = hdr >> 24;
    const u32 t1l = tmp, t1h = tmp + TB, t2 = tmp + 2 * TB;
    const u32 pbo = rec + 256 + n1, nb = (n1 + 7) >> 3;
    dec_l2(pbo + nb, n2, base, wide, t2, lane);
    const u32 tab = base | ((base + 1) << 8) | ((base + 2) << 16);
    u32 run = 0;
    for (u32 p = 0; p < nb; p += 64) {
        const u32 idx = p + lane;
        const u32x2 bb = ld64(rec + 256 + idx * 8u);
        const u32 b8 = ld8(pbo + idx);
        const u32 a0 = (bb.x >> 7) & 0x01010101u, a1 = (bb.y >> 7) & 0x01010101u;
        const u32 more = gather8(a0, a1) & b8;
        const u32 cnt = __popc(more), inc = scan(cnt), o2 = run + inc - cnt;
        run += last(inc);
        const u32 c0 = a0 | spread2(b8 & 15u), c1 = a1 | spread2(b8 >> 4);
        u32 s0, s1; fetch8(t2 + o2, s0, s1);
        const u32x2 sel = ld64(L_LUTE + more * 8u);
        const u32 e0 = perm(0u, tab, c0) | perm(s1, s0, sel.x), e1 = perm(0u, tab, c1) | perm(s1, s0, sel.y);
        st64(t1l + idx * 8u, bfi(0x80808080u, e0 << 7, bb.x), bfi(0x80808080u, e1 << 7, bb.y));
        st64(t1h + idx * 8u, (e0 >> 1) & 0x7f7f7f7fu, (e1 >> 1) & 0x7f7f7f7fu);
    }
    CBAR();
    const u32 m = ld32(rec + lane * 4);
    u32 o[4]; offsets4(m, o);
#pragma unroll
    for (int j = 0; j < 4; ++j) {
        u32 l0, l1, h0, h1;
        fetch8(t1l + o[j], l0, l1); fetch8(t1h + o[j], h0, h1);
        const u32x2 sel = ld64(L_LUTE + ((m >> (8 * j)) & 255u) * 8u);
        const u32 L0 = perm(l1, l0, sel.x), L1 = perm(l1, l0, sel.y), H0 = perm(h1, h0, sel.x), H1 = perm(h1, h0, sel.y);
        gst_agent(dst + (size_t)(j >> 1) * (16 * NFULL * 2) + (j & 1) * 64,
                  (u32x4){perm(H0, L0, 0x05010400u), perm(H0, L0, 0x07030602u), perm(H1, L1, 0x05010400u), perm(H1, L1, 0x07030602u)});
    }
}

__device__ __forceinline__ u32 enc_l2(u32 l2a, u32 n2, u32 base, u32 t2, bool& wide, int lane) {
    const u32 pb = (n2 + 7) >> 3, l2b = l2a + pb, l3 = l2b + pb;
    const u32 top = (base + 5) * 0x01010101u;
    u32 run;
    wide = false;
    for (;;) {
        if (lane < 16) st8(l3 + lane, 0u);
        zero_lds((l3 + 15u) & ~15u, (wide ? n2 : n2 >> 1) + 8u, lane);
        CBAR();
        u32 out = 0;
        run = 0;
        for (u32 p = 0; p < pb; p += 64) {
            const u32 idx = p + lane;
            const u32x2 e = ld64(t2 + idx * 8u);
            const int left = (int)n2 - (int)(idx * 8u);
            const u32 valid = left >= 8 ? 255u : (left > 0 ? (1u << left) - 1u : 0u);
            const u32 h0 = subb(top, e.x), h1 = subb(top, e.y);
            const u32 n0 = ~geb<8>(h0) & (h0 >> 1), n1 = ~geb<8>(h1) & (h1 >> 1);
            const u32 more = ~gather8(n0 & 0x01010101u, n1 & 0x01010101u) & valid;
            const u32 a8 = gather8((h0 >> 2) & ~h0 & 0x01010101u, (h1 >> 2) & ~h1 & 0x01010101u) | more;
            const u32 b8 = ~gather8((h0 >> 2) & 0x01010101u, (h1 >> 2) & 0x01010101u) | more;
            if (left > 0) { st8(l2a + idx, a8 & valid); st8(l2b + idx, b8 & valid); }
            const u32 cnt = __popc(more), inc = scan(cnt), o3 = run + inc - cnt;
            run += last(inc);
            const u32x2 sel = ld64(L_LUTC + more * 8u);
            if (wide) scatter8(l3 + o3, perm(e.y, e.x, sel.x), perm(e.y, e.x, sel.y));
            else {
                u32 c0 = perm(h1, h0, sel.x), c1 = perm(h1, h0, sel.y);
                out |= geb<22>(c0) | geb<22>(c1);
                c0 -= (((c0 + 0x08080808u) >> 4) & 0x01010101u) * 6u; c1 -= (((c1 + 0x08080808u) >> 4) & 0x01010101u) * 6u;
                const u32 xx = perm(c1 | (c1 >> 4), c0 | (c0 >> 4), 0x06040200u);
                const u32 bit = l3 * 8u + o3 * 4u, q = (bit >> 5) << 2;
                const u64 w = (u64)xx << (bit & 31u);
                if ((u32)w) or32(q, (u32)w);
                if ((u32)(w >> 32)) or32(q + 4, (u32)(w >> 32));
            }
        }
        CBAR();
        if (wide || !__any(out != 0)) break;
        wide = true;
    }
    return l3 + (wide ? run : (run + 1) >> 1);
}

__device__ __forceinline__ void enc_input(const u32x4 (&v)[4], int wid, int lane) {
    const u32 rec = L_RB + wid * ENC_W, t2 = rec + 5248 + 2 * TB;
    u32 e[4][2], sum = 0;
#pragma unroll
    for (int j = 0; j < 4; ++j) {
        const u32 l0 = perm(v[j].y, v[j].x, 0x06040200u), h0 = perm(v[j].y, v[j].x, 0x07050301u);
        const u32 l1 = perm(v[j].w, v[j].z, 0x06040200u), h1 = perm(v[j].w, v[j].z, 0x07050301u);
        e[j][0] = bfi(0x01010101u, l0 >> 7, h0 << 1); e[j][1] = bfi(0x01010101u, l1 >> 7, h1 << 1);
        st64(rec + j * 512 + lane * 8, bfi(0x80808080u, h0, l0), bfi(0x80808080u, h1, l1));
        sum = __builtin_amdgcn_udot4(e[j][0], 0x01010101u, __builtin_amdgcn_udot4(e[j][1], 0x01010101u, sum, false), false);
    }
    const int tot = (int)last(scan(sum));
    const u32 base = (u32)min(max((tot - 369) >> 11, 16), 250);
    const u32 brep = base * 0x01010101u;
    u32 pa = 0, pb = 0, more = 0;
#pragma unroll
    for (int j = 0; j < 4; ++j) {
        const u32 d0 = subb(e[j][0], brep), d1 = subb(e[j][1], brep);
        more |= gather8(geb<3>(d0), geb<3>(d1)) << (8 * j);
        pa |= gather8(d0 & 0x01010101u, d1 & 0x01010101u) << (8 * j);
        pb |= gather8((d0 >> 1) & 0x01010101u, (d1 >> 1) & 0x01010101u) << (8 * j);
    }
    st32(rec + 2048 + lane * 4, pa | more); st32(rec + 2304 + lane * 4, pb | more);
    u32 o[4]; const u32 n2 = offsets4(more, o);
    zero_lds(t2, n2, lane);
    CBAR();
#pragma unroll
    for (int j = 0; j < 4; ++j) {
        const u32x2 sel = ld64(L_LUTC + ((more >> (8 * j)) & 255u) * 8u);
        scatter8(t2 + o[j], perm(e[j][1], e[j][0], sel.x), perm(e[j][1], e[j][0], sel.y));
    }
    CBAR();
    bool wide; const u32 end = enc_l2(rec + 2560, n2, base, t2, wide, lane);
    if (lane == 0) { *ctl(C_HDR + wid) = (n2 << 12) | (base << 24); *ctl(C_SIZE + wid) = ((end - rec + 15) >> 4) | (wide ? 0x8000u : 0u); }
}

__device__ __forceinline__ void enc_output(const u32x4 (&v)[4], int wid, int lane) {
    const u32 rec = L_RB + wid * ENC_W, t1l = rec + 5248, t1h = t1l + TB, t2 = t1h + TB;
    u32 l[4][2], h[4][2], m = 0, sum = 0;
#pragma unroll
    for (int j = 0; j < 4; ++j) {
        l[j][0] = perm(v[j].y, v[j].x, 0x06040200u); h[j][0] = perm(v[j].y, v[j].x, 0x07050301u);
        l[j][1] = perm(v[j].w, v[j].z, 0x06040200u); h[j][1] = perm(v[j].w, v[j].z, 0x07050301u);
        m |= gather8(geb<1>(l[j][0] | h[j][0]), geb<1>(l[j][1] | h[j][1])) << (8 * j);
        sum = __builtin_amdgcn_udot4(bfi(0x01010101u, l[j][0] >> 7, h[j][0] << 1), 0x01010101u, sum, false);
        sum = __builtin_amdgcn_udot4(bfi(0x01010101u, l[j][1] >> 7, h[j][1] << 1), 0x01010101u, sum, false);
    }
    st32(rec + lane * 4, m);
    u32 o[4]; const u32 n1 = offsets4(m, o);
    for (u32 i = lane * 16; i < n1; i += 1024) { const u32x4 z = {0u, 0u, 0u, 0u}; st128(t1l + i, z); st128(t1h + i, z); st128(t2 + i, z); }
    const u32 tot = last(scan(sum));
    const u32 base = uni((u32)min(max((int)((float)tot / (float)max(n1, 1u) - 0.18f), 16), 250));
    const u32 brep = base * 0x01010101u;
    CBAR();
#pragma unroll
    for (int j = 0; j < 4; ++j) {
        const u32x2 sel = ld64(L_LUTC + ((m >> (8 * j)) & 255u) * 8u);
        scatter8(t1l + o[j], perm(l[j][1], l[j][0], sel.x), perm(l[j][1], l[j][0], sel.y));
        scatter8(t1h + o[j], perm(h[j][1], h[j][0], sel.x), perm(h[j][1], h[j][0], sel.y));
    }
    CBAR();
    const u32 nb = (n1 + 7) >> 3, pbo = rec + 256 + n1;
    u32 run = 0, pbs = 0;
#pragma unroll
    for (int p = 0; p < 4; ++p) {
        if ((u32)p * 64u < nb) {
            const u32 idx = p * 64 + lane;
            const u32x2 ll = ld64(t1l + idx * 8u), hh = ld64(t1h + idx * 8u);
            const int left = (int)n1 - (int)(idx * 8u);
            const u32 valid = left >= 8 ? 255u : (left > 0 ? (1u << left) - 1u : 0u);
            const u32 e0 = bfi(0x01010101u, ll.x >> 7, hh.x << 1), e1 = bfi(0x01010101u, ll.y >> 7, hh.y << 1);
            const u32 d0 = subb(e0, brep), d1 = subb(e1, brep);
            const u32 g0 = geb<3>(d0), g1 = geb<3>(d1);
            const u32 more = gather8(g0, g1) & valid;
            if (left > 0) st64(rec + 256 + idx * 8u, bfi(0x80808080u, (d0 | g0) << 7, ll.x), bfi(0x80808080u, (d1 | g1) << 7, ll.y));
            pbs |= ((gather8((d0 >> 1) & 0x01010101u, (d1 >> 1) & 0x01010101u) | more) & valid) << (8 * p);
            const u32 cnt = __popc(more), inc = scan(cnt), o2 = run + inc - cnt;
            run += last(inc);
            const u32x2 sel = ld64(L_LUTC + more * 8u);
            scatter8(t2 + o2, perm(e1, e0, sel.x), perm(e1, e0, sel.y));
        }
    }
    CBAR();
#pragma unroll
    for (int p = 0; p < 4; ++p)
        if ((u32)(p * 64 + lane) < nb) st8(pbo + p * 64 + lane, pbs >> (8 * p));
    CBAR();
    bool wide; const u32 end = enc_l2(pbo + nb, run, base, t2, wide, lane);
    if (lane == 0) { *ctl(C_HDR + wid) = n1 | (run << 12) | (base << 24); *ctl(C_SIZE + wid) = ((end - rec + 15) >> 4) | (wide ? 0x8000u : 0u); }
}

__device__ __forceinline__ u32 emit_block(char* slot, int wid, int lane) {
    const u32 rec = L_RB + wid * ENC_W + lane * 16;
    CBAR();
    const u32x4 r0 = ld128(rec), r1 = ld128(rec + 1024), r2 = ld128(rec + 2048);
    wg_barrier();
    const u32 sz = *ctl(C_SIZE + (lane & 7));
    u32 inc = sz & 0x7fffu;
    inc += __builtin_amdgcn_update_dpp(0u, inc, 0x111, 0xf, 0xf, false);
    inc += __builtin_amdgcn_update_dpp(0u, inc, 0x112, 0xf, 0xf, false);
    inc += __builtin_amdgcn_update_dpp(0u, inc, 0x114, 0xf, 0xf, false);
    inc += 3u;
    const u32 mysz = __builtin_amdgcn_readlane(sz, wid) & 0x7fffu, myend = __builtin_amdgcn_readlane(inc, wid);
    const u32 total = __builtin_amdgcn_readlane(inc, 7);
    char* d = slot + (size_t)(myend - mysz + lane) * 16;
    if ((u32)lane < mysz) gst_sys(d, r0);
    if ((u32)lane + 64u < mysz) gst_sys(d + 1024, r1);
    if ((u32)lane + 128u < mysz) gst_sys(d + 2048, r2);
    for (u32 i = 192; i + lane < mysz; i += 64) gst_sys(d + (size_t)i * 16, ld128(rec + i * 16));
    if (wid == 0) {
        const u32 en = inc | (sz & 0x8000u);
        u32x4 hv;
        hv.x = __builtin_amdgcn_readlane(en, 0) | (__builtin_amdgcn_readlane(en, 1) << 16);
        hv.y = __builtin_amdgcn_readlane(en, 2) | (__builtin_amdgcn_readlane(en, 3) << 16);
        hv.z = __builtin_amdgcn_readlane(en, 4) | (__builtin_amdgcn_readlane(en, 5) << 16);
        hv.w = __builtin_amdgcn_readlane(en, 6) | (__builtin_amdgcn_readlane(en, 7) << 16);
        if (lane < 2) hv = ld128(L_CTL + 4 * C_HDR + lane * 16);
        if (lane < 3) gst_sys(slot + lane * 16, hv);
    }
    vm_drain();
    wg_barrier();
    return total;
}

__device__ __forceinline__ void flag_store(const Params& P, int dest, int pos, u32 val) {
    u32* f = (u32*)(P.ws[dest] + OFF_FL) + (P.rank * 1024 + pos);
    __hip_atomic_store((GAS u32*)f, val, __ATOMIC_RELAXED, __HIP_MEMORY_SCOPE_SYSTEM);
}
__device__ __forceinline__ void flag_all(const Params& P, int pos, u32 val) {
#pragma unroll
    for (int p = 0; p < NR; ++p) if (p != P.rank) flag_store(P, p, pos, val);
}
__device__ __forceinline__ void ready_add(char* ws, int idx, int local) {
    __hip_atomic_fetch_add((GAS u32*)(ws + OFF_RDY + idx * 128 + local * 64), 1u, __ATOMIC_RELAXED, __HIP_MEMORY_SCOPE_AGENT);
}
__device__ __forceinline__ char* ws_of(const Params& P, int r) {
    char* p = P.ws[0];
#pragma unroll
    for (int i = 1; i < NR; ++i) p = r == i ? P.ws[i] : p;
    return p;
}

#define STAMP(i) do {} while (0)
#define STAMPW(i) do {} while (0)

__device__ __forceinline__ bool pos_is_x(int pos) { return pos < 16 || (pos >= 48 && pos < 144) || pos >= 176; }
__device__ __forceinline__ int pos_xb(int pos) { return pos < 16 ? pos : (pos < 144 ? pos - 32 : pos - 64); }
__device__ __forceinline__ int pos_wb(int pos) { return pos < 48 ? pos - 16 : pos - 112; }
__device__ __forceinline__ void task_of(int rank, int task, int& pos, int& dest) {
    dest = -1;
    if (task < 16) pos = task;
    else if (task < 272) { pos = 16 + ((task - 16) >> 3); dest = (rank + 1 + (task & 7)) & 7; }
    else if (task < 368) pos = 48 + task - 272;
    else if (task < 624) { pos = 144 + ((task - 368) >> 3); dest = (rank + 1 + (task & 7)) & 7; }
    else pos = 176 + task - 624;
}
__device__ __forceinline__ void pack_load(const Params& P, int task, int wid, int lane, u32x4 (&t)[4]) {
    int pos, dest; task_of(P.rank, task, pos, dest);
    const char* s; size_t s1, s2;
    if (dest < 0) {
        const int xb = pos_xb(pos);
        s = P.x + (size_t)((xb >> 3) * 256 + wid * 32 + (lane & 15)) * 1024 + (xb & 7) * 128 + (lane >> 4) * 16;
        s1 = 64; s2 = 16384;
    } else {
        const int wb = pos_wb(pos);
        s = P.w + (size_t)((wb & 3) * 128 + (wid >> 2) * 64 + (lane >> 2)) * (NFULL * 2) + (dest * 2048 + (wb >> 2) * 128 + (wid & 3) * 32) * 2 + (lane & 3) * 16;
        s1 = 16 * NFULL * 2; s2 = 2 * s1;
    }
    t[0] = *(const GAS u32x4*)s; t[1] = *(const GAS u32x4*)(s + s1); t[2] = *(const GAS u32x4*)(s + s2); t[3] = *(const GAS u32x4*)(s + s1 + s2);
}
__device__ __forceinline__ void pack_task(const Params& P, char* ws, int task, u32 call, int wid, int lane, u32x4 (&v)[4]) {
    int pos, dest; task_of(P.rank, task, pos, dest);
    size_t lo, so; int rdy;
    if (dest < 0) {
        const int xb = pos_xb(pos);
        rdy = xb >> 3; lo = OFF_XA + (size_t)(rdy * 64 + (xb & 7)) * 32768; so = OFF_CX + (size_t)xb * SLOT;
    } else {
        const int wb = pos_wb(pos);
        rdy = 16 + (wb >> 2); lo = OFF_WB + (size_t)((wb >> 2) * 64 + (wb & 3) * 2) * 16384; so = OFF_CW + (size_t)(dest * 64 + wb) * SLOT;
        const u32 tr = L_RB + wid * ENC_W + 5248;
#pragma unroll
        for (int i = 0; i < 4; ++i) st128(tr + i * 1024 + lane * 16, v[i]);
        CBAR();
        const u32 ra = tr + (lane >> 4) * 512 + (((lane >> 2) & 3) * 8 + (lane & 3)) * 2;
#pragma unroll
        for (int j = 0; j < 4; ++j) {
            u32 q[8];
#pragma unroll
            for (int e = 0; e < 8; ++e) q[e] = *(LAS unsigned short*)lp(ra + (j & 1) * 2048 + e * 64 + (j >> 1) * 8);
            v[j] = (u32x4){q[0] | (q[1] << 16), q[2] | (q[3] << 16), q[4] | (q[5] << 16), q[6] | (q[7] << 16)};
        }
        CBAR();
    }
    const bool local = dest < 0 || dest == P.rank;
    if (local) {
        char* d = ws + lo + wid * 4096 + lane * 16;
#pragma unroll
        for (int j = 0; j < 4; ++j) gst_agent(d + j * 1024, v[j]);
    }
    u32 len = 0;
    if (dest != P.rank) { enc_input(v, wid, lane); len = emit_block(ws + so, wid, lane); }
    else { vm_drain(); wg_barrier(); }
    if (threadIdx.x == 0) {
        if (dest < 0) flag_all(P, pos, (call << 12) | len);
        else if (!local) __hip_atomic_store((GAS u32*)(ws_of(P, dest) + OFF_FL) + (P.rank * 1024 + pos), (call << 12) | len, __ATOMIC_RELAXED, __HIP_MEMORY_SCOPE_SYSTEM);
        if (local) ready_add(ws, rdy, 1);
    }
    JOIN();
}

__device__ __forceinline__ void tile_of(int set, int t16, int& pm, int& pn) {
    const bool first = set == 0 || set == 7;
    const int pair = set < 7 ? set : (set < 14 ? set - 7 : 7), half = set < 7 ? 0 : (set < 14 ? 1 : set & 1);
    pm = 2 * pair + (first ? t16 & 1 : t16 >> 3); pn = 8 * half + (first ? t16 >> 1 : t16 & 7);
}

__device__ __forceinline__ void decoder(const Params& P, char* ws, int src, int d, u32 call, int wid, int lane) {
    const u32 seq = call & 0xfffffu;
    const u64 tstart = now();
    GAS u32* tmo = (GAS u32*)(ws + OFF_TMO);
    if (wid == 1) {
        const GAS u32* fl = (const GAS u32*)(ws + OFF_FL) + src * 1024;
        bool dead = false;
        for (u32 i = 0;; ++i) {
            u32 pos = d;
            if (i) {
                while (ctl_ld(C_PROG) < i) { if (now() - tstart > 2 * DEADLINE) break; __builtin_amdgcn_s_sleep(2); }
                u32 t = 0;
                if (lane == 0) t = __hip_atomic_fetch_add((GAS u32*)(ws + OFF_CLM + src * 128), 1u, __ATOMIC_RELAXED, __HIP_MEMORY_SCOPE_AGENT);
                pos = uni(t) - (call - 1u) * (u32)NPOS + (u32)DPP_;
            }
            if (pos >= (u32)NPOS) { *ctl(C_QUEUE + (i & 7u)) = ((i + 1u) << 22) | (1023u << 12); break; }
            u32 f = 0;
            while (!dead) {
                f = uni(__hip_atomic_load(fl + pos, __ATOMIC_RELAXED, __HIP_MEMORY_SCOPE_SYSTEM));
                if ((f >> 12) == seq) break;
                if (uni(__hip_atomic_load(tmo, __ATOMIC_RELAXED, __HIP_MEMORY_SCOPE_AGENT)) == call || now() - tstart > DEADLINE) {
                    __hip_atomic_store(tmo, call, __ATOMIC_RELAXED, __HIP_MEMORY_SCOPE_AGENT);
                    *ctl(C_DEAD) = 1; dead = true; f = 0;
                }
                __builtin_amdgcn_s_sleep(1);
            }
            CBAR();
            *ctl(C_QUEUE + (i & 7u)) = ((i + 1u) << 22) | (pos << 12) | (f & 0xfffu);
        }
    } else if (wid == 0) {
        const char* sws = ws_of(P, src);
        u32 t0 = 0, t1 = 0, t2 = 0, cur = 0;
        u32 i = 0;
        for (;; ++i) {
            const u32 need = 8u * (i >> 1);
            u32 f = ctl_ld(C_QUEUE + (i & 7u));
            if ((f >> 22) != i + 1u || ctl_ld(C_DONE + (i & 1u)) < need) {
                asm volatile("s_waitcnt vmcnt(2)" ::: "memory"); if (t0 > cur) *ctl(C_LANDED) = t0;
                asm volatile("s_waitcnt vmcnt(1)" ::: "memory"); if (t1 > cur) *ctl(C_LANDED) = t1;
                vm_drain();
                cur = i << 6; *ctl(C_LANDED) = cur;
                while (((f = ctl_ld(C_QUEUE + (i & 7u))) >> 22) != i + 1u || ctl_ld(C_DONE + (i & 1u)) < need) {
                    if (now() - tstart > 2 * DEADLINE) { f = 1023u << 12; break; }
                    __builtin_amdgcn_s_sleep(1);
                }
            }
            CBAR();
            const int pos = (f >> 12) & 1023u;
            if (pos >= NPOS) break;
            *ctl(C_PROG) = i + 1u;
            size_t so;
            if (pos < NIN) so = pos_is_x(pos) ? OFF_CX + (size_t)pos_xb(pos) * SLOT : OFF_CW + (size_t)(P.rank * 64 + pos_wb(pos)) * SLOT;
            else so = OFF_CO + (size_t)(pos - NIN) * SLOT;
            const u32 len = min(f & 0xfffu, (u32)(LSLOT / 16));
            const char* s = sws + so + lane * 16;
            const u32 lb = (u32)(size_t)lp(L_RB + (i & 1u) * LSLOT);
            for (u32 k = 0; k * 64u < len; ++k) {
                if (k * 64u + lane < len) {
                    u32 keep;
                    asm volatile("s_mov_b32 %0, m0\n\ts_mov_b32 m0, %2\n\ts_nop 0\n\tglobal_load_lds_dwordx4 %1, off sc0 sc1\n\ts_mov_b32 m0, %0"
                                 : "=&s"(keep) : "v"(s + (size_t)k * 1024), "s"(lb + k * 1024u) : "memory");
                }
                asm volatile("s_waitcnt vmcnt(3)" ::: "memory");
                if (t0 > cur) { cur = t0; *ctl(C_LANDED) = cur; }
                t0 = t1; t1 = t2; t2 = (i << 6) + k + 1;
            }
        }
        asm volatile("s_waitcnt vmcnt(2)" ::: "memory"); if (t0 > cur) *ctl(C_LANDED) = t0;
        asm volatile("s_waitcnt vmcnt(1)" ::: "memory"); if (t1 > cur) *ctl(C_LANDED) = t1;
        vm_drain();
        *ctl(C_NBLK) = i + 1u;
        *ctl(C_LANDED) = 0xffffffffu;
        STAMPW(50);
    } else {
        const u32 tmp = L_DT + (wid - 2) * 3 * TB;
        const int slab = (src - P.rank) & 7;
        for (;;) {
            const u32 r = ctl_inc(C_CLAIM);
            const u32 i = r >> 3, sub = r & 7u;
            const u32 slot = L_RB + (i & 1u) * LSLOT;
            bool late = false;
            while (ctl_ld(C_LANDED) < (i << 6) + 1u) { if (now() - tstart > 2 * DEADLINE) { late = true; break; } __builtin_amdgcn_s_sleep(1); }
            CBAR();
            if (late || ctl_ld(C_NBLK) - 1u <= i) break;
            const int pos = min((ctl_ld(C_QUEUE + (i & 7u)) >> 12) & 1023u, (u32)(NPOS - 1));
            const u32 hdr = uni(ld32(slot + sub * 4));
            const u32 en = uni(*(LAS unsigned short*)lp(slot + 32 + sub * 2));
            const u32 st = sub ? (uni(*(LAS unsigned short*)lp(slot + 30 + sub * 2)) & 0x7fffu) : 3u;
            const u32 endp = min(((en & 0x7fffu) + 63u) >> 6, 63u);
            while (ctl_ld(C_LANDED) < (i << 6) + endp) { if (now() - tstart > 2 * DEADLINE) break; __builtin_amdgcn_s_sleep(1); }
            CBAR();
            const bool dead = ctl_ld(C_DEAD) != 0;
            const u32 rec = slot + min(st, (u32)(LSLOT / 16)) * 16u;
            if (pos < NIN) {
                size_t o; int rdy;
                if (pos_is_x(pos)) { const int xb = pos_xb(pos); rdy = xb >> 3; o = OFF_XA + (size_t)(rdy * 64 + slab * 8 + (xb & 7)) * 32768; }
                else { const int wb = pos_wb(pos); rdy = 16 + (wb >> 2); o = OFF_WB + (size_t)((wb >> 2) * 64 + slab * 8 + (wb & 3) * 2) * 16384; }
                if (!dead) dec_input(rec, hdr, (en >> 15) != 0, tmp, ws + o + sub * 4096 + lane * 16, lane);
                asm volatile("s_waitcnt lgkmcnt(0)" ::: "memory");
                ctl_inc(C_DONE + (i & 1u));
                vm_drain();
                if ((ctl_inc(C_STORED + (i & 15u)) & 7u) == 7u) { if (lane == 0) ready_add(ws, rdy, 0); JOIN(); }
            } else {
                const int ob = pos - NIN;
                int pm, pn; tile_of(ob >> 5, ob & 15, pm, pn);
                char* dst = P.out + ((size_t)(pm * 256 + (sub >> 1) * 64 + ((ob >> 4) & 1) * 32 + (lane & 15)) * NFULL + src * 2048 + pn * 128 + (sub & 1) * 64 + (lane >> 4) * 8) * 2;
                if (!dead) dec_output(rec, hdr, (en >> 15) != 0, tmp, dst, lane);
                else gst_agent(dst, (u32x4){0x7fc07fc0u, 0x7fc07fc0u, 0x7fc07fc0u, 0x7fc07fc0u});
                asm volatile("s_waitcnt lgkmcnt(0)" ::: "memory");
                ctl_inc(C_DONE + (i & 1u));
            }
        }
    }
}

__device__ __forceinline__ bool wait_operands(char* ws, int pm, int pn, u32 call, u32 mask, int lane) {
    const GAS u32* a = (const GAS u32*)(lane < 4 ? ws + OFF_RDY + ((lane & 1) ? 16 + pn : pm) * 128 + (lane < 2 ? 64 : 0) : ws + OFF_TMO);
    const u32 need = call * (lane == 0 ? 8u : lane == 1 ? 4u : lane == 2 ? 56u : 28u);
    const u64 t0 = now();
    u32 ok;
    for (;;) {
        const u32 v = __hip_atomic_load(a, __ATOMIC_RELAXED, __HIP_MEMORY_SCOPE_AGENT);
        ok = (u32)__ballot((int)(v - need) >= 0) & 15u;
        const bool dead = __builtin_amdgcn_readlane(v, 4) == call;
        if (!dead && (ok & mask) == mask) break;
        if (dead || now() - t0 > DEADLINE) {
            __hip_atomic_store((GAS u32*)(ws + OFF_TMO), call, __ATOMIC_RELAXED, __HIP_MEMORY_SCOPE_AGENT);
            *ctl(C_DEAD) = 1; ok = 15u;
            break;
        }
        __builtin_amdgcn_s_sleep(4);
    }
    __builtin_amdgcn_fence(__ATOMIC_ACQUIRE, "agent");
    vm_drain();
    return ok == 15u;
}

__device__ __forceinline__ void gemm_tile(const Params& P, char* ws, int set, int t16, u32 call, int wid, int lane) {
    int pm, pn; tile_of(set, t16, pm, pn);
    bool have_all = true;
    if (wid == 0) have_all = wait_operands(ws, pm, pn, call, 3u, lane);
    wg_barrier();
    STAMP(8 + (set >> 3) * 4);
    const char* ga = ws + OFF_XA + (size_t)pm * (2u << 20);
    const char* gb = ws + OFF_WB + (size_t)pn * (1u << 20);
    const u32 voa = wid * 4096 + lane * 16, vob = wid * 2048 + lane * 16;
    const u32 lbase = (u32)(size_t)lp(L_RB);
    const u32 lda = lbase + wid * 4096, ldb = lbase + 32768 + wid * 2048;
#define GLDS(vo, gp, lo, off) do { u32 keep_; asm volatile( \
        "s_mov_b32 %0, m0\n\ts_mov_b32 m0, %3\n\ts_nop 3\n\tglobal_load_lds_dwordx4 %1, %2 offset:" #off "\n\ts_mov_b32 m0, %0" \
        : "=&s"(keep_) : "v"(vo), "s"(gp), "s"(lo) : "memory"); } while (0)
#define GLDF(vo, gp, lo, off) asm volatile("s_mov_b32 m0, %2\n\ts_nop 0\n\tglobal_load_lds_dwordx4 %0, %1 offset:" #off ::"v"(vo), "s"(gp), "s"(lo) : "memory")
#define KA(kt) (ga + (size_t)(kt) * 32768)
#define KB(kt) (gb + (size_t)(kt) * 16384)
#define STAGE_LD(so, kt) do { const char* a_ = KA(kt); const char* b_ = KB(kt); \
        GLDS(voa, a_, lda + (so), 0); GLDS(voa, a_, lda + (so), 1024); GLDS(voa, a_, lda + (so), 2048); GLDS(voa, a_, lda + (so), 3072); \
        GLDS(vob, b_, ldb + (so), 0); GLDS(vob, b_, ldb + (so), 1024); } while (0)
    const u32 ra = L_RB + (wid >> 1) * 8192 + lane * 16, rb = L_RB + 32768 + (wid & 1) * 8192 + lane * 16;
#define FRAGS(A, B, so, kb) do { _Pragma("unroll") for (int q_ = 0; q_ < 4; ++q_) { \
        B[q_] = *(const LAS bf16x8*)lp(rb + (so) + q_ * 2048 + (kb) * 1024); A[q_] = *(const LAS bf16x8*)lp(ra + (so) + q_ * 2048 + (kb) * 1024); } } while (0)
#define MROW(A, B, m_) do { _Pragma("unroll") for (int n_ = 0; n_ < 4; ++n_) \
        acc[m_][n_] = __builtin_amdgcn_mfma_f32_16x16x32_bf16(B[n_], A[m_], acc[m_][n_], 0, 0, 0); } while (0)
#define SB() __builtin_amdgcn_sched_barrier(0)
#define MM1(A, B, m_, n_) acc[m_][n_] = __builtin_amdgcn_mfma_f32_16x16x32_bf16(B[n_], A[m_], acc[m_][n_], 0, 0, 0)
#define RDB(B, q_, kb) B[q_] = *(const LAS bf16x8*)lp(rb + so + (q_) * 2048 + (kb) * 1024)
#define RDA(A, q_, kb) A[q_] = *(const LAS bf16x8*)lp(ra + so + (q_) * 2048 + (kb) * 1024)
#define HALF(A, B, NA, NB, kb, G0, G1, G2) do { SB(); \
        MM1(A, B, 0, 0); SB(); RDA(NA, 0, kb); SB(); MM1(A, B, 0, 1); SB(); RDB(NB, 0, kb); SB(); MM1(A, B, 0, 2); SB(); RDB(NB, 1, kb); SB(); MM1(A, B, 0, 3); SB(); RDB(NB, 2, kb); SB(); \
        MM1(A, B, 1, 0); SB(); RDB(NB, 3, kb); SB(); MM1(A, B, 1, 1); SB(); RDA(NA, 1, kb); SB(); MM1(A, B, 1, 2); SB(); RDA(NA, 2, kb); SB(); MM1(A, B, 1, 3); SB(); RDA(NA, 3, kb); SB(); \
        MM1(A, B, 2, 0); SB(); G0; SB(); MM1(A, B, 2, 1); MM1(A, B, 2, 2); SB(); G1; SB(); MM1(A, B, 2, 3); \
        MM1(A, B, 3, 0); SB(); G2; SB(); MM1(A, B, 3, 1); MM1(A, B, 3, 2); MM1(A, B, 3, 3); SB(); } while (0)
#define MMA(A, B) do { MROW(A, B, 0); MROW(A, B, 1); MROW(A, B, 2); MROW(A, B, 3); } while (0)
    f32x4 acc[4][4];
#pragma unroll
    for (int m = 0; m < 4; ++m)
#pragma unroll
        for (int n = 0; n < 4; ++n) acc[m][n] = (f32x4){0.f, 0.f, 0.f, 0.f};
    bf16x8 a0[4], b0[4], a1[4], b1[4];
    STAGE_LD(0, 0); STAGE_LD(STAGE, 1);
    { const char* a_ = KA(2); GLDS(voa, a_, lda + 2 * STAGE, 0); GLDS(voa, a_, lda + 2 * STAGE, 1024); GLDS(voa, a_, lda + 2 * STAGE, 2048); }
    asm volatile("s_waitcnt vmcnt(9)\n\ts_barrier" ::: "memory");
    FRAGS(a0, b0, 0u, 0);
    u32 so = 0, sn, sp = 2 * STAGE;
    const char* pa = KA(2); const char* pb = KB(2);
#pragma unroll 1
    for (int phase = 0; phase < 2; ++phase) {
#pragma unroll 1
        for (int n = phase ? 56 : 5; n > 0; --n) {
            HALF(a0, b0, a1, b1, 1, GLDF(voa, pa, lda + sp, 3072), GLDF(vob, pb, ldb + sp, 0),
                 GLDF(vob, pb, ldb + sp, 1024); pa += 32768; pb += 16384; sp = so; sn = so == 2 * STAGE ? 0 : so + STAGE; asm volatile("" : "+s"(pa), "+s"(pb), "+s"(sn)));
            asm volatile("s_waitcnt vmcnt(6) lgkmcnt(0)\n\ts_barrier" ::: "memory");
            so = sn;
            HALF(a1, b1, a0, b0, 0, GLDF(voa, pa, lda + sp, 0), GLDF(voa, pa, lda + sp, 1024), GLDF(voa, pa, lda + sp, 2048));
        }
        if (phase == 0 && !have_all) { wait_operands(ws, pm, pn, call, 15u, lane); STAMP(11 + (set >> 3) * 4); }
    }
    {
        const char* a2 = KA(63); const char* b2 = KB(63);
        GLDS(voa, a2, lda + sp, 3072); GLDS(vob, b2, ldb + sp, 0); GLDS(vob, b2, ldb + sp, 1024);
        FRAGS(a1, b1, so, 1); MMA(a0, b0);
        asm volatile("s_waitcnt vmcnt(6) lgkmcnt(0)\n\ts_barrier" ::: "memory");
        so = so == 2 * STAGE ? 0 : so + STAGE;
        FRAGS(a0, b0, so, 0); MMA(a1, b1);
        FRAGS(a1, b1, so, 1); MMA(a0, b0);
        asm volatile("s_waitcnt vmcnt(0) lgkmcnt(0)\n\ts_barrier" ::: "memory");
        so = so == 2 * STAGE ? 0 : so + STAGE;
        FRAGS(a0, b0, so, 0); MMA(a1, b1);
        FRAGS(a1, b1, so, 1); MMA(a0, b0); MMA(a1, b1);
    }
#undef GLDS
#undef KA
#undef KB
#undef STAGE_LD
#undef FRAGS
#undef MROW
#undef MMA
#undef MM1
#undef GLDF
#undef RDA
#undef RDB
#undef HALF
#undef SB
    wg_barrier();
    STAMP(9 + (set >> 3) * 4);
    const float floor0 = ctl_ld(C_DEAD) != 0 ? __builtin_nanf("") : 0.f;
    char* dst = P.out + ((size_t)(pm * 256 + (wid >> 1) * 64 + (lane & 15)) * NFULL + P.rank * 2048 + pn * 128 + (wid & 1) * 64 + (lane >> 4) * 8) * 2;
#pragma unroll
    for (int b = 0; b < 2; ++b) {
        u32x4 v[4];
#pragma unroll
        for (int j = 0; j < 4; ++j) {
            const f32x4 p = acc[2 * b + (j >> 1)][2 * (j & 1)], q = acc[2 * b + (j >> 1)][2 * (j & 1) + 1];
            float f[8] = {p[0], p[1], p[2], p[3], q[0], q[1], q[2], q[3]};
            u32 r[4];
#pragma unroll
            for (int e = 0; e < 4; ++e) {
                const float lo = __builtin_elementwise_maximum(f[2 * e], floor0), hi = __builtin_elementwise_maximum(f[2 * e + 1], floor0);
                asm("v_cvt_pk_bf16_f32 %0, %1, %2" : "=v"(r[e]) : "v"(lo), "v"(hi));
                r[e] &= 0x7fff7fffu;
            }
            v[j] = (u32x4){r[0], r[1], r[2], r[3]};
            gst_agent(dst + (size_t)(2 * b + (j >> 1)) * (16 * NFULL * 2) + (j & 1) * 64, v[j]);
        }
        enc_output(v, wid, lane);
        const int ob = set * 32 + b * 16 + t16;
        const u32 len = emit_block(ws + OFF_CO + (size_t)ob * SLOT, wid, lane);
        if (threadIdx.x == 0) flag_all(P, NIN + ob, (call << 12) | len);
        JOIN();
    }
    STAMP(10 + (set >> 3) * 4);
}

__global__ void __launch_bounds__(512, 2) fused_gemm_ar(const Params P) {
    const int tid = threadIdx.x, lane = tid & 63, wid = uni(tid >> 6), blk = blockIdx.x;
    char* ws = ws_of(P, P.rank);
    STAMP(0);
    const int xcd = blk & 7, idx = blk >> 3;
    const bool is_dec = blk < NDEC;
    u32x4 nx[4];
    if (!is_dec) pack_load(P, blk - NDEC, wid, lane, nx);
    if (tid < 256) {
        u32 e[2] = {0x0c0c0c0cu, 0x0c0c0c0cu}, c[2] = {0x0c0c0c0cu, 0x0c0c0c0cu};
        u64 ee = 0x0c0c0c0c0c0c0c0cull, cc = ee; int k = 0;
#pragma unroll
        for (int i = 0; i < 8; ++i)
            if ((tid >> i) & 1) { ee ^= (u64)(0x0c ^ k) << (8 * i); cc ^= (u64)(0x0c ^ i) << (8 * k); ++k; }
        e[0] = (u32)ee; e[1] = (u32)(ee >> 32); c[0] = (u32)cc; c[1] = (u32)(cc >> 32);
        st64(L_LUTE + tid * 8, e[0], e[1]); st64(L_LUTC + tid * 8, c[0], c[1]);
    } else if (tid < 256 + 128) {
        u32 v = 0;
        if (tid == 256 + C_CALL) v = __hip_atomic_fetch_add((GAS u32*)(ws + OFF_CTR + blk * 64), 1u, __ATOMIC_RELAXED, __HIP_MEMORY_SCOPE_AGENT) + 1u;
        *ctl(tid - 256) = v;
    }
    wg_barrier();
    const u32 call = ctl_ld(C_CALL);
    if (!is_dec) {
        const int late = idx >= 30 ? 96 + blk - 240 : (xcd >= 2 ? (idx - 14) * 6 + xcd - 2 : NTASK);
        for (int task = blk - NDEC; task < NTASK;) {
            const int next = task < 288 ? task + 144 : (task < 432 ? 432 + late : task + 112);
            u32x4 v[4] = {nx[0], nx[1], nx[2], nx[3]};
            if (next < NTASK) pack_load(P, next, wid, lane, nx);
            pack_task(P, ws, task, call, wid, lane, v);
            task = next;
        }
    }
    STAMP(1);
    if (is_dec) {
        const int j = idx >= 7 ? idx - 7 : idx;
        decoder(P, ws, (P.rank + 1 + j) & 7, (idx >= 7 ? 8 : 0) + xcd, call, wid, lane);
    } else if (idx < 30) {
        for (int rr = 0; rr < 2; ++rr) gemm_tile(P, ws, xcd + 8 * rr, idx - 14, call, wid, lane);
    }
}

}

extern "C" void kernel_launch(int rank, int nranks, ncclComm_t comm, hipStream_t stream,
                              void* const* d_in, const int* in_sizes, int n_in,
                              void* d_out, int out_size, const int* in_dtypes, int out_dtype,
                              void* const* d_out_all, void* const* d_ws_all, size_t ws_bytes) {
    if (nranks != NR || (unsigned)rank >= (unsigned)NR || n_in != 2 || ws_bytes < WS_NEED ||
        in_sizes[0] != 4096 * 512 || in_sizes[1] != 512 * NFULL || out_size != 4096 * NFULL) {
        fprintf(stderr, "kernel_launch: unexpected problem shape\n");
        return;
    }
    Params P;
    P.x = (const char*)d_in[0]; P.w = (const char*)d_in[1]; P.out = (char*)d_out;
    for (int p = 0; p < NR; ++p) P.ws[p] = (char*)d_ws_all[p];
    P.rank = rank; P.pad = 0;
    hipLaunchKernelGGL(fused_gemm_ar, dim3(NWG), dim3(512), 0, stream, P);
    hipError_t e = hipGetLastError();
    if (e != hipSuccess) fprintf(stderr, "launch failed: %s\n", hipGetErrorString(e));
}
```

```cpp
#include <hip/hip_runtime.h>
#include <rccl/rccl.h>
#include <cstdio>

namespace {

typedef unsigned u32;
typedef unsigned long long u64;
typedef u32 u32x4 __attribute__((ext_vector_type(4)));
typedef u32 u32x2 __attribute__((ext_vector_type(2)));
typedef short bf16x8 __attribute__((ext_vector_type(8)));
typedef float f32x4 __attribute__((ext_vector_type(4)));
#define LAS __attribute__((address_space(3)))
#define GAS __attribute__((address_space(1)))

constexpr int NR = 8;
constexpr int NFULL = 16384;
constexpr int NWG = 256;
constexpr int DPP_ = 16;
constexpr int NIN = 192, NOUT = 512;
constexpr int NPOS = NIN + NOUT;
constexpr int NDEC = 7 * DPP_;
constexpr int NTASK = 640;
#define QD 3
#define NRAW 10
#define PUSHQ 0
constexpr u64 DEADLINE = 50000000ull;

constexpr size_t OFF_CTR = 0;
constexpr size_t OFF_TMO = 16384;
constexpr size_t OFF_RDY = 16384 + 256;
constexpr size_t OFF_CLM = 24576;
constexpr size_t OFF_FLR = 28672;
constexpr size_t OFF_FL = 32768;
constexpr size_t OFF_DBG = 65536;
constexpr size_t OFF_XA = 1u << 20;
constexpr size_t OFF_WB = OFF_XA + (32u << 20);
constexpr size_t SLOT = 43008;
constexpr size_t OFF_CX = OFF_WB + (16u << 20);
constexpr size_t OFF_CW = OFF_CX + 128 * SLOT;
constexpr size_t OFF_CO = OFF_CW + 512 * SLOT;
constexpr size_t WS_NEED = OFF_CO + 512 * SLOT;

constexpr int L_LUTE = 0;
constexpr int L_LUTC = 2048;
constexpr int L_CTL = 4096;
constexpr int L_RB = 4608;
constexpr int STAGE = 49152;
constexpr int LDS_BYTES = L_RB + 3 * STAGE;
constexpr int TB = 2112;
constexpr int ENC_W = 5248 + 3 * TB;
constexpr int LSLOT = 41984;
constexpr int L_DT = L_RB + 2 * LSLOT;
static_assert(L_DT + 6 * 3 * TB <= LDS_BYTES, "lds");
static_assert(8 * ENC_W <= 3 * STAGE, "lds");
constexpr int C_HDR = 0, C_SIZE = 8, C_CALL = 16, C_DEAD = 17, C_LANDED = 18, C_CLAIM = 19, C_DONE = 20  ,
              C_NBLK = 22, C_PROG = 23, C_STORED = 24  , C_QUEUE = 40  , C_PUSH = 48;

struct Params {
    const char* x; const char* w; char* out;
    char* ws[NR];
    int rank; int pad;
};

__shared__ __attribute__((aligned(16))) unsigned char smem[LDS_BYTES];

#define CBAR() asm volatile("" ::: "memory")
__device__ __forceinline__ LAS unsigned char* lp(u32 o) { return (LAS unsigned char*)smem + o; }
__device__ __forceinline__ u32 ld8(u32 o) { return *lp(o); }
__device__ __forceinline__ u32 ld32(u32 o) { return *(LAS u32*)lp(o); }
__device__ __forceinline__ u32x2 ld64(u32 o) { return *(LAS u32x2*)lp(o); }
__device__ __forceinline__ u32x4 ld128(u32 o) { return *(LAS u32x4*)lp(o); }
__device__ __forceinline__ void st8(u32 o, u32 v) { *lp(o) = (unsigned char)v; }
__device__ __forceinline__ void st32(u32 o, u32 v) { *(LAS u32*)lp(o) = v; }
__device__ __forceinline__ void st64(u32 o, u32 a, u32 b) { *(LAS u32x2*)lp(o) = (u32x2){a, b}; }
__device__ __forceinline__ void st128(u32 o, u32x4 v) { *(LAS u32x4*)lp(o) = v; }
__device__ __forceinline__ void or32(u32 o, u32 v) { __hip_atomic_fetch_or((LAS u32*)lp(o), v, __ATOMIC_RELAXED, __HIP_MEMORY_SCOPE_WORKGROUP); }
__device__ __forceinline__ volatile LAS u32* ctl(int i) { return (volatile LAS u32*)lp(L_CTL + 4 * i); }
__device__ __forceinline__ u32 uni(u32 v) { return __builtin_amdgcn_readfirstlane(v); }
__device__ __forceinline__ u32 ctl_ld(int i) { return uni(*ctl(i)); }
__device__ __forceinline__ u32 ctl_inc(int i) {
    u32 r; u64 sv;
    asm volatile("s_mov_b64 %1, exec\n\ts_mov_b64 exec, 1\n\tds_add_rtn_u32 %0, %2, %3\n\ts_waitcnt lgkmcnt(0)\n\ts_mov_b64 exec, %1"
                 : "=&v"(r), "=&s"(sv) : "v"((u32)(size_t)lp(L_CTL + 4 * i)), "v"(1u) : "memory");
    return uni(r);
}
#define JOIN() __builtin_amdgcn_wave_barrier()
__device__ __forceinline__ u32 perm(u32 hi, u32 lo, u32 sel) { return __builtin_amdgcn_perm(hi, lo, sel); }
__device__ __forceinline__ u32 bfi(u32 m, u32 a, u32 b) { return (m & a) | (~m & b); }
__device__ __forceinline__ u64 now() { return __builtin_amdgcn_s_memrealtime(); }
__device__ __forceinline__ void wg_barrier() { asm volatile("s_waitcnt lgkmcnt(0)\n\ts_barrier" ::: "memory"); }
__device__ __forceinline__ void vm_drain() { asm volatile("s_waitcnt vmcnt(0)" ::: "memory"); }

__device__ __forceinline__ void gst_agent(void* p, u32x4 v) { asm volatile("global_store_dwordx4 %0, %1, off sc1\n\ts_nop 1" ::"v"(p), "v"(v) : "memory"); }
__device__ __forceinline__ void gst_sys(void* p, u32x4 v) { asm volatile("global_store_dwordx4 %0, %1, off sc0 sc1\n\ts_nop 1" ::"v"(p), "v"(v) : "memory"); }

__device__ __forceinline__ u32 scan(u32 x) {
    x += __builtin_amdgcn_update_dpp(0u, x, 0x111, 0xf, 0xf, false);
    x += __builtin_amdgcn_update_dpp(0u, x, 0x112, 0xf, 0xf, false);
    x += __builtin_amdgcn_update_dpp(0u, x, 0x114, 0xf, 0xf, false);
    x += __builtin_amdgcn_update_dpp(0u, x, 0x118, 0xf, 0xf, false);
    x += __builtin_amdgcn_update_dpp(0u, x, 0x142, 0xa, 0xf, false);
    x += __builtin_amdgcn_update_dpp(0u, x, 0x143, 0xc, 0xf, false);
    return x;
}
__device__ __forceinline__ u32 last(u32 x) { return __builtin_amdgcn_readlane(x, 63); }
__device__ __forceinline__ u32 offsets4(u32 m, u32 (&o)[4]) {
    const u32 c01 = __popc(m & 0xffu) | (__popc(m & 0xff00u) << 16), c23 = __popc(m & 0xff0000u) | (__popc(m >> 24) << 16);
    const u32 s01 = scan(c01), s23 = scan(c23);
    const u32 t01 = last(s01), t23 = last(s23);
    const u32 e01 = s01 - c01, e23 = s23 - c23;
    const u32 b1 = t01 & 0xffffu, b2 = b1 + (t01 >> 16), b3 = b2 + (t23 & 0xffffu);
    o[0] = e01 & 0xffffu; o[1] = (e01 >> 16) + b1; o[2] = (e23 & 0xffffu) + b2; o[3] = (e23 >> 16) + b3;
    return b3 + (t23 >> 16);
}
__device__ __forceinline__ u32 gather8(u32 y0, u32 y1) {
    return __builtin_amdgcn_udot4(y1, 0x80402010u, __builtin_amdgcn_udot4(y0, 0x08040201u, 0u, false), false);
}
__device__ __forceinline__ u32 spread1(u32 n4) { return __umul24(n4, 0x204081u) & 0x01010101u; }
__device__ __forceinline__ u32 spread2(u32 n4) { return __umul24(n4, 0x408102u) & 0x02020202u; }
__device__ __forceinline__ u32 subb(u32 a, u32 brep) {
    return ((a | 0x80808080u) - (brep & 0x7f7f7f7fu)) ^ ((a ^ ~brep) & 0x80808080u);
}
template <int K> __device__ __forceinline__ u32 geb(u32 d) {
    return ((((d & 0x7f7f7f7fu) + (u32)(0x80 - K) * 0x01010101u) | d) >> 7) & 0x01010101u;
}
__device__ __forceinline__ void fetch8(u32 a, u32& s0, u32& s1) {
    const u32 q = a & ~3u;
    const u32 d0 = ld32(q), d1 = ld32(q + 4), d2 = ld32(q + 8);
    s0 = __builtin_amdgcn_alignbyte(d1, d0, a); s1 = __builtin_amdgcn_alignbyte(d2, d1, a);
}
__device__ __forceinline__ void scatter8(u32 a, u32 c0, u32 c1) {
    const u32 q = a & ~3u, s = (a & 3u) * 8u;
    const u64 lo = (((u64)c1 << 32) | c0) << s;
    const u32 w0 = (u32)lo, w1 = (u32)(lo >> 32), w2 = (c1 >> 1) >> (31u - s);
    if (w0) or32(q, w0);
    if (w1) or32(q + 4, w1);
    if (w2) or32(q + 8, w2);
}
__device__ __forceinline__ void zero_lds(u32 a, u32 bytes, int lane) {
    for (u32 i = lane * 16; i < bytes; i += 1024) st128(a + i, (u32x4){0u, 0u, 0u, 0u});
}

__device__ __forceinline__ void dec_l2(u32 l2a, u32 n2, u32 base, bool wide, u32 t2, int lane) {
    const u32 pb = (n2 + 7) >> 3, l2b = l2a + pb, l3 = l2b + pb;
    const u32 tab = (base - 2) | ((base - 1) << 8) | ((base + 3) << 16);
    const u32 top = (base + 5) * 0x01010101u;
    u32 run = 0;
    for (u32 p = 0; p < pb; p += 64) {
        const u32 idx = p + lane;
        const u32 a8 = ld8(l2a + idx), b8 = ld8(l2b + idx), more = a8 & b8;
        const u32 cnt = __popc(more), inc = scan(cnt), o3 = run + inc - cnt;
        run += last(inc);
        const u32 c0 = spread1(a8 & 15u) | spread2(b8 & 15u), c1 = spread1(a8 >> 4) | spread2(b8 >> 4);
        u32 s0, s1;
        if (wide) fetch8(l3 + o3, s0, s1);
        else {
            const u32 bit = l3 * 8u + o3 * 4u, q = (bit >> 5) << 2;
            const u32 xx = __builtin_amdgcn_alignbit(ld32(q + 4), ld32(q), bit);
            const u32 lo = xx & 0x0f0f0f0fu, hi = (xx >> 4) & 0x0f0f0f0fu;
            s0 = perm(hi, lo, 0x05010400u); s1 = perm(hi, lo, 0x07030602u);
            const u32 g0 = ((s0 + 0x0e0e0e0eu) >> 4) & 0x01010101u, g1 = ((s1 + 0x0e0e0e0eu) >> 4) & 0x01010101u;
            s0 = top - s0 - g0 * 6u; s1 = top - s1 - g1 * 6u;
        }
        const u32x2 sel = ld64(L_LUTE + more * 8u);
        st64(t2 + idx * 8u, perm(0u, tab, c0) | perm(s1, s0, sel.x), perm(0u, tab, c1) | perm(s1, s0, sel.y));
    }
    CBAR();
}

__device__ __forceinline__ void dec_input(u32 rec, u32 hdr, bool wide, u32 tmp, char* dst, int lane) {
    const u32 n2 = min((hdr >> 12) & 0xfffu, 2048u), base = hdr >> 24, t2 = tmp + 2 * TB;
    dec_l2(rec + 2560, n2, base, wide, t2, lane);
    const u32 pa = ld32(rec + 2048 + lane * 4), pb = ld32(rec + 2304 + lane * 4), more = pa & pb;
    const u32 tab = base | ((base + 1) << 8) | ((base + 2) << 16);
    u32 o[4]; offsets4(more, o);
#pragma unroll
    for (int j = 0; j < 4; ++j) {
        const u32x2 mb = ld64(rec + j * 512 + lane * 8);
        const u32 c0 = spread1((pa >> (8 * j)) & 15u) | spread2((pb >> (8 * j)) & 15u);
        const u32 c1 = spread1((pa >> (8 * j + 4)) & 15u) | spread2((pb >> (8 * j + 4)) & 15u);
        u32 s0, s1; fetch8(t2 + o[j], s0, s1);
        const u32x2 sel = ld64(L_LUTE + ((more >> (8 * j)) & 255u) * 8u);
        const u32 e0 = perm(0u, tab, c0) | perm(s1, s0, sel.x), e1 = perm(0u, tab, c1) | perm(s1, s0, sel.y);
        const u32 h0 = bfi(0x80808080u, mb.x, e0 >> 1), h1 = bfi(0x80808080u, mb.y, e1 >> 1);
        const u32 l0 = bfi(0x80808080u, e0 << 7, mb.x), l1 = bfi(0x80808080u, e1 << 7, mb.y);
        gst_agent(dst + j * 1024, (u32x4){perm(h0, l0, 0x05010400u), perm(h0, l0, 0x07030602u), perm(h1, l1, 0x05010400u), perm(h1, l1, 0x07030602u)});
    }
}

__device__ __forceinline__ void dec_output(u32 rec, u32 hdr, bool wide, u32 tmp, char* dst, int lane) {
    const u32 n1 = min(hdr & 0xfffu, 2048u), n2 = min((hdr >> 12) & 0xfffu, 2048u), base = hdr >> 24;
    const u32 t1l = tmp, t1h = tmp + TB, t2 = tmp + 2 * TB;
    const u32 pbo = rec + 256 + n1, nb = (n1 + 7) >> 3;
    dec_l2(pbo + nb, n2, base, wide, t2, lane);
    const u32 tab = base | ((base + 1) << 8) | ((base + 2) << 16);
    u32 run = 0;
    for (u32 p = 0; p < nb; p += 64) {
        const u32 idx = p + lane;
        const u32x2 bb = ld64(rec + 256 + idx * 8u);
        const u32 b8 = ld8(pbo + idx);
        const u32 a0 = (bb.x >> 7) & 0x01010101u, a1 = (bb.y >> 7) & 0x01010101u;
        const u32 more = gather8(a0, a1) & b8;
        const u32 cnt = __popc(more), inc = scan(cnt), o2 = run + inc - cnt;
        run += last(inc);
        const u32 c0 = a0 | spread2(b8 & 15u), c1 = a1 | spread2(b8 >> 4);
        u32 s0, s1; fetch8(t2 + o2, s0, s1);
        const u32x2 sel = ld64(L_LUTE + more * 8u);
        const u32 e0 = perm(0u, tab, c0) | perm(s1, s0, sel.x), e1 = perm(0u, tab, c1) | perm(s1, s0, sel.y);
        st64(t1l + idx * 8u, bfi(0x80808080u, e0 << 7, bb.x), bfi(0x80808080u, e1 << 7, bb.y));
        st64(t1h + idx * 8u, (e0 >> 1) & 0x7f7f7f7fu, (e1 >> 1) & 0x7f7f7f7fu);
    }
    CBAR();
    const u32 m = ld32(rec + lane * 4);
    u32 o[4]; offsets4(m, o);
#pragma unroll
    for (int j = 0; j < 4; ++j) {
        u32 l0, l1, h0, h1;
        fetch8(t1l + o[j], l0, l1); fetch8(t1h + o[j], h0, h1);
        const u32x2 sel = ld64(L_LUTE + ((m >> (8 * j)) & 255u) * 8u);
        const u32 L0 = perm(l1, l0, sel.x), L1 = perm(l1, l0, sel.y), H0 = perm(h1, h0, sel.x), H1 = perm(h1, h0, sel.y);
        gst_agent(dst + (size_t)(j >> 1) * (16 * NFULL * 2) + (j & 1) * 64,
                  (u32x4){perm(H0, L0, 0x05010400u), perm(H0, L0, 0x07030602u), perm(H1, L1, 0x05010400u), perm(H1, L1, 0x07030602u)});
    }
}

__device__ __forceinline__ u32 enc_l2(u32 l2a, u32 n2, u32 base, u32 t2, bool& wide, int lane) {
    const u32 pb = (n2 + 7) >> 3, l2b = l2a + pb, l3 = l2b + pb;
    const u32 top = (base + 5) * 0x01010101u;
    u32 run;
    wide = false;
    for (;;) {
        if (lane < 16) st8(l3 + lane, 0u);
        zero_lds((l3 + 15u) & ~15u, (wide ? n2 : n2 >> 1) + 8u, lane);
        CBAR();
        u32 out = 0;
        run = 0;
        for (u32 p = 0; p < pb; p += 64) {
            const u32 idx = p + lane;
            const u32x2 e = ld64(t2 + idx * 8u);
            const int left = (int)n2 - (int)(idx * 8u);
            const u32 valid = left >= 8 ? 255u : (left > 0 ? (1u << left) - 1u : 0u);
            const u32 h0 = subb(top, e.x), h1 = subb(top, e.y);
            const u32 n0 = ~geb<8>(h0) & (h0 >> 1), n1 = ~geb<8>(h1) & (h1 >> 1);
            const u32 more = ~gather8(n0 & 0x01010101u, n1 & 0x01010101u) & valid;
            const u32 a8 = gather8((h0 >> 2) & ~h0 & 0x01010101u, (h1 >> 2) & ~h1 & 0x01010101u) | more;
            const u32 b8 = ~gather8((h0 >> 2) & 0x01010101u, (h1 >> 2) & 0x01010101u) | more;
            if (left > 0) { st8(l2a + idx, a8 & valid); st8(l2b + idx, b8 & valid); }
            const u32 cnt = __popc(more), inc = scan(cnt), o3 = run + inc - cnt;
            run += last(inc);
            const u32x2 sel = ld64(L_LUTC + more * 8u);
            if (wide) scatter8(l3 + o3, perm(e.y, e.x, sel.x), perm(e.y, e.x, sel.y));
            else {
                u32 c0 = perm(h1, h0, sel.x), c1 = perm(h1, h0, sel.y);
                out |= geb<22>(c0) | geb<22>(c1);
                c0 -= (((c0 + 0x08080808u) >> 4) & 0x01010101u) * 6u; c1 -= (((c1 + 0x08080808u) >> 4) & 0x01010101u) * 6u;
                const u32 xx = perm(c1 | (c1 >> 4), c0 | (c0 >> 4), 0x06040200u);
                const u32 bit = l3 * 8u + o3 * 4u, q = (bit >> 5) << 2;
                const u64 w = (u64)xx << (bit & 31u);
                if ((u32)w) or32(q, (u32)w);
                if ((u32)(w >> 32)) or32(q + 4, (u32)(w >> 32));
            }
        }
        CBAR();
        if (wide || !__any(out != 0)) break;
        wide = true;
    }
    return l3 + (wide ? run : (run + 1) >> 1);
}

__device__ __forceinline__ void enc_input(const u32x4 (&v)[4], int wid, int lane) {
    const u32 rec = L_RB + wid * ENC_W, t2 = rec + 5248 + 2 * TB;
    u32 e[4][2], sum = 0;
#pragma unroll
    for (int j = 0; j < 4; ++j) {
        const u32 l0 = perm(v[j].y, v[j].x, 0x06040200u), h0 = perm(v[j].y, v[j].x, 0x07050301u);
        const u32 l1 = perm(v[j].w, v[j].z, 0x06040200u), h1 = perm(v[j].w, v[j].z, 0x07050301u);
        e[j][0] = bfi(0x01010101u, l0 >> 7, h0 << 1); e[j][1] = bfi(0x01010101u, l1 >> 7, h1 << 1);
        st64(rec + j * 512 + lane * 8, bfi(0x80808080u, h0, l0), bfi(0x80808080u, h1, l1));
        sum = __builtin_amdgcn_udot4(e[j][0], 0x01010101u, __builtin_amdgcn_udot4(e[j][1], 0x01010101u, sum, false), false);
    }
    const int tot = (int)last(scan(sum));
    const u32 base = (u32)min(max((tot - 369) >> 11, 16), 250);
    const u32 brep = base * 0x01010101u;
    u32 pa = 0, pb = 0, more = 0;
#pragma unroll
    for (int j = 0; j < 4; ++j) {
        const u32 d0 = subb(e[j][0], brep), d1 = subb(e[j][1], brep);
        more |= gather8(geb<3>(d0), geb<3>(d1)) << (8 * j);
        pa |= gather8(d0 & 0x01010101u, d1 & 0x01010101u) << (8 * j);
        pb |= gather8((d0 >> 1) & 0x01010101u, (d1 >> 1) & 0x01010101u) << (8 * j);
    }
    st32(rec + 2048 + lane * 4, pa | more); st32(rec + 2304 + lane * 4, pb | more);
    u32 o[4]; const u32 n2 = offsets4(more, o);
    zero_lds(t2, n2, lane);
    CBAR();
#pragma unroll
    for (int j = 0; j < 4; ++j) {
        const u32x2 sel = ld64(L_LUTC + ((more >> (8 * j)) & 255u) * 8u);
        scatter8(t2 + o[j], perm(e[j][1], e[j][0], sel.x), perm(e[j][1], e[j][0], sel.y));
    }
    CBAR();
    bool wide; const u32 end = enc_l2(rec + 2560, n2, base, t2, wide, lane);
    if (lane == 0) { *ctl(C_HDR + wid) = (n2 << 12) | (base << 24); *ctl(C_SIZE + wid) = ((end - rec + 15) >> 4) | (wide ? 0x8000u : 0u); }
}

__device__ __forceinline__ void enc_output(const u32x4 (&v)[4], int wid, int lane) {
    const u32 rec = L_RB + wid * ENC_W, t1l = rec + 5248, t1h = t1l + TB, t2 = t1h + TB;
    u32 l[4][2], h[4][2], m = 0, sum = 0;
#pragma unroll
    for (int j = 0; j < 4; ++j) {
        l[j][0] = perm(v[j].y, v[j].x, 0x06040200u); h[j][0] = perm(v[j].y, v[j].x, 0x07050301u);
        l[j][1] = perm(v[j].w, v[j].z, 0x06040200u); h[j][1] = perm(v[j].w, v[j].z, 0x07050301u);
        m |= gather8(geb<1>(l[j][0] | h[j][0]), geb<1>(l[j][1] | h[j][1])) << (8 * j);
        sum = __builtin_amdgcn_udot4(bfi(0x01010101u, l[j][0] >> 7, h[j][0] << 1), 0x01010101u, sum, false);
        sum = __builtin_amdgcn_udot4(bfi(0x01010101u, l[j][1] >> 7, h[j][1] << 1), 0x01010101u, sum, false);
    }
    st32(rec + lane * 4, m);
    u32 o[4]; const u32 n1 = offsets4(m, o);
    for (u32 i = lane * 16; i < n1; i += 1024) { const u32x4 z = {0u, 0u, 0u, 0u}; st128(t1l + i, z); st128(t1h + i, z); st128(t2 + i, z); }
    const u32 tot = last(scan(sum));
    const u32 base = uni((u32)min(max((int)((float)tot / (float)max(n1, 1u) - 0.18f), 16), 250));
    const u32 brep = base * 0x01010101u;
    CBAR();
#pragma unroll
    for (int j = 0; j < 4; ++j) {
        const u32x2 sel = ld64(L_LUTC + ((m >> (8 * j)) & 255u) * 8u);
        scatter8(t1l + o[j], perm(l[j][1], l[j][0], sel.x), perm(l[j][1], l[j][0], sel.y));
        scatter8(t1h + o[j], perm(h[j][1], h[j][0], sel.x), perm(h[j][1], h[j][0], sel.y));
    }
    CBAR();
    const u32 nb = (n1 + 7) >> 3, pbo = rec + 256 + n1;
    u32 run = 0, pbs = 0;
#pragma unroll
    for (int p = 0; p < 4; ++p) {
        if ((u32)p * 64u < nb) {
            const u32 idx = p * 64 + lane;
            const u32x2 ll = ld64(t1l + idx * 8u), hh = ld64(t1h + idx * 8u);
            const int left = (int)n1 - (int)(idx * 8u);
            const u32 valid = left >= 8 ? 255u : (left > 0 ? (1u << left) - 1u : 0u);
            const u32 e0 = bfi(0x01010101u, ll.x >> 7, hh.x << 1), e1 = bfi(0x01010101u, ll.y >> 7, hh.y << 1);
            const u32 d0 = subb(e0, brep), d1 = subb(e1, brep);
            const u32 g0 = geb<3>(d0), g1 = geb<3>(d1);
            const u32 more = gather8(g0, g1) & valid;
            if (left > 0) st64(rec + 256 + idx * 8u, bfi(0x80808080u, (d0 | g0) << 7, ll.x), bfi(0x80808080u, (d1 | g1) << 7, ll.y));
            pbs |= ((gather8((d0 >> 1) & 0x01010101u, (d1 >> 1) & 0x01010101u) | more) & valid) << (8 * p);
            const u32 cnt = __popc(more), inc = scan(cnt), o2 = run + inc - cnt;
            run += last(inc);
            const u32x2 sel = ld64(L_LUTC + more * 8u);
            scatter8(t2 + o2, perm(e1, e0, sel.x), perm(e1, e0, sel.y));
        }
    }
    CBAR();
#pragma unroll
    for (int p = 0; p < 4; ++p)
        if ((u32)(p * 64 + lane) < nb) st8(pbo + p * 64 + lane, pbs >> (8 * p));
    CBAR();
    bool wide; const u32 end = enc_l2(pbo + nb, run, base, t2, wide, lane);
    if (lane == 0) { *ctl(C_HDR + wid) = n1 | (run << 12) | (base << 24); *ctl(C_SIZE + wid) = ((end - rec + 15) >> 4) | (wide ? 0x8000u : 0u); }
}

__device__ __forceinline__ u32 emit_block(char* slot, int wid, int lane) {
    const u32 rec = L_RB + wid * ENC_W + lane * 16;
    CBAR();
    const u32x4 r0 = ld128(rec), r1 = ld128(rec + 1024), r2 = ld128(rec + 2048);
    wg_barrier();
    const u32 sz = *ctl(C_SIZE + (lane & 7));
    u32 inc = sz & 0x7fffu;
    inc += __builtin_amdgcn_update_dpp(0u, inc, 0x111, 0xf, 0xf, false);
    inc += __builtin_amdgcn_update_dpp(0u, inc, 0x112, 0xf, 0xf, false);
    inc += __builtin_amdgcn_update_dpp(0u, inc, 0x114, 0xf, 0xf, false);
    inc += 3u;
    const u32 mysz = __builtin_amdgcn_readlane(sz, wid) & 0x7fffu, myend = __builtin_amdgcn_readlane(inc, wid);
    const u32 total = __builtin_amdgcn_readlane(inc, 7);
    char* d = slot + (size_t)(myend - mysz + lane) * 16;
    if ((u32)lane < mysz) gst_sys(d, r0);
    if ((u32)lane + 64u < mysz) gst_sys(d + 1024, r1);
    if ((u32)lane + 128u < mysz) gst_sys(d + 2048, r2);
    for (u32 i = 192; i + lane < mysz; i += 64) gst_sys(d + (size_t)i * 16, ld128(rec + i * 16));
    if (wid == 0) {
        const u32 en = inc | (sz & 0x8000u);
        u32x4 hv;
        hv.x = __builtin_amdgcn_readlane(en, 0) | (__builtin_amdgcn_readlane(en, 1) << 16);
        hv.y = __builtin_amdgcn_readlane(en, 2) | (__builtin_amdgcn_readlane(en, 3) << 16);
        hv.z = __builtin_amdgcn_readlane(en, 4) | (__builtin_amdgcn_readlane(en, 5) << 16);
        hv.w = __builtin_amdgcn_readlane(en, 6) | (__builtin_amdgcn_readlane(en, 7) << 16);
        if (lane < 2) hv = ld128(L_CTL + 4 * C_HDR + lane * 16);
        if (lane < 3) gst_sys(slot + lane * 16, hv);
    }
    vm_drain();
    wg_barrier();
    return total;
}

__device__ __forceinline__ void flag_store(char* wsl, int rank, int pos, u32 val) {
    __hip_atomic_store((GAS u32*)(wsl + OFF_FL) + (rank * 1024 + pos), val, __ATOMIC_RELAXED, __HIP_MEMORY_SCOPE_SYSTEM);
}
__device__ __forceinline__ void ready_add(char* ws, int idx, int local) {
    __hip_atomic_fetch_add((GAS u32*)(ws + OFF_RDY + idx * 128 + local * 64), 1u, __ATOMIC_RELAXED, __HIP_MEMORY_SCOPE_AGENT);
}
#define STAMP(i) do {} while (0)
#define STAMPW(i) do {} while (0)

__device__ __forceinline__ bool pos_is_x(int pos) { return pos < 16 || (pos >= 48 && pos < 144) || pos >= 176; }
__device__ __forceinline__ int pos_xb(int pos) { return pos < 16 ? pos : (pos < 144 ? pos - 32 : pos - 64); }
__device__ __forceinline__ int pos_wb(int pos) { return pos < 48 ? pos - 16 : pos - 112; }
__device__ __forceinline__ void task_of(int task, int& pos, int& rel) {
    rel = -1;
    if (task < 16) pos = task;
    else if (task < 272) { pos = 16 + ((task - 16) >> 3); rel = (task + 1) & 7; }
    else if (task < 368) pos = 48 + task - 272;
    else if (task < 624) { pos = 144 + ((task - 368) >> 3); rel = (task + 1) & 7; }
    else pos = 176 + task - 624;
}
__device__ __forceinline__ void pack_load(const Params& P, int task, int wid, int lane, u32x4 (&t)[4]) {
    int pos, rel; task_of(task, pos, rel);
    const char* s; size_t s1, s2;
    if (rel < 0) {
        const int xb = pos_xb(pos);
        s = P.x + (size_t)((xb >> 3) * 256 + wid * 32 + (lane & 15)) * 1024 + (xb & 7) * 128 + (lane >> 4) * 16;
        s1 = 64; s2 = 16384;
    } else {
        const int wb = pos_wb(pos);
        s = P.w + (size_t)((wb & 3) * 128 + (wid >> 2) * 64 + (lane >> 2)) * (NFULL * 2) + (((P.rank + rel) & 7) * 2048 + (wb >> 2) * 128 + (wid & 3) * 32) * 2 + (lane & 3) * 16;
        s1 = 16 * NFULL * 2; s2 = 2 * s1;
    }
    t[0] = *(const GAS u32x4*)s; t[1] = *(const GAS u32x4*)(s + s1); t[2] = *(const GAS u32x4*)(s + s2); t[3] = *(const GAS u32x4*)(s + s1 + s2);
}
__device__ __forceinline__ void pack_task(const Params& P, char* ws, char* wsl, int task, u32 call, int wid, int lane, u32x4 (&v)[4]) {
    int pos, rel; task_of(task, pos, rel);
    size_t lo, so; int rdy;
    if (rel < 0) {
        const int xb = pos_xb(pos);
        rdy = xb >> 3; lo = OFF_XA + (size_t)(rdy * 64 + (xb & 7)) * 32768; so = OFF_CX + (size_t)xb * SLOT;
    } else {
        const int wb = pos_wb(pos);
        rdy = 16 + (wb >> 2); lo = OFF_WB + (size_t)((wb >> 2) * 64 + (wb & 3) * 2) * 16384; so = OFF_CW + (size_t)(rel * 64 + wb) * SLOT;
        const u32 tr = L_RB + wid * ENC_W + 5248;
#pragma unroll
        for (int i = 0; i < 4; ++i) st128(tr + i * 1024 + lane * 16, v[i]);
        CBAR();
        const u32 ra = tr + (lane >> 4) * 512 + (((lane >> 2) & 3) * 8 + (lane & 3)) * 2;
#pragma unroll
        for (int j = 0; j < 4; ++j) {
            u32 q[8];
#pragma unroll
            for (int e = 0; e < 8; ++e) q[e] = *(LAS unsigned short*)lp(ra + (j & 1) * 2048 + e * 64 + (j >> 1) * 8);
            v[j] = (u32x4){q[0] | (q[1] << 16), q[2] | (q[3] << 16), q[4] | (q[5] << 16), q[6] | (q[7] << 16)};
        }
        CBAR();
    }
    const bool local = rel <= 0, packed = rel != 0 && pos >= NRAW;
    if (local) {
        char* d = ws + lo + wid * 4096 + lane * 16;
#pragma unroll
        for (int j = 0; j < 4; ++j) gst_agent(d + j * 1024, v[j]);
    }
    u32 len = 0;
    if (packed) { enc_input(v, wid, lane); len = emit_block(ws + so, wid, lane); }
    else { vm_drain(); wg_barrier(); }
    if (wid == 0) {
        if (packed && (rel < 0 ? lane >= 1 && lane < 8 : lane == rel)) flag_store(wsl, P.rank, pos, (call << 12) | len);
        if (local && lane == 0) ready_add(ws, rdy, 1);
    }
    JOIN();
}

__device__ __forceinline__ void tile_of(int set, int t16, int& pm, int& pn) {
    const bool first = set == 0 || set == 7;
    const int pair = set < 7 ? set : (set < 14 ? set - 7 : 7), half = set < 7 ? 0 : (set < 14 ? 1 : set & 1);
    pm = 2 * pair + (first ? t16 & 1 : t16 >> 3); pn = 8 * half + (first ? t16 >> 1 : t16 & 7);
}

__device__ __forceinline__ void decoder(const Params& P, char* ws, char* sws, int slab, int d, u32 call, int wid, int lane, const u32x4 (&raw)[4]) {
    const int src = (P.rank + slab) & 7;
    const u32 seq = call & 0xfffffu;
    const u64 tstart = now();
    GAS u32* tmo = (GAS u32*)(ws + OFF_TMO);
    if (wid == 1) {
        const GAS u32* fl = (const GAS u32*)(ws + OFF_FL) + src * 1024;
        bool dead = false;
        for (u32 i = 0;; ++i) {
            u32 pos = NRAW + d;
            if (i) {
                while (ctl_ld(C_PROG) < i) { if (now() - tstart > 2 * DEADLINE) break; __builtin_amdgcn_s_sleep(2); }
                u32 t = 0;
                if (lane == 0) t = __hip_atomic_fetch_add((GAS u32*)(ws + OFF_CLM + src * 128), 1u, __ATOMIC_RELAXED, __HIP_MEMORY_SCOPE_AGENT);
                pos = uni(t) - (call - 1u) * (u32)(NPOS - NRAW) + (u32)(NRAW + DPP_);
            }
            if (pos >= (u32)NPOS) { *ctl(C_QUEUE + (i & 7u)) = ((i + 1u) << 22) | (1023u << 12); break; }
            u32 f = 0;
            while (!dead) {
                f = uni(__hip_atomic_load(fl + pos, __ATOMIC_RELAXED, __HIP_MEMORY_SCOPE_SYSTEM));
                if ((f >> 12) == seq) break;
                if (uni(__hip_atomic_load(tmo, __ATOMIC_RELAXED, __HIP_MEMORY_SCOPE_AGENT)) == call || now() - tstart > DEADLINE) {
                    __hip_atomic_store(tmo, call, __ATOMIC_RELAXED, __HIP_MEMORY_SCOPE_AGENT);
                    *ctl(C_DEAD) = 1; dead = true; f = 0;
                }
                __builtin_amdgcn_s_sleep(1);
            }
            CBAR();
            *ctl(C_QUEUE + (i & 7u)) = ((i + 1u) << 22) | (pos << 12) | (f & 0xfffu);
        }
    } else if (wid == 0) {
        u32 tg[QD - 1] = {}, cur = 0;
#define DRAIN_PUBLISH() do { _Pragma("unroll") for (int q_ = 0; q_ < QD - 2; ++q_) { \
            if (q_ == 0) asm volatile("s_waitcnt vmcnt(%0)" ::"n"(QD - 2) : "memory"); else if (q_ == 1) asm volatile("s_waitcnt vmcnt(%0)" ::"n"(QD > 3 ? QD - 3 : 0) : "memory"); \
            else if (q_ == 2) asm volatile("s_waitcnt vmcnt(%0)" ::"n"(QD > 4 ? QD - 4 : 0) : "memory"); else asm volatile("s_waitcnt vmcnt(%0)" ::"n"(QD > 5 ? QD - 5 : 0) : "memory"); \
            if (q_ < 4 && tg[q_] > cur) *ctl(C_LANDED) = tg[q_]; } vm_drain(); } while (0)
        u32 i = 0;
        for (;; ++i) {
            const u32 need = 8u * (i >> 1);
            u32 f = ctl_ld(C_QUEUE + (i & 7u));
            if ((f >> 22) != i + 1u || ctl_ld(C_DONE + (i & 1u)) < need) {
                DRAIN_PUBLISH();
                cur = i << 6; *ctl(C_LANDED) = cur;
                while (((f = ctl_ld(C_QUEUE + (i & 7u))) >> 22) != i + 1u || ctl_ld(C_DONE + (i & 1u)) < need) {
                    if (now() - tstart > 2 * DEADLINE) { f = 1023u << 12; break; }
                    __builtin_amdgcn_s_sleep(1);
                }
            }
            CBAR();
            const int pos = (f >> 12) & 1023u;
            if (pos >= NPOS) break;
            *ctl(C_PROG) = i + 1u;
            size_t so;
            if (pos < NIN) so = pos_is_x(pos) ? OFF_CX + (size_t)pos_xb(pos) * SLOT : OFF_CW + (size_t)((8 - slab) * 64 + pos_wb(pos)) * SLOT;
            else so = OFF_CO + (size_t)(pos - NIN) * SLOT;
            const u32 len = min(f & 0xfffu, (u32)(LSLOT / 16));
            const char* s = sws + so + lane * 16;
            const u32 lb = (u32)(size_t)lp(L_RB + (i & 1u) * LSLOT);
            for (u32 k = 0; k * 64u < len; ++k) {
                if (k * 64u + lane < len) {
                    u32 keep;
                    asm volatile("s_mov_b32 %0, m0\n\ts_mov_b32 m0, %2\n\ts_nop 0\n\tglobal_load_lds_dwordx4 %1, off sc0 sc1\n\ts_mov_b32 m0, %0"
                                 : "=&s"(keep) : "v"(s + (size_t)k * 1024), "s"(lb + k * 1024u) : "memory");
                }
                asm volatile("s_waitcnt vmcnt(%0)" ::"n"(QD - 1) : "memory");
                if (tg[0] > cur) { cur = tg[0]; *ctl(C_LANDED) = cur; }
#pragma unroll
                for (int q = 0; q < QD - 2; ++q) tg[q] = tg[q + 1];
                tg[QD - 2] = (i << 6) + k + 1;
            }
        }
        DRAIN_PUBLISH();
        *ctl(C_NBLK) = i + 1u;
        *ctl(C_LANDED) = 0xffffffffu;
        STAMPW(50);
    } else {
        const u32 tmp = L_DT + (wid - 2) * 3 * TB;
        if (wid - 2 < NRAW / 2) {
            const int g = d * (NRAW / 2) + wid - 2;
            char* dst = sws + OFF_XA + (size_t)((g >> 6) * 64 + (8 - slab) * 8 + ((g >> 3) & 7)) * 32768 + (g & 7) * 4096 + lane * 16;
#pragma unroll
            for (int j = 0; j < 4; ++j) { gst_sys(dst + j * 1024, raw[j]); asm volatile("s_waitcnt vmcnt(%0)" ::"n"(PUSHQ) : "memory"); }
            vm_drain();
            if (ctl_inc(C_PUSH) == NRAW / 2 - 1) {
                if (lane == 0) __hip_atomic_store((GAS u32*)(sws + OFF_FLR) + (P.rank * 16 + d), call, __ATOMIC_RELAXED, __HIP_MEMORY_SCOPE_SYSTEM);
                JOIN();
                STAMPW(52);
            }
        }
        for (;;) {
            const u32 r = ctl_inc(C_CLAIM);
            const u32 i = r >> 3, sub = r & 7u;
            const u32 slot = L_RB + (i & 1u) * LSLOT;
            bool late = false;
            while (ctl_ld(C_LANDED) < (i << 6) + 1u) { if (now() - tstart > 2 * DEADLINE) { late = true; break; } __builtin_amdgcn_s_sleep(1); }
            CBAR();
            if (late || ctl_ld(C_NBLK) - 1u <= i) break;
            const int pos = min((ctl_ld(C_QUEUE + (i & 7u)) >> 12) & 1023u, (u32)(NPOS - 1));
            const u32 hdr = uni(ld32(slot + sub * 4));
            const u32 en = uni(*(LAS unsigned short*)lp(slot + 32 + sub * 2));
            const u32 st = sub ? (uni(*(LAS unsigned short*)lp(slot + 30 + sub * 2)) & 0x7fffu) : 3u;
            const u32 endp = min(((en & 0x7fffu) + 63u) >> 6, 63u);
            while (ctl_ld(C_LANDED) < (i << 6) + endp) { if (now() - tstart > 2 * DEADLINE) break; __builtin_amdgcn_s_sleep(1); }
            CBAR();
            const bool dead = ctl_ld(C_DEAD) != 0;
            const u32 rec = slot + min(st, (u32)(LSLOT / 16)) * 16u;
            if (pos < NIN) {
                size_t o; int rdy;
                if (pos_is_x(pos)) { const int xb = pos_xb(pos); rdy = xb >> 3; o = OFF_XA + (size_t)(rdy * 64 + slab * 8 + (xb & 7)) * 32768; }
                else { const int wb = pos_wb(pos); rdy = 16 + (wb >> 2); o = OFF_WB + (size_t)((wb >> 2) * 64 + slab * 8 + (wb & 3) * 2) * 16384; }
                if (!dead) dec_input(rec, hdr, (en >> 15) != 0, tmp, ws + o + sub * 4096 + lane * 16, lane);
                asm volatile("s_waitcnt lgkmcnt(0)" ::: "memory");
                ctl_inc(C_DONE + (i & 1u));
                vm_drain();
                if ((ctl_inc(C_STORED + (i & 15u)) & 7u) == 7u) { if (lane == 0) ready_add(ws, rdy, 0); JOIN(); }
            } else {
                const int ob = pos - NIN;
                int pm, pn; tile_of(ob >> 5, ob & 15, pm, pn);
                char* dst = P.out + ((size_t)(pm * 256 + (sub >> 1) * 64 + ((ob >> 4) & 1) * 32 + (lane & 15)) * NFULL + src * 2048 + pn * 128 + (sub & 1) * 64 + (lane >> 4) * 8) * 2;
                if (!dead) dec_output(rec, hdr, (en >> 15) != 0, tmp, dst, lane);
                else gst_agent(dst, (u32x4){0x7fc07fc0u, 0x7fc07fc0u, 0x7fc07fc0u, 0x7fc07fc0u});
                asm volatile("s_waitcnt lgkmcnt(0)" ::: "memory");
                ctl_inc(C_DONE + (i & 1u));
            }
        }
    }
}

__device__ __forceinline__ bool wait_operands(char* ws, int rank, int pm, int pn, u32 call, u32 mask, int lane) {
    const GAS u32* a = (const GAS u32*)(lane < 4 ? ws + OFF_RDY + ((lane & 1) ? 16 + pn : pm) * 128 + (lane < 2 ? 64 : 0) : ws + OFF_TMO);
    const int pushed = min(max(NRAW - 8 * pm, 0), 8);
    const u32 need = call * (lane == 0 ? 8u : lane == 1 ? 4u : lane == 2 ? 56u - 7u * pushed : 28u);
    const u64 t0 = now();
    u32 ok;
    for (;;) {
        const u32 v = __hip_atomic_load(a, __ATOMIC_RELAXED, __HIP_MEMORY_SCOPE_AGENT);
        ok = (u32)__ballot((int)(v - need) >= 0) & 15u;
        if (pushed) {
            const u64 f = __hip_atomic_load((const GAS u64*)(ws + OFF_FLR) + lane, __ATOMIC_RELAXED, __HIP_MEMORY_SCOPE_SYSTEM);
            if (!__all((lane >> 3) == rank || ((int)((u32)f - call) >= 0 && (int)((u32)(f >> 32) - call) >= 0))) ok &= ~4u;
        }
        const bool dead = __builtin_amdgcn_readlane(v, 4) == call;
        if (!dead && (ok & mask) == mask) break;
        if (dead || now() - t0 > DEADLINE) {
            __hip_atomic_store((GAS u32*)(ws + OFF_TMO), call, __ATOMIC_RELAXED, __HIP_MEMORY_SCOPE_AGENT);
            *ctl(C_DEAD) = 1; ok = 15u;
            break;
        }
        __builtin_amdgcn_s_sleep(4);
    }
    if (pushed) __builtin_amdgcn_fence(__ATOMIC_ACQUIRE, ""); else __builtin_amdgcn_fence(__ATOMIC_ACQUIRE, "agent");
    vm_drain();
    return ok == 15u;
}

__device__ __forceinline__ void gemm_tile(const Params& P, char* ws, char* wsl, int set, int t16, u32 call, int wid, int lane) {
    int pm, pn; tile_of(set, t16, pm, pn);
    bool have_all = true;
    if (wid == 0) have_all = wait_operands(ws, P.rank, pm, pn, call, 3u, lane);
    wg_barrier();
    STAMP(8 + (set >> 3) * 4);
    const char* ga = ws + OFF_XA + (size_t)pm * (2u << 20);
    const char* gb = ws + OFF_WB + (size_t)pn * (1u << 20);
    const u32 voa = wid * 4096 + lane * 16, vob = wid * 2048 + lane * 16;
    const u32 lbase = (u32)(size_t)lp(L_RB);
    const u32 lda = lbase + wid * 4096, ldb = lbase + 32768 + wid * 2048;
#define GLDS(vo, gp, lo, off) do { u32 keep_; asm volatile( \
        "s_mov_b32 %0, m0\n\ts_mov_b32 m0, %3\n\ts_nop 3\n\tglobal_load_lds_dwordx4 %1, %2 offset:" #off "\n\ts_mov_b32 m0, %0" \
        : "=&s"(keep_) : "v"(vo), "s"(gp), "s"(lo) : "memory"); } while (0)
#define GLDF(vo, gp, lo, off) asm volatile("s_mov_b32 m0, %2\n\ts_nop 0\n\tglobal_load_lds_dwordx4 %0, %1 offset:" #off ::"v"(vo), "s"(gp), "s"(lo) : "memory")
#define KA(kt) (ga + (size_t)(kt) * 32768)
#define KB(kt) (gb + (size_t)(kt) * 16384)
#define STAGE_LD(so, kt) do { const char* a_ = KA(kt); const char* b_ = KB(kt); \
        GLDS(voa, a_, lda + (so), 0); GLDS(voa, a_, lda + (so), 1024); GLDS(voa, a_, lda + (so), 2048); GLDS(voa, a_, lda + (so), 3072); \
        GLDS(vob, b_, ldb + (so), 0); GLDS(vob, b_, ldb + (so), 1024); } while (0)
    const u32 ra = L_RB + (wid >> 1) * 8192 + lane * 16, rb = L_RB + 32768 + (wid & 1) * 8192 + lane * 16;
#define FRAGS(A, B, so, kb) do { _Pragma("unroll") for (int q_ = 0; q_ < 4; ++q_) { \
        B[q_] = *(const LAS bf16x8*)lp(rb + (so) + q_ * 2048 + (kb) * 1024); A[q_] = *(const LAS bf16x8*)lp(ra + (so) + q_ * 2048 + (kb) * 1024); } } while (0)
#define MROW(A, B, m_) do { _Pragma("unroll") for (int n_ = 0; n_ < 4; ++n_) \
        acc[m_][n_] = __builtin_amdgcn_mfma_f32_16x16x32_bf16(B[n_], A[m_], acc[m_][n_], 0, 0, 0); } while (0)
#define SB() __builtin_amdgcn_sched_barrier(0)
#define MM1(A, B, m_, n_) acc[m_][n_] = __builtin_amdgcn_mfma_f32_16x16x32_bf16(B[n_], A[m_], acc[m_][n_], 0, 0, 0)
#define RDB(B, q_, kb) B[q_] = *(const LAS bf16x8*)lp(rb + so + (q_) * 2048 + (kb) * 1024)
#define RDA(A, q_, kb) A[q_] = *(const LAS bf16x8*)lp(ra + so + (q_) * 2048 + (kb) * 1024)
#define HALF(A, B, NA, NB, kb, G0, G1, G2) do { SB(); \
        MM1(A, B, 0, 0); SB(); RDA(NA, 0, kb); SB(); MM1(A, B, 0, 1); SB(); RDB(NB, 0, kb); SB(); MM1(A, B, 0, 2); SB(); RDB(NB, 1, kb); SB(); MM1(A, B, 0, 3); SB(); RDB(NB, 2, kb); SB(); \
        MM1(A, B, 1, 0); SB(); RDB(NB, 3, kb); SB(); MM1(A, B, 1, 1); SB(); RDA(NA, 1, kb); SB(); MM1(A, B, 1, 2); SB(); RDA(NA, 2, kb); SB(); MM1(A, B, 1, 3); SB(); RDA(NA, 3, kb); SB(); \
        MM1(A, B, 2, 0); SB(); G0; SB(); MM1(A, B, 2, 1); MM1(A, B, 2, 2); SB(); G1; SB(); MM1(A, B, 2, 3); \
        MM1(A, B, 3, 0); SB(); G2; SB(); MM1(A, B, 3, 1); MM1(A, B, 3, 2); MM1(A, B, 3, 3); SB(); } while (0)
#define MMA(A, B) do { MROW(A, B, 0); MROW(A, B, 1); MROW(A, B, 2); MROW(A, B, 3); } while (0)
    f32x4 acc[4][4];
#pragma unroll
    for (int m = 0; m < 4; ++m)
#pragma unroll
        for (int n = 0; n < 4; ++n) acc[m][n] = (f32x4){0.f, 0.f, 0.f, 0.f};
    bf16x8 a0[4], b0[4], a1[4], b1[4];
    STAGE_LD(0, 0); STAGE_LD(STAGE, 1);
    { const char* a_ = KA(2); GLDS(voa, a_, lda + 2 * STAGE, 0); GLDS(voa, a_, lda + 2 * STAGE, 1024); GLDS(voa, a_, lda + 2 * STAGE, 2048); }
    asm volatile("s_waitcnt vmcnt(9)\n\ts_barrier" ::: "memory");
    FRAGS(a0, b0, 0u, 0);
    u32 so = 0, sn, sp = 2 * STAGE;
    const char* pa = KA(2); const char* pb = KB(2);
#pragma unroll 1
    for (int phase = 0; phase < 2; ++phase) {
#pragma unroll 1
        for (int n = phase ? 56 : 5; n > 0; --n) {
            HALF(a0, b0, a1, b1, 1, GLDF(voa, pa, lda + sp, 3072), GLDF(vob, pb, ldb + sp, 0),
                 GLDF(vob, pb, ldb + sp, 1024); pa += 32768; pb += 16384; sp = so; sn = so == 2 * STAGE ? 0 : so + STAGE; asm volatile("" : "+s"(pa), "+s"(pb), "+s"(sn)));
            asm volatile("s_waitcnt vmcnt(6) lgkmcnt(0)\n\ts_barrier" ::: "memory");
            so = sn;
            HALF(a1, b1, a0, b0, 0, GLDF(voa, pa, lda + sp, 0), GLDF(voa, pa, lda + sp, 1024), GLDF(voa, pa, lda + sp, 2048));
        }
        if (phase == 0 && !have_all) { wait_operands(ws, P.rank, pm, pn, call, 15u, lane); STAMP(11 + (set >> 3) * 4); }
    }
    {
        const char* a2 = KA(63); const char* b2 = KB(63);
        GLDS(voa, a2, lda + sp, 3072); GLDS(vob, b2, ldb + sp, 0); GLDS(vob, b2, ldb + sp, 1024);
        FRAGS(a1, b1, so, 1); MMA(a0, b0);
        asm volatile("s_waitcnt vmcnt(6) lgkmcnt(0)\n\ts_barrier" ::: "memory");
        so = so == 2 * STAGE ? 0 : so + STAGE;
        FRAGS(a0, b0, so, 0); MMA(a1, b1);
        FRAGS(a1, b1, so, 1); MMA(a0, b0);
        asm volatile("s_waitcnt vmcnt(0) lgkmcnt(0)\n\ts_barrier" ::: "memory");
        so = so == 2 * STAGE ? 0 : so + STAGE;
        FRAGS(a0, b0, so, 0); MMA(a1, b1);
        FRAGS(a1, b1, so, 1); MMA(a0, b0); MMA(a1, b1);
    }
#undef GLDS
#undef KA
#undef KB
#undef STAGE_LD
#undef FRAGS
#undef MROW
#undef MMA
#undef MM1
#undef GLDF
#undef RDA
#undef RDB
#undef HALF
#undef SB
    wg_barrier();
    STAMP(9 + (set >> 3) * 4);
    const float floor0 = ctl_ld(C_DEAD) != 0 ? __builtin_nanf("") : 0.f;
    char* dst = P.out + ((size_t)(pm * 256 + (wid >> 1) * 64 + (lane & 15)) * NFULL + P.rank * 2048 + pn * 128 + (wid & 1) * 64 + (lane >> 4) * 8) * 2;
#pragma unroll
    for (int b = 0; b < 2; ++b) {
        u32x4 v[4];
#pragma unroll
        for (int j = 0; j < 4; ++j) {
            const f32x4 p = acc[2 * b + (j >> 1)][2 * (j & 1)], q = acc[2 * b + (j >> 1)][2 * (j & 1) + 1];
            float f[8] = {p[0], p[1], p[2], p[3], q[0], q[1], q[2], q[3]};
            u32 r[4];
#pragma unroll
            for (int e = 0; e < 4; ++e) {
                const float lo = __builtin_elementwise_maximum(f[2 * e], floor0), hi = __builtin_elementwise_maximum(f[2 * e + 1], floor0);
                asm("v_cvt_pk_bf16_f32 %0, %1, %2" : "=v"(r[e]) : "v"(lo), "v"(hi));
                r[e] &= 0x7fff7fffu;
            }
            v[j] = (u32x4){r[0], r[1], r[2], r[3]};
            gst_agent(dst + (size_t)(2 * b + (j >> 1)) * (16 * NFULL * 2) + (j & 1) * 64, v[j]);
        }
        enc_output(v, wid, lane);
        const int ob = set * 32 + b * 16 + t16;
        const u32 len = emit_block(ws + OFF_CO + (size_t)ob * SLOT, wid, lane);
        if (threadIdx.x >= 1 && threadIdx.x < 8) flag_store(wsl, P.rank, NIN + ob, (call << 12) | len);
        JOIN();
    }
    STAMP(10 + (set >> 3) * 4);
}

__global__ void __launch_bounds__(512, 2) fused_gemm_ar(const Params P) {
    const int tid = threadIdx.x, lane = tid & 63, wid = uni(tid >> 6), blk = blockIdx.x;
    char* ws = P.ws[0];
    const int xcd = blk & 7, idx = blk >> 3;
    const bool is_dec = blk < NDEC;
    const int slab = 1 + (idx >= 7 ? idx - 7 : idx), d = (idx >= 7 ? 8 : 0) + xcd;
    u32x4 nx[4];
    u32 warm = 0;
    char* wsl = nullptr; char* sws = nullptr;
    if (!is_dec) {
        pack_load(P, blk - NDEC, wid, lane, nx);
        wsl = P.ws[lane & 7];
        if (wid == 7) warm = *(const GAS u32*)(wsl + OFF_FL + P.rank * 4096);
    } else {
        sws = P.ws[slab & 7];
        if (wid >= 2) {
            const int g = d * (NRAW / 2) + min(wid - 2, NRAW / 2 - 1);
            pack_load(P, g >> 3, g & 7, lane, nx);
            warm = *(const GAS u32*)(sws + OFF_XA + (size_t)((g >> 6) * 64 + (8 - slab) * 8) * 32768);
        } else {
            u64 pc; asm volatile("s_getpc_b64 %0" : "=s"(pc));
            warm = *(const GAS u32*)((pc & ~127ull) + (u64)tid * 128u);
        }
    }
    STAMP(0);
    if (tid < 256) {
        u32 e[2] = {0x0c0c0c0cu, 0x0c0c0c0cu}, c[2] = {0x0c0c0c0cu, 0x0c0c0c0cu};
        u64 ee = 0x0c0c0c0c0c0c0c0cull, cc = ee; int k = 0;
#pragma unroll
        for (int i = 0; i < 8; ++i)
            if ((tid >> i) & 1) { ee ^= (u64)(0x0c ^ k) << (8 * i); cc ^= (u64)(0x0c ^ i) << (8 * k); ++k; }
        e[0] = (u32)ee; e[1] = (u32)(ee >> 32); c[0] = (u32)cc; c[1] = (u32)(cc >> 32);
        st64(L_LUTE + tid * 8, e[0], e[1]); st64(L_LUTC + tid * 8, c[0], c[1]);
    } else if (tid < 256 + 128) {
        u32 v = 0;
        if (tid == 256 + C_CALL) v = __hip_atomic_fetch_add((GAS u32*)(ws + OFF_CTR + blk * 64), 1u, __ATOMIC_RELAXED, __HIP_MEMORY_SCOPE_AGENT) + 1u;
        *ctl(tid - 256) = v;
    }
    wg_barrier();
    const u32 call = ctl_ld(C_CALL);
    if (!is_dec) {
        const int late = idx >= 30 ? 96 + blk - 240 : (xcd >= 2 ? (idx - 14) * 6 + xcd - 2 : NTASK);
        for (int task = blk - NDEC; task < NTASK;) {
            const int next = task < 288 ? task + 144 : (task < 432 ? 432 + late : task + 112);
            u32x4 v[4] = {nx[0], nx[1], nx[2], nx[3]};
            if (next < NTASK) pack_load(P, next, wid, lane, nx);
            pack_task(P, ws, wsl, task, call, wid, lane, v);
            task = next;
        }
    }
    asm volatile("" ::"v"(warm));
    STAMP(1);
    if (is_dec) {
        decoder(P, ws, sws, slab, d, call, wid, lane, nx);
    } else if (idx < 30) {
        for (int rr = 0; rr < 2; ++rr) gemm_tile(P, ws, wsl, xcd + 8 * rr, idx - 14, call, wid, lane);
    }
}

}

extern "C" void kernel_launch(int rank, int nranks, ncclComm_t comm, hipStream_t stream,
                              void* const* d_in, const int* in_sizes, int n_in,
                              void* d_out, int out_size, const int* in_dtypes, int out_dtype,
                              void* const* d_out_all, void* const* d_ws_all, size_t ws_bytes) {
    if (nranks != NR || (unsigned)rank >= (unsigned)NR || n_in != 2 || ws_bytes < WS_NEED ||
        in_sizes[0] != 4096 * 512 || in_sizes[1] != 512 * NFULL || out_size != 4096 * NFULL) {
        fprintf(stderr, "kernel_launch: unexpected problem shape\n");
        return;
    }
    Params P;
    P.x = (const char*)d_in[0]; P.w = (const char*)d_in[1]; P.out = (char*)d_out;
    for (int p = 0; p < NR; ++p) P.ws[p] = (char*)d_ws_all[(rank + p) % NR];
    P.rank = rank; P.pad = 0;
    hipLaunchKernelGGL(fused_gemm_ar, dim3(NWG), dim3(512), 0, stream, P);
    hipError_t e = hipGetLastError();
    if (e != hipSuccess) fprintf(stderr, "launch failed: %s\n", hipGetErrorString(e));
}
```

```cpp
#include <hip/hip_runtime.h>
#include <rccl/rccl.h>
#include <cstdio>

namespace {

typedef unsigned u32;
typedef unsigned long long u64;
typedef u32 u32x4 __attribute__((ext_vector_type(4)));
typedef u32 u32x2 __attribute__((ext_vector_type(2)));
typedef short bf16x8 __attribute__((ext_vector_type(8)));
typedef float f32x4 __attribute__((ext_vector_type(4)));
#define LAS __attribute__((address_space(3)))
#define GAS __attribute__((address_space(1)))

constexpr int NR = 8;
constexpr int NFULL = 16384;
constexpr int NWG = 256;
constexpr int DPP_ = 16;
constexpr int NIN = 192, NOUT = 512;
constexpr int NPOS = NIN + NOUT;
constexpr int NDEC = 7 * DPP_;
constexpr int NTASK = 640;
#define QD 3
#define QT 8
#define NTAIL 0
#define NRAW 8
#define PUSHQ 0
constexpr u64 DEADLINE = 50000000ull;

constexpr size_t OFF_CTR = 0;
constexpr size_t OFF_TMO = 16384;
constexpr size_t OFF_RDY = 16384 + 256;
constexpr size_t OFF_CLM = 24576;
constexpr size_t OFF_FLR = 28672;
constexpr size_t OFF_FL = 32768;
constexpr size_t OFF_DBG = 65536;
constexpr size_t OFF_XA = 1u << 20;
constexpr size_t OFF_WB = OFF_XA + (32u << 20);
constexpr size_t SLOT = 46080;
constexpr size_t OFF_CX = OFF_WB + (16u << 20);
constexpr size_t OFF_CW = OFF_CX + 128 * SLOT;
constexpr size_t OFF_CO = OFF_CW + 512 * SLOT;
constexpr size_t WS_NEED = OFF_CO + 512 * SLOT;

constexpr int L_LUTE = 0;
constexpr int L_LUTC = 2048;
constexpr int L_CTL = 4096;
constexpr int L_RB = 4608;
constexpr int STAGE = 49152;
constexpr int LDS_BYTES = L_RB + 3 * STAGE;
constexpr int TB = 2112;
constexpr int RECB = 5696;
constexpr int ENC_W = RECB + 4 * TB;
constexpr int LSLOT = 46080;
constexpr int L_DT = L_RB + 2 * LSLOT;
static_assert(L_DT + 6 * 4 * TB <= LDS_BYTES, "lds");
static_assert(8 * ENC_W <= 3 * STAGE, "lds");
constexpr int C_HDR = 0  , C_CALL = 16, C_DEAD = 17, C_LANDED = 18, C_CLAIM = 19,
              C_NBLK = 22, C_PROG = 23, C_STORED = 24  , C_QUEUE = 40  , C_PUSH = 21, C_DONE = 48  , C_SLOT = 56  ;

struct Params {
    const char* x; const char* w; char* out;
    char* ws[NR];
    int rank; int pad;
};

__shared__ __attribute__((aligned(16))) unsigned char smem[LDS_BYTES];

#define CBAR() asm volatile("" ::: "memory")
__device__ __forceinline__ LAS unsigned char* lp(u32 o) { return (LAS unsigned char*)smem + o; }
__device__ __forceinline__ u32 ld8(u32 o) { return *lp(o); }
__device__ __forceinline__ u32 ld32(u32 o) { return *(LAS u32*)lp(o); }
__device__ __forceinline__ u32x2 ld64(u32 o) { return *(LAS u32x2*)lp(o); }
__device__ __forceinline__ u32x4 ld128(u32 o) { return *(LAS u32x4*)lp(o); }
__device__ __forceinline__ void st8(u32 o, u32 v) { *lp(o) = (unsigned char)v; }
__device__ __forceinline__ void st32(u32 o, u32 v) { *(LAS u32*)lp(o) = v; }
__device__ __forceinline__ void st64(u32 o, u32 a, u32 b) { *(LAS u32x2*)lp(o) = (u32x2){a, b}; }
__device__ __forceinline__ void st128(u32 o, u32x4 v) { *(LAS u32x4*)lp(o) = v; }
__device__ __forceinline__ void or32(u32 o, u32 v) { __hip_atomic_fetch_or((LAS u32*)lp(o), v, __ATOMIC_RELAXED, __HIP_MEMORY_SCOPE_WORKGROUP); }
__device__ __forceinline__ volatile LAS u32* ctl(int i) { return (volatile LAS u32*)lp(L_CTL + 4 * i); }
__device__ __forceinline__ u32 uni(u32 v) { return __builtin_amdgcn_readfirstlane(v); }
__device__ __forceinline__ u32 ctl_ld(int i) { return uni(*ctl(i)); }
__device__ __forceinline__ u32 ctl_inc(int i) {
    u32 r; u64 sv;
    asm volatile("s_mov_b64 %1, exec\n\ts_mov_b64 exec, 1\n\tds_add_rtn_u32 %0, %2, %3\n\ts_waitcnt lgkmcnt(0)\n\ts_mov_b64 exec, %1"
                 : "=&v"(r), "=&s"(sv) : "v"((u32)(size_t)lp(L_CTL + 4 * i)), "v"(1u) : "memory");
    return uni(r);
}
__device__ __forceinline__ void ctl_add1(int i) {
    u64 sv;
    asm volatile("s_mov_b64 %0, exec\n\ts_mov_b64 exec, 1\n\tds_add_u32 %1, %2\n\ts_mov_b64 exec, %0"
                 : "=&s"(sv) : "v"((u32)(size_t)lp(L_CTL + 4 * i)), "v"(1u) : "memory");
}
#define JOIN() __builtin_amdgcn_wave_barrier()
__device__ __forceinline__ u32 perm(u32 hi, u32 lo, u32 sel) { return __builtin_amdgcn_perm(hi, lo, sel); }
__device__ __forceinline__ u32 bfi(u32 m, u32 a, u32 b) { return (m & a) | (~m & b); }
__device__ __forceinline__ u64 now() { return __builtin_amdgcn_s_memrealtime(); }
__device__ __forceinline__ void wg_barrier() { asm volatile("s_waitcnt lgkmcnt(0)\n\ts_barrier" ::: "memory"); }
__device__ __forceinline__ void vm_drain() { asm volatile("s_waitcnt vmcnt(0)" ::: "memory"); }

__device__ __forceinline__ void gst_agent(void* p, u32x4 v) { asm volatile("global_store_dwordx4 %0, %1, off sc1\n\ts_nop 1" ::"v"(p), "v"(v) : "memory"); }
__device__ __forceinline__ void gst_sys(void* p, u32x4 v) { asm volatile("global_store_dwordx4 %0, %1, off sc0 sc1\n\ts_nop 1" ::"v"(p), "v"(v) : "memory"); }

__device__ __forceinline__ u32 scan(u32 x) {
    x += __builtin_amdgcn_update_dpp(0u, x, 0x111, 0xf, 0xf, false);
    x += __builtin_amdgcn_update_dpp(0u, x, 0x112, 0xf, 0xf, false);
    x += __builtin_amdgcn_update_dpp(0u, x, 0x114, 0xf, 0xf, false);
    x += __builtin_amdgcn_update_dpp(0u, x, 0x118, 0xf, 0xf, false);
    x += __builtin_amdgcn_update_dpp(0u, x, 0x142, 0xa, 0xf, false);
    x += __builtin_amdgcn_update_dpp(0u, x, 0x143, 0xc, 0xf, false);
    return x;
}
__device__ __forceinline__ u32 last(u32 x) { return __builtin_amdgcn_readlane(x, 63); }
__device__ __forceinline__ u32 offsets4(u32 m, u32 (&o)[4]) {
    const u32 c01 = __popc(m & 0xffu) | (__popc(m & 0xff00u) << 16), c23 = __popc(m & 0xff0000u) | (__popc(m >> 24) << 16);
    const u32 s01 = scan(c01), s23 = scan(c23);
    const u32 t01 = last(s01), t23 = last(s23);
    const u32 e01 = s01 - c01, e23 = s23 - c23;
    const u32 b1 = t01 & 0xffffu, b2 = b1 + (t01 >> 16), b3 = b2 + (t23 & 0xffffu);
    o[0] = e01 & 0xffffu; o[1] = (e01 >> 16) + b1; o[2] = (e23 & 0xffffu) + b2; o[3] = (e23 >> 16) + b3;
    return b3 + (t23 >> 16);
}
__device__ __forceinline__ u32 gather8(u32 y0, u32 y1) {
    return __builtin_amdgcn_udot4(y1, 0x80402010u, __builtin_amdgcn_udot4(y0, 0x08040201u, 0u, false), false);
}
__device__ __forceinline__ u32 spread1(u32 n4) { return __umul24(n4, 0x204081u) & 0x01010101u; }
__device__ __forceinline__ u32 spread2(u32 n4) { return __umul24(n4, 0x408102u) & 0x02020202u; }
__device__ __forceinline__ u32 subb(u32 a, u32 brep) {
    return ((a | 0x80808080u) - (brep & 0x7f7f7f7fu)) ^ ((a ^ ~brep) & 0x80808080u);
}
template <int K> __device__ __forceinline__ u32 geb(u32 d) {
    return ((((d & 0x7f7f7f7fu) + (u32)(0x80 - K) * 0x01010101u) | d) >> 7) & 0x01010101u;
}
__device__ __forceinline__ void fetch8(u32 a, u32& s0, u32& s1) {
    const u32 q = a & ~3u;
    const u32 d0 = ld32(q), d1 = ld32(q + 4), d2 = ld32(q + 8);
    s0 = __builtin_amdgcn_alignbyte(d1, d0, a); s1 = __builtin_amdgcn_alignbyte(d2, d1, a);
}
__device__ __forceinline__ void or32nz(u32 o, u32 v, int lane) { or32(v ? o : (u32)lane * 4u, v); }
__device__ __forceinline__ void scatter8(u32 a, u32 c0, u32 c1, int lane) {
    const u32 q = a & ~3u, sel = 0x07060504u - perm(0u, a & 3u, 0u);
    or32nz(q, perm(c0, 0u, sel), lane); or32nz(q + 4, perm(c1, c0, sel), lane); or32nz(q + 8, perm(0u, c1, sel), lane);
}
constexpr u32 JUNK = 4352;
__device__ __forceinline__ void compact16(u32 a, u32x4 v, u32x4 z) {
    u32 ad = (u32)(size_t)lp(a - JUNK), t0, t1;
#define C16(d, f) "v_mad_u32_u16 %1, " f ", %11, %0\n\tv_mad_u32_u16 %0, " f ", 2, %0\n\tv_mad_u32_u16 %2, " f ", %11, %0 op_sel:[1,0,0,0]\n\t" \
                  "ds_write_b16 %1, " d "\n\tv_mad_u32_u16 %0, " f ", 2, %0 op_sel:[1,0,0,0]\n\tds_write_b16_d16_hi %2, " d "\n\t"
    asm volatile(C16("%3", "%7") C16("%4", "%8") C16("%5", "%9") C16("%6", "%10")
                 : "+v"(ad), "=&v"(t0), "=&v"(t1) : "v"(v.x), "v"(v.y), "v"(v.z), "v"(v.w), "v"(z.x), "v"(z.y), "v"(z.z), "v"(z.w), "s"(JUNK) : "memory");
#undef C16
}
__device__ __forceinline__ void zero_lds(u32 a, u32 bytes, int lane) {
    for (u32 i = lane * 16; i < bytes; i += 1024) st128(a + i, (u32x4){0u, 0u, 0u, 0u});
}
typedef unsigned short u16x2 __attribute__((ext_vector_type(2)));
__device__ __forceinline__ u32 pk_nmad2(u32 v, u32 k) {
    u32 r; asm("v_pk_mad_u16 %0, %1, -2, %2 op_sel_hi:[1,0,1]" : "=v"(r) : "v"(v), "s"(k)); return r;
}
__device__ __forceinline__ u32 pk_nz(u32 v) {
    u32 r; asm("v_pk_min_u16 %0, %1, 1 op_sel_hi:[1,0]" : "=v"(r) : "v"(v)); return r;
}
__device__ __forceinline__ u32 pk_min(u32 v, u32 k) {
    return __builtin_bit_cast(u32, __builtin_elementwise_min(__builtin_bit_cast(u16x2, v), __builtin_bit_cast(u16x2, k)));
}
__device__ __forceinline__ u32 valid8(int left) { return left >= 8 ? 255u : (left > 0 ? (1u << left) - 1u : 0u); }

__device__ __forceinline__ void fetch_l4(u32 l4, u32 o, u32 top, bool wide, u32& s0, u32& s1) {
    if (wide) { fetch8(l4 + o, s0, s1); s0 = subb(top, s0); s1 = subb(top, s1); }
    else {
        const u32 bit = l4 * 8u + o * 4u, q = (bit >> 5) << 2;
        const u32 xx = __builtin_amdgcn_alignbit(ld32(q + 4), ld32(q), bit);
        const u32 lo = xx & 0x0f0f0f0fu, hi = (xx >> 4) & 0x0f0f0f0fu;
        s0 = perm(hi, lo, 0x05010400u); s1 = perm(hi, lo, 0x07030602u);
        const u32 g0 = ((s0 + 0x0e0e0e0eu) >> 4) & 0x01010101u, g1 = ((s1 + 0x0e0e0e0eu) >> 4) & 0x01010101u;
        s0 = top - s0 - ((g0 << 3) + g0); s1 = top - s1 - ((g1 << 3) + g1);
    }
}
template <bool LAST>
__device__ __forceinline__ void dec_level(u32 pl, u32 nb, u32 tab, u32 ts, u32 td, u32 top, bool wide, int lane) {
    u32 run = 0;
    for (u32 p = 0; p < nb; p += 64) {
        const u32 idx = p + lane;
        const u32 a8 = ld8(pl + 2 * idx), b8 = ld8(pl + 2 * idx + 1), more = (a8 | b8) ^ 255u;
        const u32x2 sel = ld64(L_LUTE + more * 8u);
        const u32 cnt = __popc(more), inc = scan(cnt), o = run + inc - cnt;
        run += last(inc);
        const u32 c0 = spread1(a8 & 15u) | spread2(b8 & 15u), c1 = spread1(a8 >> 4) | spread2(b8 >> 4);
        u32 s0, s1;
        if (!LAST) fetch8(ts + o, s0, s1);
        else fetch_l4(ts, o, top, wide, s0, s1);
        st64(td + idx * 8u, perm(0u, tab, c0) | perm(s1, s0, sel.x), perm(0u, tab, c1) | perm(s1, s0, sel.y));
    }
    CBAR();
}
__device__ __forceinline__ void dec_l23(u32 l2, u32 hdr, u32 base, u32 t2, int lane) {
    const u32 nb2 = min((hdr >> 11) & 511u, 256u), nb3 = min((hdr >> 20) & 511u, 256u), l3 = l2 + 2 * nb2;
    dec_level<true>(l3, nb3, ((base - 3) << 8) | ((base - 4) << 16) | ((base - 5) << 24), l3 + 2 * nb3, t2 + TB, (base + 5) * 0x01010101u, (hdr >> 29) & 1u, lane);
    dec_level<false>(l2, nb2, ((base + 3) << 8) | ((base - 1) << 16) | ((base - 2) << 24), t2 + TB, t2, 0u, false, lane);
}

__device__ __forceinline__ void dec_input(u32 rec, u32 hdr, u32 base, u32 tmp, char* dst, int lane) {
    const u32 t2 = tmp + 2 * TB;
    const u32 pa = ld32(rec + 2048 + lane * 4), pb = ld32(rec + 2304 + lane * 4), more = ~(pa | pb);
    u32 o[4]; offsets4(more, o);
    dec_l23(rec + 2560, hdr, base, t2, lane);
    const u32 tab = ((base + 2) << 8) | ((base + 1) << 16) | (base << 24);
    u32x4 r[4];
#pragma unroll
    for (int j = 0; j < 4; ++j) {
        const u32 m0 = ld32(rec + j * 512 + lane * 8), m1 = ld32(rec + j * 512 + lane * 8 + 4);
        const u32 c0 = spread1((pa >> (8 * j)) & 15u) | spread2((pb >> (8 * j)) & 15u);
        const u32 c1 = spread1((pa >> (8 * j + 4)) & 15u) | spread2((pb >> (8 * j + 4)) & 15u);
        u32 s0, s1; fetch8(t2 + o[j], s0, s1);
        const u32x2 sel = ld64(L_LUTE + ((more >> (8 * j)) & 255u) * 8u);
        const u32 e0 = perm(0u, tab, c0) | perm(s1, s0, sel.x), e1 = perm(0u, tab, c1) | perm(s1, s0, sel.y);
        const u32 h0 = bfi(0x80808080u, m0, e0 >> 1), h1 = bfi(0x80808080u, m1, e1 >> 1);
        const u32 l0 = bfi(0x80808080u, e0 << 7, m0), l1 = bfi(0x80808080u, e1 << 7, m1);
        r[j] = (u32x4){perm(h0, l0, 0x05010400u), perm(h0, l0, 0x07030602u), perm(h1, l1, 0x05010400u), perm(h1, l1, 0x07030602u)};
    }
#pragma unroll
    for (int j = 0; j < 4; ++j) gst_agent(dst + j * 1024, r[j]);
}

__device__ __forceinline__ void dec_output(u32 rec, u32 hdr, u32 base, u32 tmp, char* dst, int done, int lane) {
    const u32 t1l = tmp, t1h = tmp + TB, t2 = tmp + 2 * TB, t3 = t2 + TB;
    const u32 m = ld32(rec + lane * 4);
    u32 bb[3][2];
#pragma unroll
    for (int p = 0; p < 3; ++p) { bb[p][0] = ld32(rec + 256 + p * 512 + lane * 8); bb[p][1] = ld32(rec + 260 + p * 512 + lane * 8); }
    u32x2 selm[4];
#pragma unroll
    for (int j = 0; j < 4; ++j) selm[j] = ld64(L_LUTE + ((m >> (8 * j)) & 255u) * 8u);
    u32 o[4]; const u32 n1 = offsets4(m, o);
    const u32 pbo = rec + 256 + n1, nb = (n1 + 7) >> 3;
    const u32 tab = ((base + 2) << 8) | ((base + 1) << 16) | (base << 24);
    const u32 nb2 = (hdr >> 11) & 511u, nb3 = (hdr >> 20) & 511u;
    if (nb <= 192u && nb2 <= 64u && nb3 <= 64u) {
        const u32 l2 = pbo + nb, l3 = l2 + 2 * nb2;
        const u32 a3 = ld8(l3 + 2 * lane), b3 = ld8(l3 + 2 * lane + 1), a2 = ld8(l2 + 2 * lane), b2 = ld8(l2 + 2 * lane + 1);
        u32 b8[3], a[3][2], more[3]; u32x2 sel[3];
#pragma unroll
        for (int p = 0; p < 3; ++p) b8[p] = ld8(pbo + p * 64 + lane);
        const u32 more3 = (a3 | b3) ^ 255u, more2 = (a2 | b2) ^ 255u;
        const u32x2 sel3 = ld64(L_LUTE + more3 * 8u), sel2 = ld64(L_LUTE + more2 * 8u);
#pragma unroll
        for (int p = 0; p < 3; ++p) {
            a[p][0] = (bb[p][0] >> 7) & 0x01010101u; a[p][1] = (bb[p][1] >> 7) & 0x01010101u;
            more[p] = (gather8(a[p][0], a[p][1]) | b8[p]) ^ 255u;
            sel[p] = ld64(L_LUTE + more[p] * 8u);
        }
        const u32 cx = __popc(more3) | (__popc(more2) << 16), cy = __popc(more[0]) | (__popc(more[1]) << 10) | (__popc(more[2]) << 20);
        const u32 sx = scan(cx), sy = scan(cy), ty = last(sy), ex = sx - cx, ey = sy - cy;
        u32 s0, s1;
        fetch_l4(l3 + 2 * nb3, ex & 0xffffu, (base + 5) * 0x01010101u, (hdr >> 29) & 1u, s0, s1);
        ctl_add1(done);
        const u32 tab3 = ((base - 3) << 8) | ((base - 4) << 16) | ((base - 5) << 24), tab2 = ((base + 3) << 8) | ((base - 1) << 16) | ((base - 2) << 24);
        st64(t3 + lane * 8, perm(0u, tab3, spread1(a3 & 15u) | spread2(b3 & 15u)) | perm(s1, s0, sel3.x),
                            perm(0u, tab3, spread1(a3 >> 4) | spread2(b3 >> 4)) | perm(s1, s0, sel3.y));
        CBAR();
        fetch8(t3 + (ex >> 16), s0, s1);
        st64(t2 + lane * 8, perm(0u, tab2, spread1(a2 & 15u) | spread2(b2 & 15u)) | perm(s1, s0, sel2.x),
                            perm(0u, tab2, spread1(a2 >> 4) | spread2(b2 >> 4)) | perm(s1, s0, sel2.y));
        CBAR();
        u32 f[3][2];
        fetch8(t2 + (ey & 1023u), f[0][0], f[0][1]);
        fetch8(t2 + (ty & 1023u) + ((ey >> 10) & 1023u), f[1][0], f[1][1]);
        fetch8(t2 + (ty & 1023u) + ((ty >> 10) & 1023u) + (ey >> 20), f[2][0], f[2][1]);
#pragma unroll
        for (int p = 0; p < 3; ++p) {
            const u32 e0 = perm(0u, tab, a[p][0] | spread2(b8[p] & 15u)) | perm(f[p][1], f[p][0], sel[p].x);
            const u32 e1 = perm(0u, tab, a[p][1] | spread2(b8[p] >> 4)) | perm(f[p][1], f[p][0], sel[p].y);
            st64(t1l + p * 512 + lane * 8, bfi(0x80808080u, e0 << 7, bb[p][0]), bfi(0x80808080u, e1 << 7, bb[p][1]));
            st64(t1h + p * 512 + lane * 8, (e0 >> 1) & 0x7f7f7f7fu, (e1 >> 1) & 0x7f7f7f7fu);
        }
    } else {
        dec_l23(pbo + nb, hdr, base, t2, lane);
        u32 run = 0;
        for (u32 p = 0; p < nb; p += 64) {
            const u32 idx = p + lane;
            const u32 b0 = ld32(rec + 256 + idx * 8u), b1 = ld32(rec + 260 + idx * 8u);
            const u32 b8 = ld8(pbo + idx);
            const u32 a0 = (b0 >> 7) & 0x01010101u, a1 = (b1 >> 7) & 0x01010101u;
            const u32 more = (gather8(a0, a1) | b8) ^ 255u;
            const u32x2 sel = ld64(L_LUTE + more * 8u);
            const u32 cnt = __popc(more), inc = scan(cnt), o2 = run + inc - cnt;
            run += last(inc);
            const u32 c0 = a0 | spread2(b8 & 15u), c1 = a1 | spread2(b8 >> 4);
            u32 s0, s1; fetch8(t2 + o2, s0, s1);
            const u32 e0 = perm(0u, tab, c0) | perm(s1, s0, sel.x), e1 = perm(0u, tab, c1) | perm(s1, s0, sel.y);
            st64(t1l + idx * 8u, bfi(0x80808080u, e0 << 7, b0), bfi(0x80808080u, e1 << 7, b1));
            st64(t1h + idx * 8u, (e0 >> 1) & 0x7f7f7f7fu, (e1 >> 1) & 0x7f7f7f7fu);
        }
        CBAR();
        ctl_add1(done);
    }
    CBAR();
    u32x4 r[4];
#pragma unroll
    for (int j = 0; j < 4; ++j) {
        u32 l0, l1, h0, h1;
        fetch8(t1l + o[j], l0, l1); fetch8(t1h + o[j], h0, h1);
        const u32 L0 = perm(l1, l0, selm[j].x), L1 = perm(l1, l0, selm[j].y), H0 = perm(h1, h0, selm[j].x), H1 = perm(h1, h0, selm[j].y);
        r[j] = (u32x4){perm(H0, L0, 0x05010400u), perm(H0, L0, 0x07030602u), perm(H1, L1, 0x05010400u), perm(H1, L1, 0x07030602u)};
    }
#pragma unroll
    for (int j = 0; j < 4; ++j) gst_agent(dst + (size_t)j * (8 * NFULL * 2), r[j]);
}

__device__ __forceinline__ void enc_l234(u32 rec, u32 l2, u32 n2, u32 base, u32 t2, int wid, int lane) {
    const u32 nb2 = (n2 + 7) >> 3, t3 = t2 + TB;
    u32 n3 = 0;
    for (u32 p = 0; p < nb2; p += 64) {
        const u32 idx = p + lane;
        const u32x2 g = ld64(t2 + idx * 8u);
        const int left = (int)n2 - (int)(idx * 8u);
        const u32 k0 = geb<6>(g.x) ^ 0x01010101u, k1 = geb<6>(g.y) ^ 0x01010101u;
        const u32 a8 = gather8((g.x | ~(g.x >> 2)) & k0, (g.y | ~(g.y >> 2)) & k1), b8 = gather8((g.x >> 2) & k0, (g.y >> 2) & k1);
        const u32 more = ~(a8 | b8) & valid8(left);
        const u32x2 sel = ld64(L_LUTC + more * 8u);
        if (left > 0) { st8(l2 + 2 * idx, a8); st8(l2 + 2 * idx + 1, b8); }
        const u32 cnt = __popc(more), inc = scan(cnt), o = n3 + inc - cnt;
        n3 += last(inc);
        scatter8(t3 + o, perm(g.y, g.x, sel.x), perm(g.y, g.x, sel.y), lane);
    }
    CBAR();
    const u32 nb3 = (n3 + 7) >> 3, l3 = l2 + 2 * nb2, l4 = l3 + 2 * nb3;
    u32 n4;
    bool wide = false;
    for (;;) {
        if (lane < 16) st8(l4 + lane, 0u);
        zero_lds((l4 + 15u) & ~15u, (wide ? n3 : n3 >> 1) + 8u, lane);
        CBAR();
        u32 out = 0;
        n4 = 0;
        for (u32 p = 0; p < nb3; p += 64) {
            const u32 idx = p + lane;
            const u32x2 g = ld64(t3 + idx * 8u);
            const int left = (int)n3 - (int)(idx * 8u);
            const u32 k0 = geb<9>(g.x) ^ 0x01010101u, k1 = geb<9>(g.y) ^ 0x01010101u;
            const u32 t0 = (g.x & 0x0f0f0f0fu) + 0x03030303u, t1 = (g.y & 0x0f0f0f0fu) + 0x03030303u;
            const u32 a8 = gather8(t0 & k0, t1 & k1), b8 = gather8((t0 >> 1) & k0, (t1 >> 1) & k1);
            const u32 more = ~(a8 | b8) & valid8(left);
            const u32x2 sel = ld64(L_LUTC + more * 8u);
            if (left > 0) { st8(l3 + 2 * idx, a8); st8(l3 + 2 * idx + 1, b8); }
            const u32 cnt = __popc(more), inc = scan(cnt), o = n4 + inc - cnt;
            n4 += last(inc);
            const u32 u0 = ((g.x & 0x7f7f7f7fu) + 0x02020202u) ^ (g.x & 0x80808080u), u1 = ((g.y & 0x7f7f7f7fu) + 0x02020202u) ^ (g.y & 0x80808080u);
            u32 c0 = perm(u1, u0, sel.x), c1 = perm(u1, u0, sel.y);
            if (wide) scatter8(l4 + o, c0, c1, lane);
            else {
                out |= geb<25>(c0) | geb<25>(c1);
                const u32 q0 = ((c0 + 0x05050505u) >> 4) & 0x01010101u, q1 = ((c1 + 0x05050505u) >> 4) & 0x01010101u;
                c0 -= (q0 << 3) + q0; c1 -= (q1 << 3) + q1;
                const u32 xx = perm(c1 | (c1 >> 4), c0 | (c0 >> 4), 0x06040200u);
                const u32 bit = l4 * 8u + o * 4u, q = (bit >> 5) << 2;
                const u64 w = (u64)xx << (bit & 31u);
                or32nz(q, (u32)w, lane); or32nz(q + 4, (u32)(w >> 32), lane);
            }
        }
        CBAR();
        if (wide || !__any(out != 0)) break;
        wide = true;
    }
    const u32 end = l4 + (wide ? n4 : (n4 + 1) >> 1);
    if (lane == 0) {
        *ctl(C_HDR + wid) = ((end - rec + 3) >> 2) | (nb2 << 11) | (nb3 << 20) | (wide ? 1u << 29 : 0u);
        st8(L_CTL + 4 * (C_HDR + 8) + wid, base);
    }
}

__device__ __forceinline__ void classify4(u32 v0, u32 v1, u32 k, u32& g, u32& c) {
    const u32 g0 = pk_nmad2(v0, k), g1 = pk_nmad2(v1, k);
    g = perm(g1, g0, 0x07050301u);
    c = perm(pk_min(g1, 0x04000400u), pk_min(g0, 0x04000400u), 0x07050301u);
}

__device__ __forceinline__ void enc_input(const u32x4 (&v)[4], int wid, int lane) {
    const u32 rec = L_RB + wid * ENC_W, t2 = rec + RECB + 2 * TB;
    u32 sum = 0;
#pragma unroll
    for (int j = 0; j < 4; ++j) {
        const u32 l0 = perm(v[j].y, v[j].x, 0x06040200u), h0 = perm(v[j].y, v[j].x, 0x07050301u);
        const u32 l1 = perm(v[j].w, v[j].z, 0x06040200u), h1 = perm(v[j].w, v[j].z, 0x07050301u);
        st64(rec + j * 512 + lane * 8, bfi(0x80808080u, h0, l0), bfi(0x80808080u, h1, l1));
        sum = __builtin_amdgcn_udot4(h0 & 0x7f7f7f7fu, 0x01010101u, sum, false);
    }
    const u32 base = uni((u32)min(max(((int)last(scan(sum)) + 164) >> 9, 19), 250));
    const u32 k = (((base + 3) << 8) | 0xffu) * 0x00010001u;
    u32 g[4][2], pa = 0, pb = 0;
#pragma unroll
    for (int j = 0; j < 4; ++j) {
        u32 c0, c1;
        classify4(v[j].x, v[j].y, k, g[j][0], c0); classify4(v[j].z, v[j].w, k, g[j][1], c1);
        pa |= gather8(c0 & 0x01010101u, c1 & 0x01010101u) << (8 * j);
        const u32 b2 = gather8(c0 & 0x02020202u, c1 & 0x02020202u);
        pb |= j ? b2 << (8 * j - 1) : b2 >> 1;
    }
    st32(rec + 2048 + lane * 4, pa); st32(rec + 2304 + lane * 4, pb);
    const u32 more = ~(pa | pb);
    u32x2 sel[4];
#pragma unroll
    for (int j = 0; j < 4; ++j) sel[j] = ld64(L_LUTC + ((more >> (8 * j)) & 255u) * 8u);
    u32 o[4]; const u32 n2 = offsets4(more, o);
    for (u32 i = lane * 16; i < n2; i += 1024) { const u32x4 z = {0u, 0u, 0u, 0u}; st128(t2 + i, z); st128(t2 + TB + i, z); }
    CBAR();
#pragma unroll
    for (int j = 0; j < 4; ++j)
        scatter8(t2 + o[j], perm(g[j][1], g[j][0], sel[j].x), perm(g[j][1], g[j][0], sel[j].y), lane);
    CBAR();
    enc_l234(rec, rec + 2560, n2, base, t2, wid, lane);
}

__device__ __forceinline__ void enc_output1(const u32x4 (&v)[4], int wid, int lane, u32& n1, u32& base) {
    const u32 rec = L_RB + wid * ENC_W, t1 = rec + RECB, t2 = t1 + 2 * TB;
    u32x4 z[4];
    u32 mm[2] = {0u, 0u}, sum = 0;
#pragma unroll
    for (int j = 0; j < 4; ++j)
#pragma unroll
        for (int e = 0; e < 4; ++e) {
            const u32 w = 0x00020001u << (8 * (j & 1) + 2 * e);
            const u32 nz = pk_nz(v[j][e]);
            z[j][e] = nz;
            mm[j >> 1] = __builtin_amdgcn_udot2(__builtin_bit_cast(u16x2, nz), __builtin_bit_cast(u16x2, w), mm[j >> 1], false);
            sum = __builtin_amdgcn_sad_u16(v[j][e], 0u, sum);
        }
    const u32 m = mm[0] | (mm[1] << 16);
    st32(rec + lane * 4, m);
    u32 o[4]; n1 = offsets4(m, o);
    const u32 tot = last(scan(sum));
    for (u32 i = lane * 16; i < n1; i += 1024) { const u32x4 z0 = {0u, 0u, 0u, 0u}; st128(t2 + i, z0); st128(t2 + TB + i, z0); }
#pragma unroll
    for (int j = 0; j < 4; ++j) compact16(t1 + 2 * o[j], v[j], z[j]);
    base = uni((u32)min(max((int)((float)tot * __builtin_amdgcn_rcpf((float)max(n1, 1u)) * (1.0f / 128.0f) - 0.64f), 19), 250));
}
__device__ __forceinline__ void enc_output2(u32 n1, u32 base, int wid, int lane) {
    const u32 rec = L_RB + wid * ENC_W, t1 = rec + RECB, t2 = t1 + 2 * TB;
    const u32 k = (((base + 3) << 8) | 0xffu) * 0x00010001u;
    const u32 nb = (n1 + 7) >> 3, pbo = rec + 256 + n1;
    u32 run = 0, pbs = 0;
#pragma unroll
    for (int p = 0; p < 4; ++p) {
        if ((u32)p * 64u < nb) {
            const u32 idx = p * 64 + lane;
            const u32x4 x = ld128(t1 + idx * 16u);
            const int left = (int)n1 - (int)(idx * 8u);
            u32 g0, g1, c0, c1;
            classify4(x.x, x.y, k, g0, c0); classify4(x.z, x.w, k, g1, c1);
            const u32 a8 = gather8(c0 & 0x01010101u, c1 & 0x01010101u), b8 = gather8(c0 & 0x02020202u, c1 & 0x02020202u) >> 1;
            const u32 more = ~(a8 | b8) & valid8(left);
            const u32x2 sel = ld64(L_LUTC + more * 8u);
            if (left > 0) st64(rec + 256 + idx * 8u, bfi(0x80808080u, c0 << 7, perm(x.y, x.x, 0x06040200u)), bfi(0x80808080u, c1 << 7, perm(x.w, x.z, 0x06040200u)));
            pbs |= b8 << (8 * p);
            const u32 cnt = __popc(more), inc = scan(cnt), o2 = run + inc - cnt;
            run += last(inc);
            scatter8(t2 + o2, perm(g1, g0, sel.x), perm(g1, g0, sel.y), lane);
        }
    }
    CBAR();
#pragma unroll
    for (int p = 0; p < 4; ++p)
        if ((u32)(p * 64 + lane) < nb) st8(pbo + p * 64 + lane, pbs >> (8 * p));
    CBAR();
    enc_l234(rec, pbo + nb, run, base, t2, wid, lane);
}

__device__ __forceinline__ u32 record_ends(u32 hdr) {
    u32 inc = hdr & 0x7ffu;
    inc += __builtin_amdgcn_update_dpp(0u, inc, 0x111, 0xf, 0xf, false);
    inc += __builtin_amdgcn_update_dpp(0u, inc, 0x112, 0xf, 0xf, false);
    inc += __builtin_amdgcn_update_dpp(0u, inc, 0x114, 0xf, 0xf, false);
    return inc + 10u;
}
__device__ __forceinline__ void gst32_sys(void* p, u32 v) { asm volatile("global_store_dword %0, %1, off sc0 sc1" ::"v"(p), "v"(v) : "memory"); }

__device__ __forceinline__ u32 emit_block(char* slot, int wid, int lane) {
    const u32 rec = L_RB + wid * ENC_W;
    CBAR();
    const u32x4 r0 = ld128(rec + lane * 16), r1 = ld128(rec + 1024 + lane * 16), r2 = ld128(rec + 2048 + lane * 16);
    wg_barrier();
    const u32 hw = *ctl(C_HDR + (lane & 15));
    const u32 ends = record_ends(hw);
    const u32 mysz = __builtin_amdgcn_readlane(hw, wid) & 0x7ffu, myend = __builtin_amdgcn_readlane(ends, wid);
    const u32 total = __builtin_amdgcn_readlane(ends, 7), full = mysz >> 2;
    char* d = slot + (size_t)(myend - mysz) * 4;
    if ((u32)lane < full) gst_sys(d + lane * 16, r0);
    if ((u32)lane + 64u < full) gst_sys(d + 1024 + lane * 16, r1);
    if ((u32)lane + 128u < full) gst_sys(d + 2048 + lane * 16, r2);
    for (u32 i = 192 + lane; i < full; i += 64) gst_sys(d + (size_t)i * 16, ld128(rec + i * 16));
    if ((u32)lane < (mysz & 3u)) gst32_sys(d + full * 16 + lane * 4, ld32(rec + full * 16 + lane * 4));
    if (wid == 0 && lane < 10) gst32_sys(slot + lane * 4, hw);
    return (total + 3u) >> 2;
}

__device__ __forceinline__ void flag_store(char* wsl, int rank, int pos, u32 val) {
    __hip_atomic_store((GAS u32*)(wsl + OFF_FL) + (rank * 1024 + pos), val, __ATOMIC_RELAXED, __HIP_MEMORY_SCOPE_SYSTEM);
}
__device__ __forceinline__ void ready_add(char* ws, int idx, int local) {
    __hip_atomic_fetch_add((GAS u32*)(ws + OFF_RDY + idx * 128 + local * 64), 1u, __ATOMIC_RELAXED, __HIP_MEMORY_SCOPE_AGENT);
}
#define STAMP(i) do {} while (0)
#define STAMPW(i) do {} while (0)

__device__ __forceinline__ void build_tables(int t) {
    u64 ee = 0x0c0c0c0c0c0c0c0cull, cc = ee; int k = 0;
#pragma unroll
    for (int i = 0; i < 8; ++i)
        if ((t >> i) & 1) { ee ^= (u64)(0x0c ^ k) << (8 * i); cc ^= (u64)(0x0c ^ i) << (8 * k); ++k; }
    st64(L_LUTE + t * 8, (u32)ee, (u32)(ee >> 32)); st64(L_LUTC + t * 8, (u32)cc, (u32)(cc >> 32));
}
__device__ __forceinline__ u32 draw_call(char* ws) {
    return __hip_atomic_fetch_add((GAS u32*)(ws + OFF_CTR + blockIdx.x * 64), 1u, __ATOMIC_RELAXED, __HIP_MEMORY_SCOPE_AGENT) + 1u;
}

__device__ __forceinline__ bool pos_is_x(int pos) { return pos < 16 || (pos >= 48 && pos < 144) || pos >= 176; }
__device__ __forceinline__ int pos_xb(int pos) { return pos < 16 ? pos : (pos < 144 ? pos - 32 : pos - 64); }
__device__ __forceinline__ int pos_wb(int pos) { return pos < 48 ? pos - 16 : pos - 112; }
__device__ __forceinline__ void task_of(int task, int& pos, int& rel) {
    rel = -1;
    if (task < 16) pos = task;
    else if (task < 272) { pos = 16 + ((task - 16) >> 3); rel = (task + 1) & 7; }
    else if (task < 368) pos = 48 + task - 272;
    else if (task < 624) { pos = 144 + ((task - 368) >> 3); rel = (task + 1) & 7; }
    else pos = 176 + task - 624;
}
__device__ __forceinline__ void pack_load(const Params& P, int task, int wid, int lane, u32x4 (&t)[4]) {
    int pos, rel; task_of(task, pos, rel);
    const char* s; size_t s1, s2;
    if (rel < 0) {
        const int xb = pos_xb(pos);
        s = P.x + (size_t)((xb >> 3) * 256 + wid * 32 + (lane & 15)) * 1024 + (xb & 7) * 128 + (lane >> 4) * 16;
        s1 = 64; s2 = 16384;
    } else {
        const int wb = pos_wb(pos);
        s = P.w + (size_t)((wb & 3) * 128 + (wid >> 2) * 64 + (lane >> 2)) * (NFULL * 2) + (((P.rank + rel) & 7) * 2048 + (wb >> 2) * 128 + (wid & 3) * 32) * 2 + (lane & 3) * 16;
        s1 = 16 * NFULL * 2; s2 = 2 * s1;
    }
    t[0] = *(const GAS u32x4*)s; t[1] = *(const GAS u32x4*)(s + s1); t[2] = *(const GAS u32x4*)(s + s2); t[3] = *(const GAS u32x4*)(s + s1 + s2);
}
__device__ __forceinline__ void pack_task(const Params& P, char* ws, char* wsl, int task, int next, u32 call, int wid, int lane, u32x4 (&v)[4]) {
    int pos, rel; task_of(task, pos, rel);
    size_t lo, so; int rdy;
    if (rel < 0) {
        const int xb = pos_xb(pos);
        rdy = xb >> 3; lo = OFF_XA + (size_t)(rdy * 64 + (xb & 7)) * 32768; so = OFF_CX + (size_t)xb * SLOT;
    } else {
        const int wb = pos_wb(pos);
        rdy = 16 + (wb >> 2); lo = OFF_WB + (size_t)((wb >> 2) * 64 + (wb & 3) * 2) * 16384; so = OFF_CW + (size_t)(rel * 64 + wb) * SLOT;
        const u32 tr = L_RB + wid * ENC_W + RECB;
#pragma unroll
        for (int i = 0; i < 4; ++i) st128(tr + i * 1024 + lane * 16, v[i]);
        CBAR();
        const u32 ra = tr + (lane >> 4) * 512 + (((lane >> 2) & 3) * 8 + (lane & 3)) * 2;
#pragma unroll
        for (int j = 0; j < 4; ++j) {
            u32 q[8];
#pragma unroll
            for (int e = 0; e < 8; ++e) q[e] = *(LAS unsigned short*)lp(ra + (j & 1) * 2048 + e * 64 + (j >> 1) * 8);
            v[j] = (u32x4){q[0] | (q[1] << 16), q[2] | (q[3] << 16), q[4] | (q[5] << 16), q[6] | (q[7] << 16)};
        }
        CBAR();
    }
    const bool local = rel <= 0, packed = rel != 0 && pos >= NRAW;
    if (local) {
        char* d = ws + lo + wid * 4096 + lane * 16;
#pragma unroll
        for (int j = 0; j < 4; ++j) gst_agent(d + j * 1024, v[j]);
    }
    u32 len = 0;
    if (packed) { enc_input(v, wid, lane); len = emit_block(ws + so, wid, lane); }
    CBAR();
    pack_load(P, min(next, NTASK - 1), wid, lane, v);
    asm volatile("s_waitcnt vmcnt(4)" ::: "memory");
    wg_barrier();
    if (wid == 0) {
        if (packed && (rel < 0 ? lane >= 1 && lane < 8 : lane == rel)) flag_store(wsl, P.rank, pos, (call << 12) | len);
        if (local && lane == 0) ready_add(ws, rdy, 1);
    }
    JOIN();
}

__device__ __forceinline__ void tile_of(int set, int t16, int& pm, int& pn) {
    const bool first = set == 0 || set == 7;
    const int pair = set < 7 ? set : (set < 14 ? set - 7 : 7), half = set < 7 ? 0 : (set < 14 ? 1 : set & 1);
    pm = 2 * pair + (first ? t16 & 1 : t16 >> 3); pn = 8 * half + (first ? t16 >> 1 : t16 & 7);
}

__device__ __forceinline__ void decoder(const Params& P, char* ws, int slab, int d, int wid, int lane, u64 pc) {
    const int src = (P.rank + slab) & 7;
    char* sws = P.ws[slab & 7];
    const u64 tstart = now();
    GAS u32* tmo = (GAS u32*)(ws + OFF_TMO);
    if (wid == 1) {
        *ctl(lane) = 0u;
        wg_barrier();
        u32 call = 0;
        if (lane == 0) call = draw_call(ws);
        call = uni(call);
        *ctl(C_CALL) = call;
        const u32 seq = call & 0xfffffu;
        const GAS u32* fl = (const GAS u32*)(ws + OFF_FL) + src * 1024;
        bool dead = false;
        for (u32 i = 0;; ++i) {
            u32 pos = NRAW + d;
            if (i) {
                while (ctl_ld(C_PROG) < i) { if (now() - tstart > 2 * DEADLINE) break; __builtin_amdgcn_s_sleep(2); }
                u32 t = 0;
                if (lane == 0) t = __hip_atomic_fetch_add((GAS u32*)(ws + OFF_CLM + src * 128), 1u, __ATOMIC_RELAXED, __HIP_MEMORY_SCOPE_AGENT);
                pos = uni(t) - (call - 1u) * (u32)(NPOS - NRAW) + (u32)(NRAW + DPP_);
            }
            if (pos >= (u32)NPOS) { *ctl(C_QUEUE + (i & 7u)) = ((i + 1u) << 22) | (1023u << 12); break; }
            u32 f = 0;
            while (!dead) {
                f = uni(__hip_atomic_load(fl + pos, __ATOMIC_RELAXED, __HIP_MEMORY_SCOPE_SYSTEM));
                if ((f >> 12) == seq) break;
                if (uni(__hip_atomic_load(tmo, __ATOMIC_RELAXED, __HIP_MEMORY_SCOPE_AGENT)) == call || now() - tstart > DEADLINE) {
                    __hip_atomic_store(tmo, call, __ATOMIC_RELAXED, __HIP_MEMORY_SCOPE_AGENT);
                    *ctl(C_DEAD) = 1; dead = true; f = 0;
                }
                __builtin_amdgcn_s_sleep(1);
            }
            CBAR();
            *ctl(C_QUEUE + (i & 7u)) = ((i + 1u) << 22) | (pos << 12) | (f & 0xfffu);
        }
    } else if (wid == 0) {
        {
            const u32 o = (lane & 4) ? (u32)OFF_FL : (lane & 2) ? ((lane & 1) ? (u32)OFF_CW + (u32)(8 - slab) * (u32)(64 * SLOT) : (u32)OFF_CX) : (u32)OFF_XA + (u32)(lane & 1) * (2u << 20) + (u32)(8 - slab) * 262144u;
            const u32 w0 = __hip_atomic_load((const GAS u32*)(sws + o), __ATOMIC_RELAXED, __HIP_MEMORY_SCOPE_SYSTEM);
            const char* code = (const char*)(pc & ~127ull) + lane * 128;
            const u32 w1 = *(const GAS u32*)code, w2 = *(const GAS u32*)(code + 8192), w3 = *(const GAS u32*)(code + 16384);
            STAMP(0);
            wg_barrier();
            asm volatile("" ::"v"(w0), "v"(w1), "v"(w2), "v"(w3));
        }
        __builtin_amdgcn_s_setprio(3);
        u32 prev = 0, cur = 0;
#define PUBLISH(t) do { const u32 t_ = (t); if ((int)(t_ - cur) > 0) { cur = t_; *ctl(C_LANDED) = t_; } } while (0)
#define DRAIN_STEP(n) do { asm volatile("s_waitcnt vmcnt(%0)" ::"n"(n) : "memory"); if ((prev & 63u) >= (n)) PUBLISH(prev - (n)); } while (0)
#define DRAIN_PUBLISH() do { DRAIN_STEP(7); DRAIN_STEP(6); DRAIN_STEP(5); DRAIN_STEP(4); DRAIN_STEP(3); DRAIN_STEP(2); DRAIN_STEP(1); vm_drain(); } while (0)
        u32 i = 0, qn = 0, occ0 = 0, occ1 = 0, occ2 = 0, occ3 = 0;
#define BLOCK_DONE(b1) ((b1) == 0u || ctl_ld(C_DONE + (((b1) - 1u) & 7u)) >= (((b1) + 7u) & ~7u))
        for (;; ++i) {
            u32 f = ctl_ld(C_QUEUE + (i & 7u));
            if ((f >> 22) != i + 1u) {
                DRAIN_PUBLISH();
                PUBLISH(i << 6);
                while (((f = ctl_ld(C_QUEUE + (i & 7u))) >> 22) != i + 1u) {
                    if (now() - tstart > 2 * DEADLINE) { f = 1023u << 12; break; }
                    __builtin_amdgcn_s_sleep(1);
                }
            }
            CBAR();
            const int pos = (f >> 12) & 1023u;
            if (pos >= NPOS) break;
            const u32 len = min(f & 0xfffu, (u32)(LSLOT / 16));
            const bool big = len > (u32)(LSLOT / 32);
            const u32 q = big ? (qn + 1u) & 2u : qn;
            qn = (q + (big ? 2u : 1u)) & 3u;
            const u32 oa = q == 0 ? occ0 : q == 1 ? occ1 : q == 2 ? occ2 : occ3, ob = big ? (q == 0 ? occ1 : occ3) : 0u;
            const u32 oc = i >= 6u ? i - 5u : 0u;
            if (!(BLOCK_DONE(oa) && BLOCK_DONE(ob) && BLOCK_DONE(oc))) {
                DRAIN_PUBLISH();
                PUBLISH(i << 6);
                while (!(BLOCK_DONE(oa) && BLOCK_DONE(ob) && BLOCK_DONE(oc))) { if (now() - tstart > 2 * DEADLINE) break; __builtin_amdgcn_s_sleep(1); }
            }
            CBAR();
            occ0 = q == 0 ? i + 1u : occ0; occ1 = q == 1 || (big && q == 0) ? i + 1u : occ1;
            occ2 = q == 2 ? i + 1u : occ2; occ3 = q == 3 || (big && q == 2) ? i + 1u : occ3;
            *ctl(C_SLOT + (i & 7u)) = L_RB + q * (LSLOT / 2);
            *ctl(C_PROG) = i + 1u;
            size_t so;
            if (pos < NIN) so = pos_is_x(pos) ? OFF_CX + (size_t)pos_xb(pos) * SLOT : OFF_CW + (size_t)((8 - slab) * 64 + pos_wb(pos)) * SLOT;
            else so = OFF_CO + (size_t)(pos - NIN) * SLOT;
            const bool tail = pos >= NPOS - NTAIL;
            const char* s = sws + so + lane * 16;
            const u32 lb = (u32)(size_t)lp(L_RB + q * (LSLOT / 2));
#define PIECES(Q) for (u32 k = 0; k * 64u < len; ++k) { \
                if (k * 64u + lane < len) \
                    asm volatile("s_mov_b32 m0, %1\n\ts_waitcnt vmcnt(%2)\n\tglobal_load_lds_dwordx4 %0, off sc0 sc1" \
                                 ::"v"(s + (size_t)k * 1024), "s"(lb + k * 1024u), "n"((Q) - 1) : "memory"); \
                const int c = (int)k - ((Q) - 1); \
                if (c >= 0) PUBLISH((i << 6) + c); \
                else if ((int)(prev & 63u) + c >= 0) PUBLISH(prev + c); }
            if (tail) { PIECES(QT) } else { PIECES(QD) }
#undef PIECES
            prev = (i << 6) + ((len + 63u) >> 6);
        }
        DRAIN_PUBLISH();
        *ctl(C_NBLK) = i + 1u;
        *ctl(C_LANDED) = 0xffffffffu;
        STAMPW(50);
    } else {
        const u32 tmp = L_DT + (wid - 2) * 4 * TB;
        {
            const int g = d * (NRAW / 2) + min(wid - 2, NRAW / 2 - 1);
            u32x4 raw[4];
            pack_load(P, g >> 3, g & 7, lane, raw);
            if (wid < 6) build_tables(threadIdx.x - 128);
            wg_barrier();
            if (wid - 2 < NRAW / 2) {
                char* dst = sws + OFF_XA + (size_t)((g >> 6) * 64 + (8 - slab) * 8 + ((g >> 3) & 7)) * 32768 + (g & 7) * 4096 + lane * 16;
#pragma unroll
                for (int j = 0; j < 4; ++j) { gst_sys(dst + j * 1024, raw[j]); if (j == 0) STAMPW(1); asm volatile("s_waitcnt vmcnt(%0)" ::"n"(PUSHQ) : "memory"); }
                vm_drain();
                if (ctl_inc(C_PUSH) == NRAW / 2 - 1) {
                    u32 call;
                    while ((call = ctl_ld(C_CALL)) == 0u) { if (now() - tstart > 2 * DEADLINE) break; __builtin_amdgcn_s_sleep(1); }
                    if (lane == 0) __hip_atomic_store((GAS u32*)(sws + OFF_FLR) + (P.rank * 16 + d), call, __ATOMIC_RELAXED, __HIP_MEMORY_SCOPE_SYSTEM);
                    JOIN();
                    STAMPW(52);
                }
            }
        }
        for (;;) {
            const u32 r = ctl_inc(C_CLAIM);
            const u32 i = r >> 3, sub = r & 7u;
            bool late = false;
            while (ctl_ld(C_LANDED) < (i << 6) + 1u) { if (now() - tstart > 2 * DEADLINE) { late = true; break; } __builtin_amdgcn_s_sleep(1); }
            CBAR();
            if (late || ctl_ld(C_NBLK) - 1u <= i) break;
            const int pos = min((ctl_ld(C_QUEUE + (i & 7u)) >> 12) & 1023u, (u32)(NPOS - 1));
            const u32 slot = ctl_ld(C_SLOT + (i & 7u));
            const u32 hw = ld32(slot + (lane & 7) * 4), base = uni(ld8(slot + 32 + sub));
            const u32 hdr = __builtin_amdgcn_readlane(hw, sub), en = __builtin_amdgcn_readlane(record_ends(hw), sub);
            const u32 endp = min((en + 255u) >> 8, 63u);
            while (ctl_ld(C_LANDED) < (i << 6) + endp) { if (now() - tstart > 2 * DEADLINE) break; __builtin_amdgcn_s_sleep(1); }
            CBAR();
            const bool dead = ctl_ld(C_DEAD) != 0;
            const u32 rec = slot + min(en - (hdr & 0x7ffu), (u32)(LSLOT / 4)) * 4u;
            if (pos < NIN) {
                size_t o; int rdy;
                if (pos_is_x(pos)) { const int xb = pos_xb(pos); rdy = xb >> 3; o = OFF_XA + (size_t)(rdy * 64 + slab * 8 + (xb & 7)) * 32768; }
                else { const int wb = pos_wb(pos); rdy = 16 + (wb >> 2); o = OFF_WB + (size_t)((wb >> 2) * 64 + slab * 8 + (wb & 3) * 2) * 16384; }
                if (!dead) dec_input(rec, hdr, base, tmp, ws + o + sub * 4096 + lane * 16, lane);
                ctl_add1(C_DONE + (i & 7u));
                vm_drain();
                if ((ctl_inc(C_STORED + (i & 15u)) & 7u) == 7u) { if (lane == 0) ready_add(ws, rdy, 0); JOIN(); }
            } else {
                const int ob = pos - NIN;
                int pm, pn; tile_of(ob >> 5, ob & 15, pm, pn);
                char* dst = P.out + ((size_t)(pm * 256 + (sub >> 1) * 64 + ((ob >> 4) & 1) * 32 + (lane >> 3)) * NFULL + src * 2048 + pn * 128 + (sub & 1) * 64 + (lane & 7) * 8) * 2;
                if (!dead) dec_output(rec, hdr, base, tmp, dst, C_DONE + (i & 7u), lane);
                else { gst_agent(dst, (u32x4){0x7fc07fc0u, 0x7fc07fc0u, 0x7fc07fc0u, 0x7fc07fc0u}); ctl_add1(C_DONE + (i & 7u)); }
            }
        }
    }
}

__device__ __forceinline__ bool wait_operands(char* ws, int rank, int pm, int pn, u32 call, u32 mask, int lane) {
    const GAS u32* a = (const GAS u32*)(lane < 4 ? ws + OFF_RDY + ((lane & 1) ? 16 + pn : pm) * 128 + (lane < 2 ? 64 : 0) : ws + OFF_TMO);
    const int pushed = min(max(NRAW - 8 * pm, 0), 8);
    const u32 need = call * (lane == 0 ? 8u : lane == 1 ? 4u : lane == 2 ? 56u - 7u * pushed : 28u);
    const u64 t0 = now();
    u32 ok;
    for (;;) {
        const u32 v = __hip_atomic_load(a, __ATOMIC_RELAXED, __HIP_MEMORY_SCOPE_AGENT);
        ok = (u32)__ballot((int)(v - need) >= 0) & 15u;
        if (pushed) {
            const u64 f = __hip_atomic_load((const GAS u64*)(ws + OFF_FLR) + lane, __ATOMIC_RELAXED, __HIP_MEMORY_SCOPE_SYSTEM);
            if (!__all((lane >> 3) == rank || ((int)((u32)f - call) >= 0 && (int)((u32)(f >> 32) - call) >= 0))) ok &= ~4u;
        }
        const bool dead = __builtin_amdgcn_readlane(v, 4) == call;
        if (!dead && (ok & mask) == mask) break;
        if (dead || now() - t0 > DEADLINE) {
            __hip_atomic_store((GAS u32*)(ws + OFF_TMO), call, __ATOMIC_RELAXED, __HIP_MEMORY_SCOPE_AGENT);
            *ctl(C_DEAD) = 1; ok = 15u;
            break;
        }
        __builtin_amdgcn_s_sleep(4);
    }
    if (pushed) __builtin_amdgcn_fence(__ATOMIC_ACQUIRE, ""); else __builtin_amdgcn_fence(__ATOMIC_ACQUIRE, "agent");
    vm_drain();
    return ok == 15u;
}

__device__ __forceinline__ void gemm_tile(const Params& P, char* ws, char* wsl, int set, int t16, u32 call, int wid, int lane) {
    int pm, pn; tile_of(set, t16, pm, pn);
    bool have_all = true;
    if (wid == 0) have_all = wait_operands(ws, P.rank, pm, pn, call, 3u, lane);
    wg_barrier();
    STAMP(8 + (set >> 3) * 4);
    const char* ga = ws + OFF_XA + (size_t)pm * (2u << 20);
    const char* gb = ws + OFF_WB + (size_t)pn * (1u << 20);
    const u32 voa = wid * 4096 + lane * 16, vob = wid * 2048 + lane * 16;
    const u32 lbase = (u32)(size_t)lp(L_RB);
    const u32 lda = lbase + wid * 4096, ldb = lbase + 32768 + wid * 2048;
#define GLDS(vo, gp, lo, off) do { u32 keep_; asm volatile( \
        "s_mov_b32 %0, m0\n\ts_mov_b32 m0, %3\n\ts_nop 3\n\tglobal_load_lds_dwordx4 %1, %2 offset:" #off "\n\ts_mov_b32 m0, %0" \
        : "=&s"(keep_) : "v"(vo), "s"(gp), "s"(lo) : "memory"); } while (0)
#define GLDF(vo, gp, lo, off) asm volatile("s_mov_b32 m0, %2\n\ts_nop 0\n\tglobal_load_lds_dwordx4 %0, %1 offset:" #off ::"v"(vo), "s"(gp), "s"(lo) : "memory")
#define KA(kt) (ga + (size_t)(kt) * 32768)
#define KB(kt) (gb + (size_t)(kt) * 16384)
#define STAGE_LD(so, kt) do { const char* a_ = KA(kt); const char* b_ = KB(kt); \
        GLDS(voa, a_, lda + (so), 0); GLDS(voa, a_, lda + (so), 1024); GLDS(voa, a_, lda + (so), 2048); GLDS(voa, a_, lda + (so), 3072); \
        GLDS(vob, b_, ldb + (so), 0); GLDS(vob, b_, ldb + (so), 1024); } while (0)
    const u32 ra = L_RB + (wid >> 1) * 8192 + lane * 16, rb = L_RB + 32768 + (wid & 1) * 8192 + lane * 16;
#define FRAGS(A, B, so, kb) do { _Pragma("unroll") for (int q_ = 0; q_ < 4; ++q_) { \
        B[q_] = *(const LAS bf16x8*)lp(rb + (so) + q_ * 2048 + (kb) * 1024); A[q_] = *(const LAS bf16x8*)lp(ra + (so) + q_ * 2048 + (kb) * 1024); } } while (0)
#define MROW(A, B, m_) do { _Pragma("unroll") for (int n_ = 0; n_ < 4; ++n_) \
        acc[m_][n_] = __builtin_amdgcn_mfma_f32_16x16x32_bf16(B[n_], A[m_], acc[m_][n_], 0, 0, 0); } while (0)
#define SB() __builtin_amdgcn_sched_barrier(0)
#define MM1(A, B, m_, n_) acc[m_][n_] = __builtin_amdgcn_mfma_f32_16x16x32_bf16(B[n_], A[m_], acc[m_][n_], 0, 0, 0)
#define RDB(B, q_, kb) B[q_] = *(const LAS bf16x8*)lp(rb + so + (q_) * 2048 + (kb) * 1024)
#define RDA(A, q_, kb) A[q_] = *(const LAS bf16x8*)lp(ra + so + (q_) * 2048 + (kb) * 1024)
#define HALF(A, B, NA, NB, kb, G0, G1, G2) do { SB(); \
        MM1(A, B, 0, 0); SB(); RDA(NA, 0, kb); SB(); MM1(A, B, 0, 1); SB(); RDB(NB, 0, kb); SB(); MM1(A, B, 0, 2); SB(); RDB(NB, 1, kb); SB(); MM1(A, B, 0, 3); SB(); RDB(NB, 2, kb); SB(); \
        MM1(A, B, 1, 0); SB(); RDB(NB, 3, kb); SB(); MM1(A, B, 1, 1); SB(); RDA(NA, 1, kb); SB(); MM1(A, B, 1, 2); SB(); RDA(NA, 2, kb); SB(); MM1(A, B, 1, 3); SB(); RDA(NA, 3, kb); SB(); \
        MM1(A, B, 2, 0); SB(); G0; SB(); MM1(A, B, 2, 1); MM1(A, B, 2, 2); SB(); G1; SB(); MM1(A, B, 2, 3); \
        MM1(A, B, 3, 0); SB(); G2; SB(); MM1(A, B, 3, 1); MM1(A, B, 3, 2); MM1(A, B, 3, 3); SB(); } while (0)
#define MMA(A, B) do { MROW(A, B, 0); MROW(A, B, 1); MROW(A, B, 2); MROW(A, B, 3); } while (0)
    f32x4 acc[4][4];
#pragma unroll
    for (int m = 0; m < 4; ++m)
#pragma unroll
        for (int n = 0; n < 4; ++n) acc[m][n] = (f32x4){0.f, 0.f, 0.f, 0.f};
    bf16x8 a0[4], b0[4], a1[4], b1[4];
    STAGE_LD(0, 0); STAGE_LD(STAGE, 1);
    { const char* a_ = KA(2); GLDS(voa, a_, lda + 2 * STAGE, 0); GLDS(voa, a_, lda + 2 * STAGE, 1024); GLDS(voa, a_, lda + 2 * STAGE, 2048); }
    asm volatile("s_waitcnt vmcnt(9)\n\ts_barrier" ::: "memory");
    FRAGS(a0, b0, 0u, 0);
    u32 so = 0, sn, sp = 2 * STAGE;
    const char* pa = KA(2); const char* pb = KB(2);
#pragma unroll 1
    for (int phase = 0; phase < 2; ++phase) {
#pragma unroll 1
        for (int n = phase ? 56 : 5; n > 0; --n) {
            HALF(a0, b0, a1, b1, 1, GLDF(voa, pa, lda + sp, 3072), GLDF(vob, pb, ldb + sp, 0),
                 GLDF(vob, pb, ldb + sp, 1024); pa += 32768; pb += 16384; sp = so; sn = so == 2 * STAGE ? 0 : so + STAGE; asm volatile("" : "+s"(pa), "+s"(pb), "+s"(sn)));
            asm volatile("s_waitcnt vmcnt(6) lgkmcnt(0)\n\ts_barrier" ::: "memory");
            so = sn;
            HALF(a1, b1, a0, b0, 0, GLDF(voa, pa, lda + sp, 0), GLDF(voa, pa, lda + sp, 1024), GLDF(voa, pa, lda + sp, 2048));
        }
        if (phase == 0 && !have_all) { wait_operands(ws, P.rank, pm, pn, call, 15u, lane); STAMP(11 + (set >> 3) * 4); }
    }
    {
        const char* a2 = KA(63); const char* b2 = KB(63);
        GLDS(voa, a2, lda + sp, 3072); GLDS(vob, b2, ldb + sp, 0); GLDS(vob, b2, ldb + sp, 1024);
        FRAGS(a1, b1, so, 1); MMA(a0, b0);
        asm volatile("s_waitcnt vmcnt(6) lgkmcnt(0)\n\ts_barrier" ::: "memory");
        so = so == 2 * STAGE ? 0 : so + STAGE;
        FRAGS(a0, b0, so, 0); MMA(a1, b1);
        FRAGS(a1, b1, so, 1); MMA(a0, b0);
        asm volatile("s_waitcnt vmcnt(0) lgkmcnt(0)\n\ts_barrier" ::: "memory");
        so = so == 2 * STAGE ? 0 : so + STAGE;
        FRAGS(a0, b0, so, 0); MMA(a1, b1);
        FRAGS(a1, b1, so, 1); MMA(a0, b0); MMA(a1, b1);
    }
#undef GLDS
#undef KA
#undef KB
#undef STAGE_LD
#undef FRAGS
#undef MROW
#undef MMA
#undef MM1
#undef GLDF
#undef RDA
#undef RDB
#undef HALF
#undef SB
    wg_barrier();
    STAMP(9 + (set >> 3) * 4);
    const float floor0 = ctl_ld(C_DEAD) != 0 ? __builtin_nanf("") : 0.f;
    char* dst = P.out + ((size_t)(pm * 256 + (wid >> 1) * 64 + (lane >> 3)) * NFULL + P.rank * 2048 + pn * 128 + (wid & 1) * 64 + (lane & 7) * 8) * 2;
    const u32 img = L_RB + wid * ENC_W + RECB;
    u32 len = 0;
#pragma unroll
    for (int b = 0; b < 2; ++b) {
        u32x4 v[4];
#pragma unroll
        for (int j = 0; j < 4; ++j) {
            const f32x4 p = acc[2 * b + (j >> 1)][2 * (j & 1)], q = acc[2 * b + (j >> 1)][2 * (j & 1) + 1];
            float f[8] = {p[0], p[1], p[2], p[3], q[0], q[1], q[2], q[3]};
            u32 r[4];
#pragma unroll
            for (int e = 0; e < 4; ++e) {
                const float lo = __builtin_elementwise_maximum(f[2 * e], floor0), hi = __builtin_elementwise_maximum(f[2 * e + 1], floor0);
                asm("v_cvt_pk_bf16_f32 %0, %1, %2" : "=v"(r[e]) : "v"(lo), "v"(hi));
                r[e] &= 0x7fff7fffu;
            }
            st128(img + (j >> 1) * 2048 + (lane & 15) * 128 + ((((j & 1) * 4 + (lane >> 4)) ^ (lane & 7)) << 4), (u32x4){r[0], r[1], r[2], r[3]});
        }
        CBAR();
#pragma unroll
        for (int j = 0; j < 4; ++j) v[j] = ld128(img + j * 1024 + (lane >> 3) * 128 + (((lane ^ (lane >> 3)) & 7) << 4));
        CBAR();
        u32 n1, base;
        enc_output1(v, wid, lane, n1, base);
        if (b == 1) {
            asm volatile("s_waitcnt vmcnt(4)" ::: "memory");
            wg_barrier();
            if (threadIdx.x >= 1 && threadIdx.x < 8) flag_store(wsl, P.rank, NIN + set * 32 + t16, (call << 12) | len);
            JOIN();
        }
        enc_output2(n1, base, wid, lane);
        len = emit_block(ws + OFF_CO + (size_t)(set * 32 + b * 16 + t16) * SLOT, wid, lane);
#pragma unroll
        for (int j = 0; j < 4; ++j) gst_agent(dst + (size_t)(4 * b + j) * (8 * NFULL * 2), v[j]);
    }
    asm volatile("s_waitcnt vmcnt(4)" ::: "memory");
    wg_barrier();
    if (threadIdx.x >= 1 && threadIdx.x < 8) flag_store(wsl, P.rank, NIN + set * 32 + 16 + t16, (call << 12) | len);
    JOIN();
    STAMP(10 + (set >> 3) * 4);
}

__global__ void __launch_bounds__(512, 2) fused_gemm_ar(const Params P) {
    const int tid = threadIdx.x, lane = tid & 63, wid = uni(tid >> 6), blk = blockIdx.x;
    const int xcd = blk & 7, idx = blk >> 3;
    u64 pc; asm volatile("s_getpc_b64 %0" : "=s"(pc));
    char* ws = P.ws[0];
    if (blk < NDEC) {
        decoder(P, ws, 1 + (idx >= 7 ? idx - 7 : idx), (idx >= 7 ? 8 : 0) + xcd, wid, lane, pc);
        return;
    }
    u32x4 nx[4];
    pack_load(P, blk - NDEC, wid, lane, nx);
    u32 cv;
    {
        u64 sv;
        asm volatile("s_mov_b64 %1, exec\n\ts_cmp_eq_u32 %4, 0\n\ts_cselect_b64 exec, 1, 0\n\tglobal_atomic_add %0, %2, %3, off sc0\n\ts_mov_b64 exec, %1"
                     : "=&v"(cv), "=&s"(sv) : "v"(ws + OFF_CTR + blk * 64), "v"(1u), "s"(wid) : "memory", "scc");
    }
    STAMP(0);
    if (wid < 4) build_tables(tid);
    else if (wid == 4) *ctl(lane) = 0u;
    u32 wlo = 0, whi = 0;
#pragma unroll
    for (int p = 1; p < NR; ++p) {
        asm volatile("v_writelane_b32 %0, %2, %4\n\tv_writelane_b32 %1, %3, %4" : "+v"(wlo), "+v"(whi) : "s"((u32)(u64)P.ws[p]), "s"((u32)((u64)P.ws[p] >> 32)), "n"(p));
    }
    char* wsl = (char*)(((u64)whi << 32) | wlo);
    wg_barrier();
    u32 call = 0;
    const int late = idx >= 30 ? 96 + blk - 240 : (xcd >= 2 ? (idx - 14) * 6 + xcd - 2 : NTASK);
    for (int task = blk - NDEC; task < NTASK;) {
        const int next = task < 288 ? task + 144 : (task < 432 ? 432 + late : task + 112);
        asm volatile("s_waitcnt vmcnt(0)" : "+v"(cv), "+v"(nx[0]), "+v"(nx[1]), "+v"(nx[2]), "+v"(nx[3])::"memory");
        call = uni(cv) + 1u;
        pack_task(P, ws, wsl, task, next, call, wid, lane, nx);
        task = next;
    }
    asm volatile("s_waitcnt vmcnt(0)" : "+v"(nx[0]), "+v"(nx[1]), "+v"(nx[2]), "+v"(nx[3])::"memory");
    STAMP(1);
    if (idx < 30)
        for (int rr = 0; rr < 2; ++rr) gemm_tile(P, ws, wsl, xcd + 8 * rr, idx - 14, call, wid, lane);
}

}

extern "C" void kernel_launch(int rank, int nranks, ncclComm_t comm, hipStream_t stream,
                              void* const* d_in, const int* in_sizes, int n_in,
                              void* d_out, int out_size, const int* in_dtypes, int out_dtype,
                              void* const* d_out_all, void* const* d_ws_all, size_t ws_bytes) {
    if (nranks != NR || (unsigned)rank >= (unsigned)NR || n_in != 2 || ws_bytes < WS_NEED ||
        in_sizes[0] != 4096 * 512 || in_sizes[1] != 512 * NFULL || out_size != 4096 * NFULL) {
        fprintf(stderr, "kernel_launch: unexpected problem shape\n");
        return;
    }
    Params P;
    P.x = (const char*)d_in[0]; P.w = (const char*)d_in[1]; P.out = (char*)d_out;
    for (int p = 0; p < NR; ++p) P.ws[p] = (char*)d_ws_all[(rank + p) % NR];
    P.rank = rank; P.pad = 0;
    hipLaunchKernelGGL(fused_gemm_ar, dim3(NWG), dim3(512), 0, stream, P);
    hipError_t e = hipGetLastError();
    if (e != hipSuccess) fprintf(stderr, "launch failed: %s\n", hipGetErrorString(e));
}
```

```cpp
#include <hip/hip_runtime.h>
#include <rccl/rccl.h>
#include <cstdio>

namespace {

typedef unsigned u32;
typedef unsigned long long u64;
typedef u32 u32x4 __attribute__((ext_vector_type(4)));
typedef u32 u32x2 __attribute__((ext_vector_type(2)));
typedef short bf16x8 __attribute__((ext_vector_type(8)));
typedef float f32x4 __attribute__((ext_vector_type(4)));
#define LAS __attribute__((address_space(3)))
#define GAS __attribute__((address_space(1)))

constexpr int NR = 8;
constexpr int NFULL = 16384;
constexpr int NWG = 256;
constexpr int DPP_ = 16;
constexpr int NIN = 192, NOUT = 512;
#define LS 1
#define NSB 16
constexpr int PSPLIT = NIN + NOUT - NSB;
constexpr int NPOS = PSPLIT + (NSB << LS);
constexpr int NDEC = 7 * DPP_;
constexpr int NTASK = 640;
#define QD 3
#define QT 8
#define NTAIL 0
#define PW 4
#define PS 4
constexpr int NRAW = PW * PS / 2;
static_assert(PW <= 6 && PS <= 4 && (PW * PS) % 2 == 0 && NRAW <= 8, "push shape");
constexpr int NSOLO = (NRAW + 16 + 7) & ~7;
#define WDELAY 200
static_assert(NSOLO <= 32, "solo flags are for x blocks");
constexpr u64 DEADLINE = 50000000ull;

constexpr size_t OFF_CTR = 0;
constexpr size_t OFF_TMO = 16384;
constexpr size_t OFF_RDY = 16384 + 256;
constexpr size_t OFF_CLM = 24576;
constexpr size_t OFF_FLR = 28672;
constexpr size_t OFF_FL = 32768;
constexpr size_t OFF_LEN = 65536;
constexpr size_t OFF_GRP = 98304;
constexpr size_t SLOT = 46080;
constexpr size_t OFF_CX = 262144;
constexpr size_t OFF_CW = OFF_CX + 128 * SLOT;
constexpr size_t OFF_CO = OFF_CW + 512 * SLOT;
constexpr size_t OFF_XA = 52u << 20;
constexpr size_t OFF_WB = OFF_XA + (32u << 20);
constexpr size_t OFF_DBG = OFF_WB + (16u << 20);
constexpr size_t WS_NEED = OFF_DBG + (1u << 20);
static_assert(OFF_CO + 512 * SLOT <= OFF_XA, "workspace");

constexpr int L_LUTE = 0;
constexpr int L_LUTC = 2048;
constexpr int L_CTL = 4096;
constexpr int L_RB = 4608;
constexpr int STAGE = 49152;
constexpr int LDS_BYTES = L_RB + 3 * STAGE;
constexpr int TB = 2112;
constexpr int RECB = 5696;
constexpr int ENC_W = RECB + 4 * TB;
constexpr int LSLOT = 46080;
constexpr int L_DT = L_RB + 2 * LSLOT;
static_assert(L_DT + 6 * 4 * TB <= LDS_BYTES, "lds");
static_assert(8 * ENC_W <= 3 * STAGE, "lds");
constexpr int C_HDR = 0  , C_CALL = 16, C_DEAD = 17, C_LANDED = 18, C_CLAIM = 19,
              C_NBLK = 22, C_PROG = 23, C_STORED = 24  , C_QUEUE = 40  , C_PUSH = 21, C_DONE = 48  , C_SLOT = 56  ;

struct Params {
    const char* x; const char* w; char* out;
    char* ws[NR];
    int rank; int pad;
};

__shared__ __attribute__((aligned(16))) unsigned char smem[LDS_BYTES];

#define CBAR() asm volatile("" ::: "memory")
__device__ __forceinline__ LAS unsigned char* lp(u32 o) { return (LAS unsigned char*)smem + o; }
__device__ __forceinline__ u32 ld8(u32 o) { return *lp(o); }
__device__ __forceinline__ u32 ldv8(u32 o) { return *(volatile LAS unsigned char*)lp(o); }
__device__ __forceinline__ u32 ld32(u32 o) { return *(LAS u32*)lp(o); }
__device__ __forceinline__ u32x2 ld64(u32 o) { return *(LAS u32x2*)lp(o); }
__device__ __forceinline__ u32x4 ld128(u32 o) { return *(LAS u32x4*)lp(o); }
__device__ __forceinline__ void st8(u32 o, u32 v) { *lp(o) = (unsigned char)v; }
__device__ __forceinline__ void st32(u32 o, u32 v) { *(LAS u32*)lp(o) = v; }
__device__ __forceinline__ void st64(u32 o, u32 a, u32 b) { *(LAS u32x2*)lp(o) = (u32x2){a, b}; }
__device__ __forceinline__ void st128(u32 o, u32x4 v) { *(LAS u32x4*)lp(o) = v; }
__device__ __forceinline__ void or32(u32 o, u32 v) { __hip_atomic_fetch_or((LAS u32*)lp(o), v, __ATOMIC_RELAXED, __HIP_MEMORY_SCOPE_WORKGROUP); }
__device__ __forceinline__ volatile LAS u32* ctl(int i) { return (volatile LAS u32*)lp(L_CTL + 4 * i); }
__device__ __forceinline__ u32 uni(u32 v) { return __builtin_amdgcn_readfirstlane(v); }
__device__ __forceinline__ u32 ctl_ld(int i) { return uni(*ctl(i)); }
__device__ __forceinline__ u32 ctl_inc(int i) {
    u32 r; u64 sv;
    asm volatile("s_mov_b64 %1, exec\n\ts_mov_b64 exec, 1\n\tds_add_rtn_u32 %0, %2, %3\n\ts_waitcnt lgkmcnt(0)\n\ts_mov_b64 exec, %1"
                 : "=&v"(r), "=&s"(sv) : "v"((u32)(size_t)lp(L_CTL + 4 * i)), "v"(1u) : "memory");
    return uni(r);
}
__device__ __forceinline__ void ctl_add1(int i) {
    u64 sv;
    asm volatile("s_mov_b64 %0, exec\n\ts_mov_b64 exec, 1\n\tds_add_u32 %1, %2\n\ts_mov_b64 exec, %0"
                 : "=&s"(sv) : "v"((u32)(size_t)lp(L_CTL + 4 * i)), "v"(1u) : "memory");
}
#define JOIN() __builtin_amdgcn_wave_barrier()
__device__ __forceinline__ u32 perm(u32 hi, u32 lo, u32 sel) { return __builtin_amdgcn_perm(hi, lo, sel); }
__device__ __forceinline__ u32 bfi(u32 m, u32 a, u32 b) { return (m & a) | (~m & b); }
__device__ __forceinline__ u64 now() { return __builtin_amdgcn_s_memrealtime(); }
__device__ __forceinline__ void wg_barrier() { asm volatile("s_waitcnt lgkmcnt(0)\n\ts_barrier" ::: "memory"); }
__device__ __forceinline__ void vm_drain() { asm volatile("s_waitcnt vmcnt(0)" ::: "memory"); }

__device__ __forceinline__ void gst_agent(void* p, u32x4 v) { asm volatile("global_store_dwordx4 %0, %1, off sc1\n\ts_nop 1" ::"v"(p), "v"(v) : "memory"); }
__device__ __forceinline__ void gst_sys(void* p, u32x4 v) { asm volatile("global_store_dwordx4 %0, %1, off sc0 sc1\n\ts_nop 1" ::"v"(p), "v"(v) : "memory"); }

__device__ __forceinline__ u32 scan(u32 x) {
    x += __builtin_amdgcn_update_dpp(0u, x, 0x111, 0xf, 0xf, false);
    x += __builtin_amdgcn_update_dpp(0u, x, 0x112, 0xf, 0xf, false);
    x += __builtin_amdgcn_update_dpp(0u, x, 0x114, 0xf, 0xf, false);
    x += __builtin_amdgcn_update_dpp(0u, x, 0x118, 0xf, 0xf, false);
    x += __builtin_amdgcn_update_dpp(0u, x, 0x142, 0xa, 0xf, false);
    x += __builtin_amdgcn_update_dpp(0u, x, 0x143, 0xc, 0xf, false);
    return x;
}
__device__ __forceinline__ u32 last(u32 x) { return __builtin_amdgcn_readlane(x, 63); }
__device__ __forceinline__ u32 offsets4(u32 m, u32 (&o)[4]) {
    const u32 c01 = __popc(m & 0xffu) | (__popc(m & 0xff00u) << 16), c23 = __popc(m & 0xff0000u) | (__popc(m >> 24) << 16);
    const u32 s01 = scan(c01), s23 = scan(c23);
    const u32 t01 = last(s01), t23 = last(s23);
    const u32 e01 = s01 - c01, e23 = s23 - c23;
    const u32 b1 = t01 & 0xffffu, b2 = b1 + (t01 >> 16), b3 = b2 + (t23 & 0xffffu);
    o[0] = e01 & 0xffffu; o[1] = (e01 >> 16) + b1; o[2] = (e23 & 0xffffu) + b2; o[3] = (e23 >> 16) + b3;
    return b3 + (t23 >> 16);
}
__device__ __forceinline__ u32 gather8(u32 y0, u32 y1) {
    return __builtin_amdgcn_udot4(y1, 0x80402010u, __builtin_amdgcn_udot4(y0, 0x08040201u, 0u, false), false);
}
__device__ __forceinline__ u32 spread1(u32 n4) { return __umul24(n4, 0x204081u) & 0x01010101u; }
__device__ __forceinline__ u32 spread2(u32 n4) { return __umul24(n4, 0x408102u) & 0x02020202u; }
__device__ __forceinline__ u32 subb(u32 a, u32 brep) {
    return ((a | 0x80808080u) - (brep & 0x7f7f7f7fu)) ^ ((a ^ ~brep) & 0x80808080u);
}
template <int K> __device__ __forceinline__ u32 geb(u32 d) {
    return ((((d & 0x7f7f7f7fu) + (u32)(0x80 - K) * 0x01010101u) | d) >> 7) & 0x01010101u;
}
__device__ __forceinline__ void fetch8(u32 a, u32& s0, u32& s1) {
    const u32 q = a & ~3u;
    const u32 d0 = ld32(q), d1 = ld32(q + 4), d2 = ld32(q + 8);
    s0 = __builtin_amdgcn_alignbyte(d1, d0, a); s1 = __builtin_amdgcn_alignbyte(d2, d1, a);
}
__device__ __forceinline__ void or32nz(u32 o, u32 v, int lane) { or32(v ? o : (u32)lane * 4u, v); }
__device__ __forceinline__ void scatter8(u32 a, u32 c0, u32 c1, int lane) {
    const u32 q = a & ~3u, sel = 0x07060504u - perm(0u, a & 3u, 0u);
    or32nz(q, perm(c0, 0u, sel), lane); or32nz(q + 4, perm(c1, c0, sel), lane); or32nz(q + 8, perm(0u, c1, sel), lane);
}
constexpr u32 JUNK = 4352;
__device__ __forceinline__ void compact16(u32 a, u32x4 v, u32x4 z) {
    u32 ad = (u32)(size_t)lp(a - JUNK), t0, t1;
#define C16(d, f) "v_mad_u32_u16 %1, " f ", %11, %0\n\tv_mad_u32_u16 %0, " f ", 2, %0\n\tv_mad_u32_u16 %2, " f ", %11, %0 op_sel:[1,0,0,0]\n\t" \
                  "ds_write_b16 %1, " d "\n\tv_mad_u32_u16 %0, " f ", 2, %0 op_sel:[1,0,0,0]\n\tds_write_b16_d16_hi %2, " d "\n\t"
    asm volatile(C16("%3", "%7") C16("%4", "%8") C16("%5", "%9") C16("%6", "%10")
                 : "+v"(ad), "=&v"(t0), "=&v"(t1) : "v"(v.x), "v"(v.y), "v"(v.z), "v"(v.w), "v"(z.x), "v"(z.y), "v"(z.z), "v"(z.w), "s"(JUNK) : "memory");
#undef C16
}
__device__ __forceinline__ void zero_lds(u32 a, u32 bytes, int lane) {
    for (u32 i = lane * 16; i < bytes; i += 1024) st128(a + i, (u32x4){0u, 0u, 0u, 0u});
}
typedef unsigned short u16x2 __attribute__((ext_vector_type(2)));
__device__ __forceinline__ u32 pk_nmad2(u32 v, u32 k) {
    u32 r; asm("v_pk_mad_u16 %0, %1, -2, %2 op_sel_hi:[1,0,1]" : "=v"(r) : "v"(v), "s"(k)); return r;
}
__device__ __forceinline__ u32 pk_nz(u32 v) {
    u32 r; asm("v_pk_min_u16 %0, %1, 1 op_sel_hi:[1,0]" : "=v"(r) : "v"(v)); return r;
}
__device__ __forceinline__ u32 pk_min(u32 v, u32 k) {
    return __builtin_bit_cast(u32, __builtin_elementwise_min(__builtin_bit_cast(u16x2, v), __builtin_bit_cast(u16x2, k)));
}
__device__ __forceinline__ u32 valid8(int left) { return left >= 8 ? 255u : (left > 0 ? (1u << left) - 1u : 0u); }

#define SB() __builtin_amdgcn_sched_barrier(0)
struct Fetch {
    u32 a, d0, d1, d2;
    __device__ __forceinline__ void ask(u32 a_) { a = a_; const u32 q = a_ & ~3u; d0 = ld32(q); d1 = ld32(q + 4); d2 = ld32(q + 8); }
    __device__ __forceinline__ u32 lo() const { return __builtin_amdgcn_alignbyte(d1, d0, a); }
    __device__ __forceinline__ u32 hi() const { return __builtin_amdgcn_alignbyte(d2, d1, a); }
};
__device__ __forceinline__ void fetch_l4(u32 l4, u32 o, u32 top, bool wide, u32& s0, u32& s1) {
    if (wide) { fetch8(l4 + o, s0, s1); s0 = subb(top, s0); s1 = subb(top, s1); }
    else {
        const u32 bit = l4 * 8u + o * 4u, q = (bit >> 5) << 2;
        const u32 xx = __builtin_amdgcn_alignbit(ld32(q + 4), ld32(q), bit);
        const u32 lo = xx & 0x0f0f0f0fu, hi = (xx >> 4) & 0x0f0f0f0fu;
        s0 = perm(hi, lo, 0x05010400u); s1 = perm(hi, lo, 0x07030602u);
        const u32 g0 = ((s0 + 0x0e0e0e0eu) >> 4) & 0x01010101u, g1 = ((s1 + 0x0e0e0e0eu) >> 4) & 0x01010101u;
        s0 = top - s0 - ((g0 << 3) + g0); s1 = top - s1 - ((g1 << 3) + g1);
    }
}
template <bool LAST>
__device__ __forceinline__ void dec_level(u32 pl, u32 nb, u32 tab, u32 ts, u32 td, u32 top, bool wide, int lane) {
    u32 run = 0;
    for (u32 p = 0; p < nb; p += 64) {
        const u32 idx = p + lane;
        const u32 a8 = ldv8(pl + 2 * idx), b8 = ldv8(pl + 2 * idx + 1), more = (a8 | b8) ^ 255u;
        const u32x2 sel = ld64(L_LUTE + more * 8u);
        const u32 cnt = __popc(more), inc = scan(cnt), o = run + inc - cnt;
        run += last(inc);
        const u32 c0 = spread1(a8 & 15u) | spread2(b8 & 15u), c1 = spread1(a8 >> 4) | spread2(b8 >> 4);
        u32 s0, s1;
        if (!LAST) fetch8(ts + o, s0, s1);
        else fetch_l4(ts, o, top, wide, s0, s1);
        st64(td + idx * 8u, perm(0u, tab, c0) | perm(s1, s0, sel.x), perm(0u, tab, c1) | perm(s1, s0, sel.y));
    }
    CBAR();
}
__device__ __forceinline__ void dec_l23(u32 l2, u32 hdr, u32 base, u32 t2, int lane) {
    const u32 nb2 = min((hdr >> 11) & 511u, 256u), nb3 = min((hdr >> 20) & 511u, 256u), l3 = l2 + 2 * nb2;
    dec_level<true>(l3, nb3, ((base - 3) << 8) | ((base - 4) << 16) | ((base - 5) << 24), l3 + 2 * nb3, t2 + TB, (base + 5) * 0x01010101u, (hdr >> 29) & 1u, lane);
    dec_level<false>(l2, nb2, ((base + 3) << 8) | ((base - 1) << 16) | ((base - 2) << 24), t2 + TB, t2, 0u, false, lane);
}

__device__ __forceinline__ void dec_input(u32 rec, u32 hdr, u32 base, u32 tmp, char* dst, int lane) {
    const u32 t2 = tmp + 2 * TB;
    const u32 pa = ld32(rec + 2048 + lane * 4), pb = ld32(rec + 2304 + lane * 4), more = ~(pa | pb);
    u32 o[4]; offsets4(more, o);
    dec_l23(rec + 2560, hdr, base, t2, lane);
    const u32 tab = ((base + 2) << 8) | ((base + 1) << 16) | (base << 24);
    u32x4 r[4];
#pragma unroll
    for (int j = 0; j < 4; ++j) {
        const u32 m0 = ld32(rec + j * 512 + lane * 8), m1 = ld32(rec + j * 512 + lane * 8 + 4);
        const u32 c0 = spread1((pa >> (8 * j)) & 15u) | spread2((pb >> (8 * j)) & 15u);
        const u32 c1 = spread1((pa >> (8 * j + 4)) & 15u) | spread2((pb >> (8 * j + 4)) & 15u);
        u32 s0, s1; fetch8(t2 + o[j], s0, s1);
        const u32x2 sel = ld64(L_LUTE + ((more >> (8 * j)) & 255u) * 8u);
        const u32 e0 = perm(0u, tab, c0) | perm(s1, s0, sel.x), e1 = perm(0u, tab, c1) | perm(s1, s0, sel.y);
        const u32 h0 = bfi(0x80808080u, m0, e0 >> 1), h1 = bfi(0x80808080u, m1, e1 >> 1);
        const u32 l0 = bfi(0x80808080u, e0 << 7, m0), l1 = bfi(0x80808080u, e1 << 7, m1);
        r[j] = (u32x4){perm(h0, l0, 0x05010400u), perm(h0, l0, 0x07030602u), perm(h1, l1, 0x05010400u), perm(h1, l1, 0x07030602u)};
    }
#pragma unroll
    for (int j = 0; j < 4; ++j) gst_agent(dst + j * 1024, r[j]);
}

__device__ __forceinline__ void dec_output(u32 rec, u32 hdr, u32 base, u32 tmp, char* dst, int done, int lane) {
    const u32 t1l = tmp, t1h = tmp + TB, t2 = tmp + 2 * TB, t3 = t2 + TB;
    const u32 m = ld32(rec + lane * 4);
    u32 bb[3][2], a[3][2], a8[3];
#pragma unroll
    for (int p = 0; p < 3; ++p) { bb[p][0] = ld32(rec + 256 + p * 512 + lane * 8); bb[p][1] = ld32(rec + 260 + p * 512 + lane * 8); }
    SB();
    u32x2 selm[4];
#pragma unroll
    for (int j = 0; j < 4; ++j) selm[j] = ld64(L_LUTE + ((m >> (8 * j)) & 255u) * 8u);
    u32 o[4]; const u32 n1 = offsets4(m, o);
#pragma unroll
    for (int p = 0; p < 3; ++p) {
        a[p][0] = (bb[p][0] >> 7) & 0x01010101u; a[p][1] = (bb[p][1] >> 7) & 0x01010101u; a8[p] = gather8(a[p][0], a[p][1]);
        asm volatile("" : "+v"(a8[p]));
    }
    const u32 pbo = rec + 256 + n1, nb = (n1 + 7) >> 3;
    const u32 tab = ((base + 2) << 8) | ((base + 1) << 16) | (base << 24);
    const u32 nb2 = (hdr >> 11) & 511u, nb3 = (hdr >> 20) & 511u;
    if (nb <= 192u && nb2 <= 64u && nb3 <= 64u && !((hdr >> 29) & 1u)) {
        const u32 l2 = pbo + nb, l3 = l2 + 2 * nb2, l4 = l3 + 2 * nb3;
        const u32 a3 = ldv8(l3 + 2 * lane), b3 = ldv8(l3 + 2 * lane + 1), a2 = ldv8(l2 + 2 * lane), b2 = ldv8(l2 + 2 * lane + 1);
        u32 b8[3], more[3]; u32x2 sel[3];
#pragma unroll
        for (int p = 0; p < 3; ++p) b8[p] = ld8(pbo + p * 64 + lane);
        SB();
        const u32 more3 = (a3 | b3) ^ 255u, more2 = (a2 | b2) ^ 255u;
        const u32x2 sel3 = ld64(L_LUTE + more3 * 8u), sel2 = ld64(L_LUTE + more2 * 8u);
#pragma unroll
        for (int p = 0; p < 3; ++p) { more[p] = (a8[p] | b8[p]) ^ 255u; sel[p] = ld64(L_LUTE + more[p] * 8u); }
        const u32 cx = __popc(more3) | (__popc(more2) << 16), cy = __popc(more[0]) | (__popc(more[1]) << 10) | (__popc(more[2]) << 20);
        const u32 sx = scan(cx), sy = scan(cy), ty = last(sy), ex = sx - cx, ey = sy - cy;
        const u32 bit = l4 * 8u + (ex & 0xffffu) * 4u, q4 = (bit >> 5) << 2;
        const u32 n0 = ld32(q4), n1_ = ld32(q4 + 4);
        ctl_add1(done);
        SB();
        const u32 tab3 = ((base - 3) << 8) | ((base - 4) << 16) | ((base - 5) << 24), tab2 = ((base + 3) << 8) | ((base - 1) << 16) | ((base - 2) << 24);
        const u32 k30 = perm(0u, tab3, spread1(a3 & 15u) | spread2(b3 & 15u)), k31 = perm(0u, tab3, spread1(a3 >> 4) | spread2(b3 >> 4));
        SB();
        {
            const u32 top = (base + 5) * 0x01010101u, xx = __builtin_amdgcn_alignbit(n1_, n0, bit);
            const u32 lo = xx & 0x0f0f0f0fu, hi = (xx >> 4) & 0x0f0f0f0fu;
            u32 s0 = perm(hi, lo, 0x05010400u), s1 = perm(hi, lo, 0x07030602u);
            const u32 g0 = ((s0 + 0x0e0e0e0eu) >> 4) & 0x01010101u, g1 = ((s1 + 0x0e0e0e0eu) >> 4) & 0x01010101u;
            s0 = top - s0 - ((g0 << 3) + g0); s1 = top - s1 - ((g1 << 3) + g1);
            st64(t3 + lane * 8, k30 | perm(s1, s0, sel3.x), k31 | perm(s1, s0, sel3.y));
        }
        CBAR();
        Fetch f3; f3.ask(t3 + (ex >> 16));
        SB();
        const u32 k20 = perm(0u, tab2, spread1(a2 & 15u) | spread2(b2 & 15u)), k21 = perm(0u, tab2, spread1(a2 >> 4) | spread2(b2 >> 4));
        const u32 o20 = t2 + (ey & 1023u), o21 = t2 + (ty & 1023u) + ((ey >> 10) & 1023u), o22 = t2 + (ty & 1023u) + ((ty >> 10) & 1023u) + (ey >> 20);
        SB();
        { const u32 s0 = f3.lo(), s1 = f3.hi(); st64(t2 + lane * 8, k20 | perm(s1, s0, sel2.x), k21 | perm(s1, s0, sel2.y)); }
        CBAR();
        Fetch f[3]; f[0].ask(o20); f[1].ask(o21); f[2].ask(o22);
        SB();
        u32 k1[3][2];
#pragma unroll
        for (int p = 0; p < 3; ++p) { k1[p][0] = perm(0u, tab, a[p][0] | spread2(b8[p] & 15u)); k1[p][1] = perm(0u, tab, a[p][1] | spread2(b8[p] >> 4)); }
        SB();
#pragma unroll
        for (int p = 0; p < 3; ++p) {
            const u32 s0 = f[p].lo(), s1 = f[p].hi();
            const u32 e0 = k1[p][0] | perm(s1, s0, sel[p].x), e1 = k1[p][1] | perm(s1, s0, sel[p].y);
            st64(t1l + p * 512 + lane * 8, bfi(0x80808080u, e0 << 7, bb[p][0]), bfi(0x80808080u, e1 << 7, bb[p][1]));
            st64(t1h + p * 512 + lane * 8, (e0 >> 1) & 0x7f7f7f7fu, (e1 >> 1) & 0x7f7f7f7fu);
        }
    } else {
        dec_l23(pbo + nb, hdr, base, t2, lane);
        u32 run = 0;
        for (u32 p = 0; p < nb; p += 64) {
            const u32 idx = p + lane;
            const u32 b0 = ld32(rec + 256 + idx * 8u), b1 = ld32(rec + 260 + idx * 8u);
            const u32 b8 = ld8(pbo + idx);
            const u32 a0 = (b0 >> 7) & 0x01010101u, a1 = (b1 >> 7) & 0x01010101u;
            const u32 more = (gather8(a0, a1) | b8) ^ 255u;
            const u32x2 sel = ld64(L_LUTE + more * 8u);
            const u32 cnt = __popc(more), inc = scan(cnt), o2 = run + inc - cnt;
            run += last(inc);
            const u32 c0 = a0 | spread2(b8 & 15u), c1 = a1 | spread2(b8 >> 4);
            u32 s0, s1; fetch8(t2 + o2, s0, s1);
            const u32 e0 = perm(0u, tab, c0) | perm(s1, s0, sel.x), e1 = perm(0u, tab, c1) | perm(s1, s0, sel.y);
            st64(t1l + idx * 8u, bfi(0x80808080u, e0 << 7, b0), bfi(0x80808080u, e1 << 7, b1));
            st64(t1h + idx * 8u, (e0 >> 1) & 0x7f7f7f7fu, (e1 >> 1) & 0x7f7f7f7fu);
        }
        CBAR();
        ctl_add1(done);
    }
    CBAR();
    Fetch fl[4], fh[4];
#pragma unroll
    for (int j = 0; j < 4; ++j) { fl[j].ask(t1l + o[j]); fh[j].ask(t1h + o[j]); }
    SB();
#pragma unroll
    for (int j = 0; j < 4; ++j) {
        const u32 l0 = fl[j].lo(), l1 = fl[j].hi(), h0 = fh[j].lo(), h1 = fh[j].hi();
        const u32 L0 = perm(l1, l0, selm[j].x), L1 = perm(l1, l0, selm[j].y), H0 = perm(h1, h0, selm[j].x), H1 = perm(h1, h0, selm[j].y);
        gst_agent(dst + (size_t)j * (8 * NFULL * 2), (u32x4){perm(H0, L0, 0x05010400u), perm(H0, L0, 0x07030602u), perm(H1, L1, 0x05010400u), perm(H1, L1, 0x07030602u)});
    }
}

__device__ __forceinline__ void enc_l234(u32 rec, u32 l2, u32 n2, u32 base, u32 t2, int wid, int lane) {
    const u32 nb2 = (n2 + 7) >> 3, t3 = t2 + TB;
    u32 n3 = 0;
    for (u32 p = 0; p < nb2; p += 64) {
        const u32 idx = p + lane;
        const u32x2 g = ld64(t2 + idx * 8u);
        const int left = (int)n2 - (int)(idx * 8u);
        const u32 k0 = geb<6>(g.x) ^ 0x01010101u, k1 = geb<6>(g.y) ^ 0x01010101u;
        const u32 a8 = gather8((g.x | ~(g.x >> 2)) & k0, (g.y | ~(g.y >> 2)) & k1), b8 = gather8((g.x >> 2) & k0, (g.y >> 2) & k1);
        const u32 more = ~(a8 | b8) & valid8(left);
        const u32x2 sel = ld64(L_LUTC + more * 8u);
        if (left > 0) { st8(l2 + 2 * idx, a8); st8(l2 + 2 * idx + 1, b8); }
        const u32 cnt = __popc(more), inc = scan(cnt), o = n3 + inc - cnt;
        n3 += last(inc);
        scatter8(t3 + o, perm(g.y, g.x, sel.x), perm(g.y, g.x, sel.y), lane);
    }
    CBAR();
    const u32 nb3 = (n3 + 7) >> 3, l3 = l2 + 2 * nb2, l4 = l3 + 2 * nb3;
    u32 n4;
    bool wide = false;
    for (;;) {
        if (lane < 16) st8(l4 + lane, 0u);
        zero_lds((l4 + 15u) & ~15u, (wide ? n3 : n3 >> 1) + 8u, lane);
        CBAR();
        u32 out = 0;
        n4 = 0;
        for (u32 p = 0; p < nb3; p += 64) {
            const u32 idx = p + lane;
            const u32x2 g = ld64(t3 + idx * 8u);
            const int left = (int)n3 - (int)(idx * 8u);
            const u32 k0 = geb<9>(g.x) ^ 0x01010101u, k1 = geb<9>(g.y) ^ 0x01010101u;
            const u32 t0 = (g.x & 0x0f0f0f0fu) + 0x03030303u, t1 = (g.y & 0x0f0f0f0fu) + 0x03030303u;
            const u32 a8 = gather8(t0 & k0, t1 & k1), b8 = gather8((t0 >> 1) & k0, (t1 >> 1) & k1);
            const u32 more = ~(a8 | b8) & valid8(left);
            const u32x2 sel = ld64(L_LUTC + more * 8u);
            if (left > 0) { st8(l3 + 2 * idx, a8); st8(l3 + 2 * idx + 1, b8); }
            const u32 cnt = __popc(more), inc = scan(cnt), o = n4 + inc - cnt;
            n4 += last(inc);
            const u32 u0 = ((g.x & 0x7f7f7f7fu) + 0x02020202u) ^ (g.x & 0x80808080u), u1 = ((g.y & 0x7f7f7f7fu) + 0x02020202u) ^ (g.y & 0x80808080u);
            u32 c0 = perm(u1, u0, sel.x), c1 = perm(u1, u0, sel.y);
            if (wide) scatter8(l4 + o, c0, c1, lane);
            else {
                out |= geb<25>(c0) | geb<25>(c1);
                const u32 q0 = ((c0 + 0x05050505u) >> 4) & 0x01010101u, q1 = ((c1 + 0x05050505u) >> 4) & 0x01010101u;
                c0 -= (q0 << 3) + q0; c1 -= (q1 << 3) + q1;
                const u32 xx = perm(c1 | (c1 >> 4), c0 | (c0 >> 4), 0x06040200u);
                const u32 bit = l4 * 8u + o * 4u, q = (bit >> 5) << 2;
                const u64 w = (u64)xx << (bit & 31u);
                or32nz(q, (u32)w, lane); or32nz(q + 4, (u32)(w >> 32), lane);
            }
        }
        CBAR();
        if (wide || !__any(out != 0)) break;
        wide = true;
    }
    const u32 end = l4 + (wide ? n4 : (n4 + 1) >> 1);
    if (lane == 0) {
        *ctl(C_HDR + wid) = ((end - rec + 3) >> 2) | (nb2 << 11) | (nb3 << 20) | (wide ? 1u << 29 : 0u);
        st8(L_CTL + 4 * (C_HDR + 8) + wid, base);
    }
}

__device__ __forceinline__ void classify4(u32 v0, u32 v1, u32 k, u32& g, u32& c) {
    const u32 g0 = pk_nmad2(v0, k), g1 = pk_nmad2(v1, k);
    g = perm(g1, g0, 0x07050301u);
    c = perm(pk_min(g1, 0x04000400u), pk_min(g0, 0x04000400u), 0x07050301u);
}

__device__ __forceinline__ void enc_input(const u32x4 (&v)[4], int wid, int lane) {
    const u32 rec = L_RB + wid * ENC_W, t2 = rec + RECB + 2 * TB;
    u32 sum = 0;
#pragma unroll
    for (int j = 0; j < 4; ++j) {
        const u32 l0 = perm(v[j].y, v[j].x, 0x06040200u), h0 = perm(v[j].y, v[j].x, 0x07050301u);
        const u32 l1 = perm(v[j].w, v[j].z, 0x06040200u), h1 = perm(v[j].w, v[j].z, 0x07050301u);
        st64(rec + j * 512 + lane * 8, bfi(0x80808080u, h0, l0), bfi(0x80808080u, h1, l1));
        sum = __builtin_amdgcn_udot4(h0 & 0x7f7f7f7fu, 0x01010101u, sum, false);
    }
    const u32 base = uni((u32)min(max(((int)last(scan(sum)) + 164) >> 9, 19), 250));
    const u32 k = (((base + 3) << 8) | 0xffu) * 0x00010001u;
    u32 g[4][2], pa = 0, pb = 0;
#pragma unroll
    for (int j = 0; j < 4; ++j) {
        u32 c0, c1;
        classify4(v[j].x, v[j].y, k, g[j][0], c0); classify4(v[j].z, v[j].w, k, g[j][1], c1);
        pa |= gather8(c0 & 0x01010101u, c1 & 0x01010101u) << (8 * j);
        const u32 b2 = gather8(c0 & 0x02020202u, c1 & 0x02020202u);
        pb |= j ? b2 << (8 * j - 1) : b2 >> 1;
    }
    st32(rec + 2048 + lane * 4, pa); st32(rec + 2304 + lane * 4, pb);
    const u32 more = ~(pa | pb);
    u32x2 sel[4];
#pragma unroll
    for (int j = 0; j < 4; ++j) sel[j] = ld64(L_LUTC + ((more >> (8 * j)) & 255u) * 8u);
    u32 o[4]; const u32 n2 = offsets4(more, o);
    for (u32 i = lane * 16; i < n2; i += 1024) { const u32x4 z = {0u, 0u, 0u, 0u}; st128(t2 + i, z); st128(t2 + TB + i, z); }
    CBAR();
#pragma unroll
    for (int j = 0; j < 4; ++j)
        scatter8(t2 + o[j], perm(g[j][1], g[j][0], sel[j].x), perm(g[j][1], g[j][0], sel[j].y), lane);
    CBAR();
    enc_l234(rec, rec + 2560, n2, base, t2, wid, lane);
}

__device__ __forceinline__ void enc_output1(const u32x4 (&v)[4], int wid, int lane, u32& n1, u32& base) {
    const u32 rec = L_RB + wid * ENC_W, t1 = rec + RECB, t2 = t1 + 2 * TB;
    u32x4 z[4];
    u32 mm[2] = {0u, 0u}, sum = 0;
#pragma unroll
    for (int j = 0; j < 4; ++j)
#pragma unroll
        for (int e = 0; e < 4; ++e) {
            const u32 w = 0x00020001u << (8 * (j & 1) + 2 * e);
            const u32 nz = pk_nz(v[j][e]);
            z[j][e] = nz;
            mm[j >> 1] = __builtin_amdgcn_udot2(__builtin_bit_cast(u16x2, nz), __builtin_bit_cast(u16x2, w), mm[j >> 1], false);
            sum = __builtin_amdgcn_sad_u16(v[j][e], 0u, sum);
        }
    const u32 m = mm[0] | (mm[1] << 16);
    st32(rec + lane * 4, m);
    u32 o[4]; n1 = offsets4(m, o);
    const u32 tot = last(scan(sum));
    for (u32 i = lane * 16; i < n1; i += 1024) { const u32x4 z0 = {0u, 0u, 0u, 0u}; st128(t2 + i, z0); st128(t2 + TB + i, z0); }
#pragma unroll
    for (int j = 0; j < 4; ++j) compact16(t1 + 2 * o[j], v[j], z[j]);
    base = uni((u32)min(max((int)((float)tot * __builtin_amdgcn_rcpf((float)max(n1, 1u)) * (1.0f / 128.0f) - 0.64f), 19), 250));
}
__device__ __forceinline__ void enc_output2(u32 n1, u32 base, int wid, int lane) {
    const u32 rec = L_RB + wid * ENC_W, t1 = rec + RECB, t2 = t1 + 2 * TB;
    const u32 k = (((base + 3) << 8) | 0xffu) * 0x00010001u;
    const u32 nb = (n1 + 7) >> 3, pbo = rec + 256 + n1;
    u32 run = 0, pbs = 0;
#pragma unroll
    for (int p = 0; p < 4; ++p) {
        if ((u32)p * 64u < nb) {
            const u32 idx = p * 64 + lane;
            const u32x4 x = ld128(t1 + idx * 16u);
            const int left = (int)n1 - (int)(idx * 8u);
            u32 g0, g1, c0, c1;
            classify4(x.x, x.y, k, g0, c0); classify4(x.z, x.w, k, g1, c1);
            const u32 a8 = gather8(c0 & 0x01010101u, c1 & 0x01010101u), b8 = gather8(c0 & 0x02020202u, c1 & 0x02020202u) >> 1;
            const u32 more = ~(a8 | b8) & valid8(left);
            const u32x2 sel = ld64(L_LUTC + more * 8u);
            if (left > 0) st64(rec + 256 + idx * 8u, bfi(0x80808080u, c0 << 7, perm(x.y, x.x, 0x06040200u)), bfi(0x80808080u, c1 << 7, perm(x.w, x.z, 0x06040200u)));
            pbs |= b8 << (8 * p);
            const u32 cnt = __popc(more), inc = scan(cnt), o2 = run + inc - cnt;
            run += last(inc);
            scatter8(t2 + o2, perm(g1, g0, sel.x), perm(g1, g0, sel.y), lane);
        }
    }
    CBAR();
#pragma unroll
    for (int p = 0; p < 4; ++p)
        if ((u32)(p * 64 + lane) < nb) st8(pbo + p * 64 + lane, pbs >> (8 * p));
    CBAR();
    enc_l234(rec, pbo + nb, run, base, t2, wid, lane);
}

__device__ __forceinline__ u32 record_ends(u32 hdr) {
    u32 inc = hdr & 0x7ffu;
    inc += __builtin_amdgcn_update_dpp(0u, inc, 0x111, 0xf, 0xf, false);
    inc += __builtin_amdgcn_update_dpp(0u, inc, 0x112, 0xf, 0xf, false);
    inc += __builtin_amdgcn_update_dpp(0u, inc, 0x114, 0xf, 0xf, false);
    return inc + 10u;
}
__device__ __forceinline__ void gst32_sys(void* p, u32 v) { asm volatile("global_store_dword %0, %1, off sc0 sc1" ::"v"(p), "v"(v) : "memory"); }

__device__ __forceinline__ u32 emit_block(char* slot, int ls, int wid, int lane) {
    const u32 rec = L_RB + wid * ENC_W;
    CBAR();
    const u32x4 r0 = ld128(rec + lane * 16), r1 = ld128(rec + 1024 + lane * 16), r2 = ld128(rec + 2048 + lane * 16);
    wg_barrier();
    const u32 hw = *ctl(C_HDR + (lane & 15));
    const u32 ends = record_ends(hw);
    const int sh = 3 - ls, w0 = wid >> sh << sh;
    const u32 mysz = __builtin_amdgcn_readlane(hw, wid) & 0x7ffu, full = mysz >> 2;
    const u32 skip = __builtin_amdgcn_readlane(ends, w0) - (__builtin_amdgcn_readlane(hw, w0) & 0x7ffu) - 10u;
    slot += (size_t)(wid >> sh) * (SLOT >> ls);
    char* d = slot + (size_t)(__builtin_amdgcn_readlane(ends, wid) - mysz - skip) * 4;
    if ((u32)lane < full) gst_sys(d + lane * 16, r0);
    if ((u32)lane + 64u < full) gst_sys(d + 1024 + lane * 16, r1);
    if ((u32)lane + 128u < full) gst_sys(d + 2048 + lane * 16, r2);
    for (u32 i = 192 + lane; i < full; i += 64) gst_sys(d + (size_t)i * 16, ld128(rec + i * 16));
    if ((u32)lane < (mysz & 3u)) gst32_sys(d + full * 16 + lane * 4, ld32(rec + full * 16 + lane * 4));
    if (wid == w0 && lane < 10) gst32_sys(slot + lane * 4, lane >= 8 || (lane >> sh << sh) == w0 ? hw : 0u);
    const u32 e1 = __builtin_amdgcn_ds_bpermute((((lane + 1) << sh) - 1) << 2, ends), e0 = __builtin_amdgcn_ds_bpermute(((lane << sh) - 1) << 2, ends);
    return (e1 - (lane ? e0 - 10u : 0u) + 3u) >> 2;
}

__device__ __forceinline__ void flag_stage(char* ws, int rel, int pos, int n, u32 val, int lane) {
    if (lane < n) asm volatile("global_store_dword %0, %1, off sc1" ::"v"(ws + OFF_LEN + (rel * 1024 + pos + lane) * 4), "v"(val) : "memory");
}
__device__ __forceinline__ void flag_send(char* ws, char* wsl, int rank, int rel, int pos, u32 n, int lane) {
    u32 t = 0;
    if (lane == 0) t = __hip_atomic_fetch_add((GAS u32*)(ws + OFF_GRP + (rel * 128 + (pos >> 3)) * 64), n, __ATOMIC_RELAXED, __HIP_MEMORY_SCOPE_AGENT);
    if (((uni(t) + n) & 7u) == 0u) {
        const int q = (pos & ~7) + (lane & 7);
        const u32 f = __hip_atomic_load((const GAS u32*)(ws + OFF_LEN) + (rel * 1024 + q), __ATOMIC_RELAXED, __HIP_MEMORY_SCOPE_AGENT);
        if (rel ? (lane >> 3) == rel : lane >= 8) __hip_atomic_store((GAS u32*)(wsl + OFF_FL) + (rank * 1024 + q), f, __ATOMIC_RELAXED, __HIP_MEMORY_SCOPE_SYSTEM);
    }
    JOIN();
}
__device__ __forceinline__ void ready_add(char* ws, int idx, int local) {
    __hip_atomic_fetch_add((GAS u32*)(ws + OFF_RDY + idx * 128 + local * 64), 1u, __ATOMIC_RELAXED, __HIP_MEMORY_SCOPE_AGENT);
}
#define STAMP(i) do {} while (0)
#define STAMPW(i) do {} while (0)

__device__ __forceinline__ void build_tables(int t) {
    u64 ee = 0x0c0c0c0c0c0c0c0cull, cc = ee; int k = 0;
#pragma unroll
    for (int i = 0; i < 8; ++i)
        if ((t >> i) & 1) { ee ^= (u64)(0x0c ^ k) << (8 * i); cc ^= (u64)(0x0c ^ i) << (8 * k); ++k; }
    st64(L_LUTE + t * 8, (u32)ee, (u32)(ee >> 32)); st64(L_LUTC + t * 8, (u32)cc, (u32)(cc >> 32));
}
__device__ __forceinline__ u32 draw_call(char* ws) {
    return __hip_atomic_fetch_add((GAS u32*)(ws + OFF_CTR + blockIdx.x * 64), 1u, __ATOMIC_RELAXED, __HIP_MEMORY_SCOPE_AGENT) + 1u;
}

__device__ __forceinline__ bool pos_is_x(int pos) { return pos < 32 || (pos >= 64 && pos < 144) || pos >= 176; }
__device__ __forceinline__ int pos_xb(int pos) { return pos < 32 ? pos : (pos < 144 ? pos - 32 : pos - 64); }
__device__ __forceinline__ int pos_wb(int pos) { return pos < 64 ? pos - 32 : pos - 112; }
__device__ __forceinline__ void task_of(int task, int& pos, int& rel) {
    rel = -1;
    if (task < 32) pos = task;
    else if (task < 288) { pos = 32 + ((task - 32) >> 3); rel = (task + 1) & 7; }
    else if (task < 368) pos = task - 224;
    else if (task < 624) { pos = 144 + ((task - 368) >> 3); rel = (task + 1) & 7; }
    else pos = 176 + task - 624;
}
__device__ __forceinline__ void pack_addr(const Params& P, int task, int wid, int lane, const char*& s, size_t& s1, size_t& s2) {
    int pos, rel; task_of(task, pos, rel);
    if (rel < 0) {
        const int xb = pos_xb(pos);
        s = P.x + (size_t)((xb >> 3) * 256 + wid * 32 + (lane & 15)) * 1024 + (xb & 7) * 128 + (lane >> 4) * 16;
        s1 = 64; s2 = 16384;
    } else {
        const int wb = pos_wb(pos);
        s = P.w + (size_t)((wb & 3) * 128 + (wid >> 2) * 64 + (lane >> 2)) * (NFULL * 2) + (((P.rank + rel) & 7) * 2048 + (wb >> 2) * 128 + (wid & 3) * 32) * 2 + (lane & 3) * 16;
        s1 = 16 * NFULL * 2; s2 = 2 * s1;
    }
}
__device__ __forceinline__ void load4(const char* s, size_t s1, size_t s2, u32x4 (&t)[4]) {
    t[0] = *(const GAS u32x4*)s; t[1] = *(const GAS u32x4*)(s + s1); t[2] = *(const GAS u32x4*)(s + s2); t[3] = *(const GAS u32x4*)(s + s1 + s2);
}
__device__ __forceinline__ void pack_task(const Params& P, char* ws, char* wsl, int task, int next, u32 call, int wid, int lane, u32x4 (&v)[4]) {
    int pos, rel; task_of(task, pos, rel);
    size_t lo, so; int rdy;
    if (rel < 0) {
        const int xb = pos_xb(pos);
        rdy = xb >> 3; lo = OFF_XA + (size_t)(rdy * 64 + (xb & 7)) * 32768; so = OFF_CX + (size_t)xb * SLOT;
    } else {
        const int wb = pos_wb(pos);
        rdy = 16 + (wb >> 2); lo = OFF_WB + (size_t)((wb >> 2) * 64 + (wb & 3) * 2) * 16384; so = OFF_CW + (size_t)(rel * 64 + wb) * SLOT;
        const u32 tr = L_RB + wid * ENC_W + RECB;
#pragma unroll
        for (int i = 0; i < 4; ++i) st128(tr + i * 1024 + lane * 16, v[i]);
        CBAR();
        const u32 ra = tr + (lane >> 4) * 512 + (((lane >> 2) & 3) * 8 + (lane & 3)) * 2;
#pragma unroll
        for (int j = 0; j < 4; ++j) {
            u32 q[8];
#pragma unroll
            for (int e = 0; e < 8; ++e) q[e] = *(LAS unsigned short*)lp(ra + (j & 1) * 2048 + e * 64 + (j >> 1) * 8);
            v[j] = (u32x4){q[0] | (q[1] << 16), q[2] | (q[3] << 16), q[4] | (q[5] << 16), q[6] | (q[7] << 16)};
        }
        CBAR();
    }
    const bool local = rel <= 0, packed = rel != 0 && pos >= NRAW;
    if (local) {
        char* d = ws + lo + wid * 4096 + lane * 16;
#pragma unroll
        for (int j = 0; j < 4; ++j) gst_agent(d + j * 1024, v[j]);
    }
    u32 len = 0;
    if (packed) {
        enc_input(v, wid, lane); len = uni(emit_block(ws + so, 0, wid, lane));
        if (wid == 0) flag_stage(ws, max(rel, 0), pos, 1, (call << 12) | len, lane);
    }
    if (packed && pos < NSOLO) {
        vm_drain();
        wg_barrier();
        if (wid == 0 && lane >= 8 && (lane & 7) == 0) gst32_sys(wsl + OFF_FL + (P.rank * 1024 + pos) * 4, (call << 12) | len);
        JOIN();
        STAMP(2);
        wg_barrier();
    }
    CBAR();
    { const char* s0; size_t s1, s2; pack_addr(P, min(next, NTASK - 1), wid, lane, s0, s1, s2); load4(s0, s1, s2, v); }
    asm volatile("s_waitcnt vmcnt(4)" ::: "memory");
    wg_barrier();
    if (wid == 0) {
        if (packed && pos >= NSOLO) flag_send(ws, wsl, P.rank, max(rel, 0), pos, 1u, lane);
        if (local && lane == 0) ready_add(ws, rdy, 1);
    }
    JOIN();
}

__device__ __forceinline__ void tile_of(int set, int t16, int& pm, int& pn) {
    const int pair = set < 7 ? set : (set < 14 ? set - 7 : 7), half = set < 7 ? 0 : (set < 14 ? 1 : set & 1);
    pm = pair < 2 ? t16 & 3 : 2 * pair + (t16 >> 3); pn = 8 * half + (pair < 2 ? 4 * pair + (t16 >> 2) : t16 & 7);
}

__device__ __forceinline__ void decoder(const Params& P, char* ws, int slab, int d, int g, int wid, int lane, const u32x4 (&raw)[4]) {
    const int src = (P.rank + slab) & 7;
    char* sws = P.ws[slab & 7];
    const u64 tstart = now();
    GAS u32* tmo = (GAS u32*)(ws + OFF_TMO);
    if (wid == 1) {
        *ctl(lane) = 0u;
        wg_barrier();
        u32 call = 0;
        if (lane == 0) call = draw_call(ws);
        call = uni(call);
        asm volatile("" ::"v"(raw[0]), "v"(raw[1]), "v"(raw[2]), "v"(raw[3]));
        *ctl(C_CALL) = call;
        STAMPW(54);
        const u32 seq = call & 0xfffffu;
        const GAS u32* fl = (const GAS u32*)(ws + OFF_FL) + src * 1024;
        bool dead = false;
        u32 pos = NRAW + d;
        for (u32 i = 0;; ++i) {
            u32 f = ~(seq << 12);
            if (i) {
                while (ctl_ld(C_PROG) < i) { if (now() - tstart > 2 * DEADLINE) break; __builtin_amdgcn_s_sleep(2); }
                const u32 w0 = pos + 1u;
                const u32 win = __hip_atomic_load(fl + min(w0 + lane, 1023u), __ATOMIC_RELAXED, __HIP_MEMORY_SCOPE_SYSTEM);
                u32 t = 0;
                if (lane == 0) t = __hip_atomic_fetch_add((GAS u32*)(ws + OFF_CLM + src * 128), 1u, __ATOMIC_RELAXED, __HIP_MEMORY_SCOPE_AGENT);
                pos = uni(t) - (call - 1u) * (u32)(NPOS - NRAW) + (u32)(NRAW + DPP_);
                if (pos - w0 < 64u) f = __builtin_amdgcn_readlane(win, pos - w0);
            }
            if (pos >= (u32)NPOS) { *ctl(C_QUEUE + (i & 7u)) = ((i + 1u) << 22) | (1023u << 12); break; }
            for (u32 n = 1; !dead && (f >> 12) != seq; ++n) {
                f = uni(__hip_atomic_load(fl + pos, __ATOMIC_RELAXED, __HIP_MEMORY_SCOPE_SYSTEM));
                if ((f >> 12) == seq) break;
                if (n & 15u) continue;
                if (uni(__hip_atomic_load(tmo, __ATOMIC_RELAXED, __HIP_MEMORY_SCOPE_AGENT)) == call || now() - tstart > DEADLINE) {
                    __hip_atomic_store(tmo, call, __ATOMIC_RELAXED, __HIP_MEMORY_SCOPE_AGENT);
                    *ctl(C_DEAD) = 1; dead = true; f = 0;
                }
                __builtin_amdgcn_s_sleep(1);
            }
            CBAR();
            *ctl(C_QUEUE + (i & 7u)) = ((i + 1u) << 22) | (pos << 12) | (dead ? 0u : f & 0xfffu);
        }
    } else if (wid == 0) {
        {
            const u32 o = lane < 32 ? (u32)OFF_XA + (u32)(8 - slab) * 262144u : (u32)OFF_FL;
            const u32 w0 = __hip_atomic_load((const GAS u32*)(sws + o), __ATOMIC_RELAXED, __HIP_MEMORY_SCOPE_SYSTEM);
            STAMP(0);
            wg_barrier();
            asm volatile("" ::"v"(w0), "v"(raw[0]), "v"(raw[1]), "v"(raw[2]), "v"(raw[3]));
        }
        __builtin_amdgcn_s_setprio(3);
        u32 prev = 0, cur = 0;
#define PUBLISH(t) do { const u32 t_ = (t); if ((int)(t_ - cur) > 0) { cur = t_; *ctl(C_LANDED) = t_; } } while (0)
#define DRAIN_STEP(n) do { asm volatile("s_waitcnt vmcnt(%0)" ::"n"(n) : "memory"); if ((prev & 63u) >= (n)) PUBLISH(prev - (n)); } while (0)
#define DRAIN_PUBLISH() do { DRAIN_STEP(7); DRAIN_STEP(6); DRAIN_STEP(5); DRAIN_STEP(4); DRAIN_STEP(3); DRAIN_STEP(2); DRAIN_STEP(1); vm_drain(); } while (0)
        u32 i = 0, qn = 0, occ0 = 0, occ1 = 0, occ2 = 0, occ3 = 0;
#define BLOCK_DONE(b1) ((b1) == 0u || ctl_ld(C_DONE + (((b1) - 1u) & 7u)) >= (((b1) + 7u) & ~7u))
        for (;; ++i) {
            u32 f = ctl_ld(C_QUEUE + (i & 7u));
            if ((f >> 22) != i + 1u) {
                DRAIN_PUBLISH();
                PUBLISH(i << 6);
                while (((f = ctl_ld(C_QUEUE + (i & 7u))) >> 22) != i + 1u) {
                    if (now() - tstart > 2 * DEADLINE) { f = 1023u << 12; break; }
                    __builtin_amdgcn_s_sleep(1);
                }
            }
            CBAR();
            const int pos = (f >> 12) & 1023u;
            if (pos >= NPOS) break;
            const u32 len = min(f & 0xfffu, (u32)(LSLOT / 16));
            const bool big = len > (u32)(LSLOT / 32);
            const u32 q = big ? (qn + 1u) & 2u : qn;
            qn = (q + (big ? 2u : 1u)) & 3u;
            const u32 oa = q == 0 ? occ0 : q == 1 ? occ1 : q == 2 ? occ2 : occ3, ob = big ? (q == 0 ? occ1 : occ3) : 0u;
            const u32 oc = i >= 6u ? i - 5u : 0u;
            if (!(BLOCK_DONE(oa) && BLOCK_DONE(ob) && BLOCK_DONE(oc))) {
                DRAIN_PUBLISH();
                PUBLISH(i << 6);
                while (!(BLOCK_DONE(oa) && BLOCK_DONE(ob) && BLOCK_DONE(oc))) { if (now() - tstart > 2 * DEADLINE) break; __builtin_amdgcn_s_sleep(1); }
            }
            CBAR();
            occ0 = q == 0 ? i + 1u : occ0; occ1 = q == 1 || (big && q == 0) ? i + 1u : occ1;
            occ2 = q == 2 ? i + 1u : occ2; occ3 = q == 3 || (big && q == 2) ? i + 1u : occ3;
            *ctl(C_SLOT + (i & 7u)) = L_RB + q * (LSLOT / 2);
            *ctl(C_PROG) = i + 1u;
            size_t so;
            if (pos < NIN) so = pos_is_x(pos) ? OFF_CX + (size_t)pos_xb(pos) * SLOT : OFF_CW + (size_t)((8 - slab) * 64 + pos_wb(pos)) * SLOT;
            else so = pos < PSPLIT ? OFF_CO + (size_t)(pos - NIN) * SLOT : OFF_CO + (size_t)(NOUT - NSB) * SLOT + (size_t)(pos - PSPLIT) * (SLOT >> LS);
            const bool tail = pos >= NPOS - NTAIL;
            const char* s = sws + so + lane * 16;
            const u32 lb = (u32)(size_t)lp(L_RB + q * (LSLOT / 2));
#define PIECES(Q) for (u32 k = 0; k * 64u < len; ++k) { \
                if (k * 64u + lane < len) \
                    asm volatile("s_mov_b32 m0, %1\n\ts_waitcnt vmcnt(%2)\n\tglobal_load_lds_dwordx4 %0, off sc0 sc1" \
                                 ::"v"(s + (size_t)k * 1024), "s"(lb + k * 1024u), "n"((Q) - 1) : "memory"); \
                const int c = (int)k - ((Q) - 1); \
                if (c >= 0) PUBLISH((i << 6) + c); \
                else if ((int)(prev & 63u) + c >= 0) PUBLISH(prev + c); }
            if (tail) { PIECES(QT) } else { PIECES(QD) }
#undef PIECES
            prev = (i << 6) + ((len + 63u) >> 6);
        }
        DRAIN_PUBLISH();
        *ctl(C_NBLK) = i + 1u;
        *ctl(C_LANDED) = 0xffffffffu;
        STAMPW(50);
    } else {
        const u32 tmp = L_DT + (wid - 2) * 4 * TB;
        {
            if (wid < 6) build_tables(threadIdx.x - 128);
            wg_barrier();
            if (wid - 2 < PW) {
                char* dst = sws + OFF_XA + (size_t)(8 - slab) * 262144 + g * 1024 + lane * 16;
#pragma unroll
                for (int j = 0; j < PS; ++j) { gst_sys(dst + j * 1024, raw[j]); if (j == 0) STAMPW(1); vm_drain(); }
                if (ctl_inc(C_PUSH) == PW - 1) {
                    u32 call;
                    while ((call = ctl_ld(C_CALL)) == 0u) { if (now() - tstart > 2 * DEADLINE) break; __builtin_amdgcn_s_sleep(1); }
                    if (lane == 0) __hip_atomic_store((GAS u32*)(sws + OFF_FLR) + (P.rank * 16 + d), call, __ATOMIC_RELAXED, __HIP_MEMORY_SCOPE_SYSTEM);
                    JOIN();
                    STAMPW(52);
                }
            }
        }
        for (;;) {
            const u32 r = ctl_inc(C_CLAIM);
            const u32 i = r >> 3, sub = r & 7u;
            bool late = false;
            while (ctl_ld(C_LANDED) < (i << 6) + 1u) { if (now() - tstart > 2 * DEADLINE) { late = true; break; } __builtin_amdgcn_s_sleep(1); }
            CBAR();
            const u32 vn = *ctl(C_NBLK), vq = *ctl(C_QUEUE + (i & 7u)), vs = *ctl(C_SLOT + (i & 7u)), vd = *ctl(C_DEAD);
            if (late || uni(vn) - 1u <= i) break;
            const int pos = min((uni(vq) >> 12) & 1023u, (u32)(NPOS - 1));
            if (pos >= PSPLIT && (sub >> (3 - LS)) != (u32)((pos - PSPLIT) & ((1 << LS) - 1))) { ctl_add1(C_DONE + (i & 7u)); continue; }
            const bool dead = uni(vd) != 0;
            const u32 slot = uni(vs);
            const u32 hw = ld32(slot + (lane & 7) * 4), base = uni(ld8(slot + 32 + sub));
            const u32 hdr = __builtin_amdgcn_readlane(hw, sub), en = __builtin_amdgcn_readlane(record_ends(hw), sub);
            const u32 endp = (i << 6) + min((en + 255u) >> 8, 63u);
            const u32 rec = slot + min(en - (hdr & 0x7ffu), (u32)(LSLOT / 4)) * 4u;
#define WAIT_RECORD() do { while (ctl_ld(C_LANDED) < endp) { if (now() - tstart > 2 * DEADLINE) break; } CBAR(); } while (0)
            if (pos < NIN) {
                size_t o; int rdy;
                if (pos_is_x(pos)) { const int xb = pos_xb(pos); rdy = xb >> 3; o = OFF_XA + (size_t)(rdy * 64 + slab * 8 + (xb & 7)) * 32768; }
                else { const int wb = pos_wb(pos); rdy = 16 + (wb >> 2); o = OFF_WB + (size_t)((wb >> 2) * 64 + slab * 8 + (wb & 3) * 2) * 16384; }
                WAIT_RECORD();
                if (!dead) dec_input(rec, hdr, base, tmp, ws + o + sub * 4096 + lane * 16, lane);
                ctl_add1(C_DONE + (i & 7u));
                vm_drain();
                if ((ctl_inc(C_STORED + (i & 15u)) & 7u) == 7u) { if (lane == 0) ready_add(ws, rdy, 0); JOIN(); }
            } else {
                const int ob = pos < PSPLIT ? pos - NIN : NOUT - NSB + ((pos - PSPLIT) >> LS);
                int pm, pn; tile_of(ob >> 5, ob & 15, pm, pn);
                char* dst = P.out + ((size_t)(pm * 256 + (sub >> 1) * 64 + ((ob >> 4) & 1) * 32 + (lane >> 3)) * NFULL + src * 2048 + pn * 128 + (sub & 1) * 64 + (lane & 7) * 8) * 2;
                asm volatile("" : "+v"(dst));
                WAIT_RECORD();
                if (!dead) dec_output(rec, hdr, base, tmp, dst, C_DONE + (i & 7u), lane);
                else { gst_agent(dst, (u32x4){0x7fc07fc0u, 0x7fc07fc0u, 0x7fc07fc0u, 0x7fc07fc0u}); ctl_add1(C_DONE + (i & 7u)); }
            }
        }
    }
}

__device__ __forceinline__ bool wait_operands(char* ws, int rank, int pm, int pn, u32 call, u32 mask, int lane) {
    const GAS u32* a = (const GAS u32*)(lane < 4 ? ws + OFF_RDY + ((lane & 1) ? 16 + pn : pm) * 128 + (lane < 2 ? 64 : 0) : ws + OFF_TMO);
    const int pushed = min(max(NRAW - 8 * pm, 0), 8);
    const u32 need = call * (lane == 0 ? 8u : lane == 1 ? 4u : lane == 2 ? 56u - 7u * pushed : 28u);
    const u64 t0 = now();
    u32 ok;
    for (;;) {
        const u32 v = __hip_atomic_load(a, __ATOMIC_RELAXED, __HIP_MEMORY_SCOPE_AGENT);
        ok = (u32)__ballot((int)(v - need) >= 0) & 15u;
        if (pushed) {
            const u64 f = __hip_atomic_load((const GAS u64*)(ws + OFF_FLR) + lane, __ATOMIC_RELAXED, __HIP_MEMORY_SCOPE_SYSTEM);
            if (!__all((lane >> 3) == rank || ((int)((u32)f - call) >= 0 && (int)((u32)(f >> 32) - call) >= 0))) ok &= ~4u;
        }
        const bool dead = __builtin_amdgcn_readlane(v, 4) == call;
        if (!dead && (ok & mask) == mask) break;
        if (dead || now() - t0 > DEADLINE) {
            __hip_atomic_store((GAS u32*)(ws + OFF_TMO), call, __ATOMIC_RELAXED, __HIP_MEMORY_SCOPE_AGENT);
            *ctl(C_DEAD) = 1; ok = 15u;
            break;
        }
        __builtin_amdgcn_s_sleep(4);
    }
    if (pushed) __builtin_amdgcn_fence(__ATOMIC_ACQUIRE, ""); else __builtin_amdgcn_fence(__ATOMIC_ACQUIRE, "agent");
    vm_drain();
    return ok == 15u;
}

__device__ __forceinline__ void gemm_tile(const Params& P, char* ws, char* wsl, int set, int t16, u32 call, int wid, int lane) {
    int pm, pn; tile_of(set, t16, pm, pn);
    bool have_all = true;
    if (wid == 0) have_all = wait_operands(ws, P.rank, pm, pn, call, 3u, lane);
    wg_barrier();
    STAMP(8 + (set >> 3) * 4);
    const char* ga = ws + OFF_XA + (size_t)pm * (2u << 20);
    const char* gb = ws + OFF_WB + (size_t)pn * (1u << 20);
    const u32 voa = wid * 4096 + lane * 16, vob = wid * 2048 + lane * 16;
    const u32 lbase = (u32)(size_t)lp(L_RB);
    const u32 lda = lbase + wid * 4096, ldb = lbase + 32768 + wid * 2048;
#define GLDS(vo, gp, lo, off) do { u32 keep_; asm volatile( \
        "s_mov_b32 %0, m0\n\ts_mov_b32 m0, %3\n\ts_nop 3\n\tglobal_load_lds_dwordx4 %1, %2 offset:" #off "\n\ts_mov_b32 m0, %0" \
        : "=&s"(keep_) : "v"(vo), "s"(gp), "s"(lo) : "memory"); } while (0)
#define GLDF(vo, gp, lo, off) asm volatile("s_mov_b32 m0, %2\n\ts_nop 0\n\tglobal_load_lds_dwordx4 %0, %1 offset:" #off ::"v"(vo), "s"(gp), "s"(lo) : "memory")
#define KA(kt) (ga + (size_t)(kt) * 32768)
#define KB(kt) (gb + (size_t)(kt) * 16384)
#define STAGE_LD(so, kt) do { const char* a_ = KA(kt); const char* b_ = KB(kt); \
        GLDS(voa, a_, lda + (so), 0); GLDS(voa, a_, lda + (so), 1024); GLDS(voa, a_, lda + (so), 2048); GLDS(voa, a_, lda + (so), 3072); \
        GLDS(vob, b_, ldb + (so), 0); GLDS(vob, b_, ldb + (so), 1024); } while (0)
    const u32 ra = L_RB + (wid >> 1) * 8192 + lane * 16, rb = L_RB + 32768 + (wid & 1) * 8192 + lane * 16;
#define FRAGS(A, B, so, kb) do { _Pragma("unroll") for (int q_ = 0; q_ < 4; ++q_) { \
        B[q_] = *(const LAS bf16x8*)lp(rb + (so) + q_ * 2048 + (kb) * 1024); A[q_] = *(const LAS bf16x8*)lp(ra + (so) + q_ * 2048 + (kb) * 1024); } } while (0)
#define MROW(A, B, m_) do { _Pragma("unroll") for (int n_ = 0; n_ < 4; ++n_) \
        acc[m_][n_] = __builtin_amdgcn_mfma_f32_16x16x32_bf16(B[n_], A[m_], acc[m_][n_], 0, 0, 0); } while (0)
#define MM1(A, B, m_, n_) acc[m_][n_] = __builtin_amdgcn_mfma_f32_16x16x32_bf16(B[n_], A[m_], acc[m_][n_], 0, 0, 0)
#define RDB(B, q_, kb) B[q_] = *(const LAS bf16x8*)lp(rb + so + (q_) * 2048 + (kb) * 1024)
#define RDA(A, q_, kb) A[q_] = *(const LAS bf16x8*)lp(ra + so + (q_) * 2048 + (kb) * 1024)
#define HALF(A, B, NA, NB, kb, G0, G1, G2) do { SB(); \
        MM1(A, B, 0, 0); SB(); RDA(NA, 0, kb); SB(); MM1(A, B, 0, 1); SB(); RDB(NB, 0, kb); SB(); MM1(A, B, 0, 2); SB(); RDB(NB, 1, kb); SB(); MM1(A, B, 0, 3); SB(); RDB(NB, 2, kb); SB(); \
        MM1(A, B, 1, 0); SB(); RDB(NB, 3, kb); SB(); MM1(A, B, 1, 1); SB(); RDA(NA, 1, kb); SB(); MM1(A, B, 1, 2); SB(); RDA(NA, 2, kb); SB(); MM1(A, B, 1, 3); SB(); RDA(NA, 3, kb); SB(); \
        MM1(A, B, 2, 0); SB(); G0; SB(); MM1(A, B, 2, 1); MM1(A, B, 2, 2); SB(); G1; SB(); MM1(A, B, 2, 3); \
        MM1(A, B, 3, 0); SB(); G2; SB(); MM1(A, B, 3, 1); MM1(A, B, 3, 2); MM1(A, B, 3, 3); SB(); } while (0)
#define MMA(A, B) do { MROW(A, B, 0); MROW(A, B, 1); MROW(A, B, 2); MROW(A, B, 3); } while (0)
    f32x4 acc[4][4];
#pragma unroll
    for (int m = 0; m < 4; ++m)
#pragma unroll
        for (int n = 0; n < 4; ++n) acc[m][n] = (f32x4){0.f, 0.f, 0.f, 0.f};
    bf16x8 a0[4], b0[4], a1[4], b1[4];
    STAGE_LD(0, 0); STAGE_LD(STAGE, 1);
    { const char* a_ = KA(2); GLDS(voa, a_, lda + 2 * STAGE, 0); GLDS(voa, a_, lda + 2 * STAGE, 1024); GLDS(voa, a_, lda + 2 * STAGE, 2048); }
    asm volatile("s_waitcnt vmcnt(9)\n\ts_barrier" ::: "memory");
    FRAGS(a0, b0, 0u, 0);
    u32 so = 0, sn, sp = 2 * STAGE;
    const char* pa = KA(2); const char* pb = KB(2);
#pragma unroll 1
    for (int phase = 0; phase < 2; ++phase) {
#pragma unroll 1
        for (int n = phase ? 56 : 5; n > 0; --n) {
            HALF(a0, b0, a1, b1, 1, GLDF(voa, pa, lda + sp, 3072), GLDF(vob, pb, ldb + sp, 0),
                 GLDF(vob, pb, ldb + sp, 1024); pa += 32768; pb += 16384; sp = so; sn = so == 2 * STAGE ? 0 : so + STAGE; asm volatile("" : "+s"(pa), "+s"(pb), "+s"(sn)));
            asm volatile("s_waitcnt vmcnt(6) lgkmcnt(0)\n\ts_barrier" ::: "memory");
            so = sn;
            HALF(a1, b1, a0, b0, 0, GLDF(voa, pa, lda + sp, 0), GLDF(voa, pa, lda + sp, 1024), GLDF(voa, pa, lda + sp, 2048));
        }
        if (phase == 0 && !have_all) { wait_operands(ws, P.rank, pm, pn, call, 15u, lane); STAMP(11 + (set >> 3) * 4); }
    }
    {
        const char* a2 = KA(63); const char* b2 = KB(63);
        GLDS(voa, a2, lda + sp, 3072); GLDS(vob, b2, ldb + sp, 0); GLDS(vob, b2, ldb + sp, 1024);
        FRAGS(a1, b1, so, 1); MMA(a0, b0);
        asm volatile("s_waitcnt vmcnt(6) lgkmcnt(0)\n\ts_barrier" ::: "memory");
        so = so == 2 * STAGE ? 0 : so + STAGE;
        FRAGS(a0, b0, so, 0); MMA(a1, b1);
        FRAGS(a1, b1, so, 1); MMA(a0, b0);
        asm volatile("s_waitcnt vmcnt(0) lgkmcnt(0)\n\ts_barrier" ::: "memory");
        so = so == 2 * STAGE ? 0 : so + STAGE;
        FRAGS(a0, b0, so, 0); MMA(a1, b1);
        FRAGS(a1, b1, so, 1); MMA(a0, b0); MMA(a1, b1);
    }
#undef GLDS
#undef KA
#undef KB
#undef STAGE_LD
#undef FRAGS
#undef MROW
#undef MMA
#undef MM1
#undef GLDF
#undef RDA
#undef RDB
#undef HALF
    wg_barrier();
    STAMP(9 + (set >> 3) * 4);
    const float floor0 = ctl_ld(C_DEAD) != 0 ? __builtin_nanf("") : 0.f;
    char* dst = P.out + ((size_t)(pm * 256 + (wid >> 1) * 64 + (lane >> 3)) * NFULL + P.rank * 2048 + pn * 128 + (wid & 1) * 64 + (lane & 7) * 8) * 2;
    const u32 img = L_RB + wid * ENC_W + RECB;
    int ls = 0, fpos = 0;
#pragma unroll
    for (int b = 0; b < 2; ++b) {
        u32x4 v[4];
#pragma unroll
        for (int j = 0; j < 4; ++j) {
            const f32x4 p = acc[2 * b + (j >> 1)][2 * (j & 1)], q = acc[2 * b + (j >> 1)][2 * (j & 1) + 1];
            float f[8] = {p[0], p[1], p[2], p[3], q[0], q[1], q[2], q[3]};
            u32 r[4];
#pragma unroll
            for (int e = 0; e < 4; ++e) {
                const float lo = __builtin_elementwise_maximum(f[2 * e], floor0), hi = __builtin_elementwise_maximum(f[2 * e + 1], floor0);
                asm("v_cvt_pk_bf16_f32 %0, %1, %2" : "=v"(r[e]) : "v"(lo), "v"(hi));
                r[e] &= 0x7fff7fffu;
            }
            st128(img + (j >> 1) * 2048 + (lane & 15) * 128 + ((((j & 1) * 4 + (lane >> 4)) ^ (lane & 7)) << 4), (u32x4){r[0], r[1], r[2], r[3]});
        }
        CBAR();
#pragma unroll
        for (int j = 0; j < 4; ++j) v[j] = ld128(img + j * 1024 + (lane >> 3) * 128 + (((lane ^ (lane >> 3)) & 7) << 4));
        CBAR();
        u32 n1, base;
        enc_output1(v, wid, lane, n1, base);
        if (b == 1) {
            asm volatile("s_waitcnt vmcnt(4)" ::: "memory");
            wg_barrier();
            if (wid == 0) flag_send(ws, wsl, P.rank, 0, fpos, 1u << ls, lane);
        }
        enc_output2(n1, base, wid, lane);
        const int ob = set * 32 + b * 16 + t16;
        ls = ob >= NOUT - NSB ? LS : 0;
        fpos = ls ? PSPLIT + ((ob - (NOUT - NSB)) << LS) : NIN + ob;
        const u32 len = emit_block(ws + OFF_CO + (size_t)ob * SLOT, ls, wid, lane);
        if (wid == 0) flag_stage(ws, 0, fpos, 1 << ls, (call << 12) | len, lane);
#pragma unroll
        for (int j = 0; j < 4; ++j) gst_agent(dst + (size_t)(4 * b + j) * (8 * NFULL * 2), v[j]);
    }
    asm volatile("s_waitcnt vmcnt(4)" ::: "memory");
    wg_barrier();
    if (wid == 0) flag_send(ws, wsl, P.rank, 0, fpos, 1u << ls, lane);
    STAMP(10 + (set >> 3) * 4);
}

__global__ void __launch_bounds__(512, 2) fused_gemm_ar(const Params P) {
    const int tid = threadIdx.x, lane = tid & 63, wid = uni(tid >> 6), blk = blockIdx.x;
    const int xcd = blk & 7, idx = blk >> 3;
    const bool is_dec = blk < NDEC;
    const int d = (idx >= 7 ? 8 : 0) + xcd, g = (d * PW + min(max(wid - 2, 0), PW - 1)) * PS;
    const bool urgent = is_dec || (u32)(blk - NDEC - NRAW) < 16u;
    {
        u64 pc;
        asm volatile("s_cmp_lg_u32 %2, 0\n\ts_cbranch_scc1 1f\n\ts_mov_b32 m0, %3\n\ts_getpc_b64 %0\n\tglobal_load_lds_dword %1, %0\n1:"
                     : "=&s"(pc) : "v"((u32)min(tid, 319) * 128u), "s"(uni(urgent)), "s"((u32)(size_t)lp(LDS_BYTES - 256)) : "memory", "scc");
    }
    const char* px = P.x;
    asm volatile("" : "+s"(px));
    const char* s0; size_t s1, s2, s3;
    if (__builtin_expect(urgent, 1)) {
        const u32 f = is_dec ? g : ((blk - NDEC) * 8 + wid) * 4;
#define XFRAG(f) (((f) >> 8) * 262144u + (((f) >> 1) & 15u) * 16384u + (((f) >> 5) & 7u) * 128u + ((f) & 1u) * 64u)
        const u32 o0 = XFRAG(f);
        s0 = px + (o0 + (u32)(lane & 15) * 1024u + (u32)(lane & 48));
        s1 = (size_t)(int)(XFRAG(f + 1) - o0); s2 = (size_t)(int)(XFRAG(f + 2) - o0); s3 = (size_t)(int)(XFRAG(f + 3) - o0);
#undef XFRAG
    } else {
        const u64 t0 = now();
        while (now() - t0 < WDELAY) __builtin_amdgcn_s_sleep(2);
        pack_addr(P, blk - NDEC, wid, lane, s0, s1, s2);
        s3 = s1 + s2;
    }
    u32x4 nx[4];
    nx[0] = *(const GAS u32x4*)s0; nx[1] = *(const GAS u32x4*)(s0 + s1); nx[2] = *(const GAS u32x4*)(s0 + s2); nx[3] = *(const GAS u32x4*)(s0 + s3);
    CBAR();
    char* ws = P.ws[0];
    if (is_dec) {
        decoder(P, ws, 1 + (idx >= 7 ? idx - 7 : idx), d, g, wid, lane, nx);
        return;
    }
    u32 cv;
    {
        u64 sv;
        asm volatile("s_cmp_lg_u32 %4, 0\n\ts_cbranch_scc1 1f\n\ts_mov_b64 %1, exec\n\ts_mov_b64 exec, 1\n\tglobal_atomic_add %0, %2, %3, off sc0\n\ts_mov_b64 exec, %1\n1:"
                     : "=&v"(cv), "=&s"(sv) : "v"(ws + OFF_CTR + blk * 64), "v"(1u), "s"(wid) : "memory", "scc");
    }
    STAMP(0);
    if (wid < 4) build_tables(tid);
    else if (wid == 4) *ctl(lane) = 0u;
    u32 wlo = 0, whi = 0;
#pragma unroll
    for (int p = 1; p < NR; ++p) {
        asm volatile("v_writelane_b32 %0, %2, %4\n\tv_writelane_b32 %1, %3, %4" : "+v"(wlo), "+v"(whi) : "s"((u32)(u64)P.ws[p]), "s"((u32)((u64)P.ws[p] >> 32)), "n"(p));
    }
    wlo = __builtin_amdgcn_ds_bpermute((lane >> 3) << 2, wlo); whi = __builtin_amdgcn_ds_bpermute((lane >> 3) << 2, whi);
    char* wsl = (char*)(((u64)whi << 32) | wlo);
    wg_barrier();
    u32 call = 0;
    const int late = idx >= 30 ? 96 + blk - 240 : (xcd >= 2 ? (idx - 14) * 6 + xcd - 2 : NTASK);
    for (int task = blk - NDEC; task < NTASK;) {
        const int next = task < 288 ? task + 144 : (task < 432 ? 432 + late : task + 112);
        asm volatile("s_waitcnt vmcnt(0)" : "+v"(cv), "+v"(nx[0]), "+v"(nx[1]), "+v"(nx[2]), "+v"(nx[3])::"memory");
        call = uni(cv) + 1u;
        pack_task(P, ws, wsl, task, next, call, wid, lane, nx);
        task = next;
    }
    asm volatile("s_waitcnt vmcnt(0)" : "+v"(nx[0]), "+v"(nx[1]), "+v"(nx[2]), "+v"(nx[3])::"memory");
    STAMP(1);
    if (idx < 30)
        for (int rr = 0; rr < 2; ++rr) gemm_tile(P, ws, wsl, xcd + 8 * rr, idx - 14, call, wid, lane);
    __builtin_amdgcn_s_waitcnt(0);
}

}

extern "C" void kernel_launch(int rank, int nranks, ncclComm_t comm, hipStream_t stream,
                              void* const* d_in, const int* in_sizes, int n_in,
                              void* d_out, int out_size, const int* in_dtypes, int out_dtype,
                              void* const* d_out_all, void* const* d_ws_all, size_t ws_bytes) {
    if (nranks != NR || (unsigned)rank >= (unsigned)NR || n_in != 2 || ws_bytes < WS_NEED ||
        in_sizes[0] != 4096 * 512 || in_sizes[1] != 512 * NFULL || out_size != 4096 * NFULL) {
        fprintf(stderr, "kernel_launch: unexpected problem shape\n");
        return;
    }
    Params P;
    P.x = (const char*)d_in[0]; P.w = (const char*)d_in[1]; P.out = (char*)d_out;
    for (int p = 0; p < NR; ++p) P.ws[p] = (char*)d_ws_all[(rank + p) % NR];
    P.rank = rank; P.pad = 0;
    hipLaunchKernelGGL(fused_gemm_ar, dim3(NWG), dim3(512), 0, stream, P);
    hipError_t e = hipGetLastError();
    if (e != hipSuccess) fprintf(stderr, "launch failed: %s\n", hipGetErrorString(e));
}
```
